```python
import numpy as np

kernel_name = "gpu_gemm_rs_m4096_k4096_n4096_f32_none"


ABSMAX_RTOL = 0.001

COMPUTE_DTYPE = "float32"
EPILOGUE = "none"
IN_QUANT = None
OUT_QUANT = None
IN_DTYPES = {"x": "float32", "w_mat": "float32"}
WORLD_SIZE = 8

def setup_inputs():

    x = np.random.randn(4096, 4096).astype(np.float32)
    w_mat = (np.random.randn(4096, 4096) / np.sqrt(4096)).astype(np.float32)
    return {"x": x, "w_mat": w_mat}

def _epilogue(y):
    return y

def reference(x, w_mat):
    y = (x @ w_mat).astype(np.float32)
    return _epilogue(y)
```

```cpp
#include <hip/hip_runtime.h>
#include <rccl/rccl.h>
#include <cstdio>

#define HIP_CHECK(expr) do { hipError_t _e = (expr); if (_e != hipSuccess) { \
    fprintf(stderr, "HIP error %s at %s:%d\n", hipGetErrorString(_e), __FILE__, __LINE__); \
    return; } } while (0)

#define RELAY 1
#define NSEND 32

typedef float f4 __attribute__((ext_vector_type(4)));
typedef float f16v __attribute__((ext_vector_type(16)));
typedef unsigned u4 __attribute__((ext_vector_type(4)));
typedef _Float16 h8 __attribute__((ext_vector_type(8)));
typedef __amdgpu_buffer_rsrc_t rsrc_t;

constexpr int NR = 8;
constexpr unsigned PIECE = 8192;
constexpr unsigned X_OFF = 0;
constexpr unsigned W_OFF = 8u << 20;
constexpr unsigned PAR_STRIDE = 24u << 20;
constexpr unsigned FLAG_OFF = 48u << 20;
constexpr unsigned FLAG_PAR = 16384;
constexpr unsigned YFLAG_OFF = FLAG_OFF + 32768;
constexpr unsigned CTR_OFF = FLAG_OFF + 36864;
constexpr unsigned TMO_OFF = FLAG_OFF + 40960;
constexpr unsigned DBG_OFF = FLAG_OFF + 65536;
constexpr unsigned WS_USED = FLAG_OFF + (1u << 20);
constexpr unsigned OUT_BYTES = 512u * 4096u * 4u;
constexpr unsigned IN_BYTES = 512u * 4096u * 4u;
constexpr unsigned long long DEADLINE = 50000000ull;
constexpr int AUX_SYS = 17;
#define AUX_LD 16
constexpr int XR = 17;
constexpr int NTHREADS = 704;

struct Args {
  const float* x;
  const float* w;
  char* ws[NR];
  float* out[NR];
  int rank;
};

struct __attribute__((aligned(16))) Smem {
  u4 xl[8][32 - XR][64];
  float slot[8][32 * 32];
  unsigned mirror[9][32];
  unsigned st[8];
  unsigned call;
  unsigned abort_;
};

__device__ __forceinline__ rsrc_t mk(const void* p, unsigned bytes) {
  return __builtin_amdgcn_make_buffer_rsrc((void*)p, (short)0, (int)bytes, 0x00020000);
}
__device__ __forceinline__ unsigned lds_ld(const unsigned* p) {
  return __builtin_amdgcn_readfirstlane(__hip_atomic_load(p, __ATOMIC_RELAXED, __HIP_MEMORY_SCOPE_WORKGROUP));
}
__device__ __forceinline__ void lds_st(unsigned* p, unsigned v) {
  __hip_atomic_store(p, v, __ATOMIC_RELAXED, __HIP_MEMORY_SCOPE_WORKGROUP);
}
__device__ __forceinline__ unsigned long long now() { return __builtin_amdgcn_s_memrealtime(); }

#define STAMP(slot) do {} while (0)

__device__ __forceinline__ unsigned wait_mirror(Smem& sm, const unsigned* p, unsigned call, unsigned long long t0) {
  unsigned v = lds_ld(p);
  while ((v >> 8) != call) {
    if (lds_ld(&sm.abort_) != 0u) break;
    if (now() - t0 > 2 * DEADLINE) { lds_st(&sm.abort_, 1u); break; }
    __builtin_amdgcn_s_sleep(1);
    v = lds_ld(p);
  }
  asm volatile("" ::: "memory");
  return v;
}
__device__ __forceinline__ void wait_lds_zero(Smem& sm, const unsigned* p, unsigned long long t0) {
  while (lds_ld(p) != 0u) {
    if (now() - t0 > 3 * DEADLINE) { lds_st(&sm.abort_, 1u); break; }
    __builtin_amdgcn_s_sleep(1);
  }
  asm volatile("" ::: "memory");
}

__device__ __forceinline__ unsigned absbits(float v) { return __builtin_bit_cast(unsigned, v) & 0x7fffffffu; }

__device__ __forceinline__ unsigned wave_code(unsigned m) {
  unsigned c = ((m >> 23) + 9u) >> 3;
  unsigned r = 0;
#pragma unroll
  for (unsigned b = 32; b; b >>= 1)
    if (__builtin_amdgcn_ballot_w64(c >= (r | b)) != 0ull) r |= b;
  return r;
}

__global__ void __launch_bounds__(NTHREADS) fused_gemm_kernel(Args a) {
  __shared__ Smem sm;
  const int tid = threadIdx.x;
  const int lane = tid & 63;
  const int wv = __builtin_amdgcn_readfirstlane(tid >> 6);
  const unsigned rank = (unsigned)a.rank & 7u;
  const unsigned h = rank >> 2;
  const unsigned b = blockIdx.x;
  const unsigned xc = b & 7u, rg = (b >> 3) & 31u;
  char* const myws = a.ws[rank];
  const rsrc_t rsMy = mk(myws, WS_USED);
  const unsigned long long t0 = now();
  STAMP(wv);

  if (tid < 9 * 32) (&sm.mirror[0][0])[tid] = 0u;
  if (tid < 8) sm.st[tid] = 0u;
  if (tid == 0) sm.abort_ = 0u;
  if (tid == 9 * 64) {
    unsigned* ctr = (unsigned*)(myws + CTR_OFF) + b;
    unsigned c = __hip_atomic_load(ctr, __ATOMIC_RELAXED, __HIP_MEMORY_SCOPE_AGENT) + 1u;
    c &= 0xffffffu; if (c == 0u) c = 2u;
    __hip_atomic_store(ctr, c, __ATOMIC_RELAXED, __HIP_MEMORY_SCOPE_AGENT);
    sm.call = c;
  }
  __syncthreads();
  const unsigned call = lds_ld(&sm.call);
  const unsigned par = call & 1u;
  const unsigned dbase = par * PAR_STRIDE;
  const unsigned fbase = FLAG_OFF + par * FLAG_PAR;
  unsigned* const tmo = (unsigned*)(myws + TMO_OFF);

  if (wv < 8) {
    const unsigned v = (unsigned)wv;
    h8 xr[XR];
    u4 B0[8], B1[8];
    unsigned cx[4];
    {
      const unsigned xb = dbase + X_OFF + (rg * 8u + v) * 4u * PIECE;
#pragma unroll
      for (int q = 0; q < 4; ++q) cx[q] = wait_mirror(sm, &sm.mirror[0][v * 4 + q], call, t0) & 255u;
#pragma unroll
      for (int f = 0; f < 8; ++f) if (16 + f >= XR) B0[f] = __builtin_amdgcn_raw_buffer_load_b128(rsMy, lane * 16, xb + (16 + f) * 1024, AUX_LD);
#pragma unroll
      for (int f = 0; f < 8; ++f) B1[f] = __builtin_amdgcn_raw_buffer_load_b128(rsMy, lane * 16, xb + (24 + f) * 1024, AUX_LD);
#pragma unroll
      for (int f = 0; f < XR; ++f) xr[f] = __builtin_bit_cast(h8, __builtin_amdgcn_raw_buffer_load_b128(rsMy, lane * 16, xb + f * 1024, AUX_LD));
#pragma unroll
      for (int f = 0; f < 8; ++f) if (16 + f >= XR) sm.xl[v][16 + f - XR][lane] = B0[f];
#pragma unroll
      for (int f = 0; f < 8; ++f) sm.xl[v][24 + f - XR][lane] = B1[f];
    }
    STAMP(16 + wv);
    const int n = lane & 31, half = lane >> 5;
    const int voff = (n >> 3) * 1024 + half * 128 + (n & 7) * 16;
    const int col = 4 * (n & 7) + (n >> 3);
    const unsigned wb = dbase + W_OFF + (xc * 8u + v) * 4u * PIECE;
    f16v acc;
#pragma unroll
    for (int i = 0; i < 16; ++i) acc[i] = 0.f;
    int unit = 0;

#define LOADB(BUF, SO) do { const unsigned so_ = (SO); \
    _Pragma("unroll") for (int f = 0; f < 8; ++f) BUF[f] = __builtin_amdgcn_raw_buffer_load_b128(rsMy, voff, so_ + (f & 3) * 256 + (f >> 2) * 4096, AUX_LD); } while (0)
#define FLUSH(T, FINAL) do { wait_lds_zero(sm, &sm.st[v], t0); \
    float* tb_ = &sm.slot[v][4 * half * 32 + col]; \
    _Pragma("unroll") for (int i = 0; i < 16; ++i) { tb_[((i & 3) + 8 * (i >> 2)) * 32] = __builtin_ldexpf(acc[i], unit); acc[i] = 0.f; } \
    __builtin_amdgcn_fence(__ATOMIC_RELEASE, "workgroup"); \
    if (lane == 0) lds_st(&sm.st[v], 2u * (unsigned)(T) + 2u + (FINAL)); } while (0)
#define MMA(BUF, Q) do { _Pragma("unroll") for (int f = 0; f < 8; ++f) { \
      h8 av_; if ((Q) * 8 + f < XR) av_ = xr[(Q) * 8 + f < XR ? (Q) * 8 + f : 0]; else av_ = __builtin_bit_cast(h8, sm.xl[v][(Q) * 8 + f >= XR ? (Q) * 8 + f - XR : 0][lane]); \
      acc = __builtin_amdgcn_mfma_f32_32x32x16_f16(av_, __builtin_bit_cast(h8, BUF[f]), acc, 0, 0, 0); } } while (0)
#define STEP(CUR, NXT, Q) do { \
      const int u_ = (int)((cx[Q] + cw) * 8u) - 256 - 30; \
      if ((Q) == 0) unit = u_; \
      else if (u_ != unit) { FLUSH(t, 0u); unit = u_; } \
      const int tn_ = (Q) == 3 ? t + 1 : t; \
      const unsigned* mp_ = &sm.mirror[1 + (tn_ & 7)][v * 4 + (((Q) + 1) & 3)]; \
      const unsigned son_ = wb + (unsigned)tn_ * (256u * PIECE) + (((Q) + 1) & 3) * PIECE; \
      const bool more_ = tn_ < 8; \
      unsigned fw_ = lds_ld(mp_); \
      const bool rdy_ = more_ && (fw_ >> 8) == call; \
      asm volatile("" ::: "memory"); \
      LOADB(NXT, rdy_ ? son_ : WS_USED); \
      __builtin_amdgcn_sched_barrier(0); \
      MMA(CUR, Q); \
      if ((Q) == 3) FLUSH(t, 1u); \
      if (more_ && !rdy_) { fw_ = wait_mirror(sm, mp_, call, t0); LOADB(NXT, son_); } \
      cw = fw_ & 255u; \
    } while (0)

    unsigned cw = wait_mirror(sm, &sm.mirror[1][v * 4], call, t0) & 255u;
    LOADB(B0, wb);
    for (int t = 0; t < 8; ++t) {
      STEP(B0, B1, 0);
      STEP(B1, B0, 1);
      STEP(B0, B1, 2);
      STEP(B1, B0, 3);
      STAMP(32 + t * 8 + wv);
    }
  } else if (wv == 8) {
    __builtin_amdgcn_s_setprio(3);
    const unsigned d = ((rg < 16u ? 4u * h : 4u * (h ^ 1u)) + ((rg >> 2) & 3u)) & 7u;
    const unsigned s = (rg & 3u) * 8u + xc;
    const unsigned hd = d >> 2;
    const bool cross = hd != h, self = d == rank;
    const rsrc_t rsD = mk(a.ws[d], WS_USED);
    const rsrc_t rsX = mk(a.x, IN_BYTES);
    const rsrc_t rsW = mk(a.w, IN_BYTES);
    const unsigned nhop = (RELAY && cross) ? 16u : 0u;
    const unsigned per = !RELAY || self ? 32u : (cross ? 24u : 40u);
    const unsigned nitems = 128u + nhop + 8u * per;
    int nst = 0;
    for (unsigned i = s; i < nitems && s < NSEND; i += NSEND) {
      unsigned kind, idx, q, src = rank;
      if (i < 128u) { kind = 0; idx = ((i >> 2) + 16u) & 31u; q = i & 3u; }
      else if (i < 128u + nhop) { kind = 1; q = 0; idx = (i - 128u) * 4u + ((d - rank) & 3u); }
      else {
        const unsigned j = i - 128u - nhop, t = j / per, k = j - t * per;
        if (per == 32u) { kind = 1; q = k >> 3; idx = t * 8u + (k & 7u); }
        else if (per == 24u) { kind = 1; q = 1u + (k >> 3); idx = t * 8u + (k & 7u); }
        else if (k < 8u) { kind = 2; q = 0; src = 4u * (h ^ 1u) + (k >> 1); idx = t * 8u + ((rank - src) & 3u) + 4u * (k & 1u); }
        else { kind = 1; q = (k - 8u) >> 3; idx = t * 8u + (k & 7u); }
      }
      const unsigned line = kind == 0 ? idx : 32u + idx;
      const unsigned foff = fbase + line * 128u + (src * 4u + q) * 4u;
      const unsigned doff = dbase + (kind == 0 ? X_OFF : W_OFF) + ((idx * 8u + src) * 4u + q) * PIECE;
      u4 o[8];
      unsigned fval;
      if (kind == 2) {
        bool ok = true;
        while (true) {
          fval = __builtin_amdgcn_readfirstlane(__builtin_amdgcn_raw_buffer_load_b32(rsMy, 0, foff, AUX_SYS));
          if ((fval >> 8) == call) break;
          if (now() - t0 > DEADLINE || __hip_atomic_load(tmo, __ATOMIC_RELAXED, __HIP_MEMORY_SCOPE_AGENT) == call) {
            __hip_atomic_store(tmo, call, __ATOMIC_RELAXED, __HIP_MEMORY_SCOPE_AGENT); ok = false; break; }
          __builtin_amdgcn_s_sleep(2);
          asm volatile("" ::: "memory");
        }
        if (!ok) break;
        asm volatile("" ::: "memory");
#pragma unroll
        for (int f = 0; f < 8; ++f) o[f] = __builtin_amdgcn_raw_buffer_load_b128(rsMy, lane * 16, doff + f * 1024, AUX_SYS);
      } else {
        f4 v[16];
        if (kind == 0) {
          const unsigned row0 = idx < 16u ? 512u * d + 32u * idx
                                          : 512u * (4u * (hd ^ 1u) + ((idx >> 2) & 3u)) + 128u * (d & 3u) + 32u * (idx & 3u);
          const int vo = (lane & 31) * 2048 + (lane >> 5) * 32;
          const unsigned so = row0 * 2048u + q * 512u;
#pragma unroll
          for (int f = 0; f < 16; ++f) v[f] = __builtin_bit_cast(f4, __builtin_amdgcn_raw_buffer_load_b128(rsX, vo, so + (f >> 1) * 64 + (f & 1) * 16, 0));
        } else {
          const int vo = (lane >> 3) * 8 * 16384 + (lane & 7) * 16;
          const unsigned so = q * 128u * 16384u + (2048u * hd + 32u * idx) * 4u;
#pragma unroll
          for (int f = 0; f < 16; ++f) v[f] = __builtin_bit_cast(f4, __builtin_amdgcn_raw_buffer_load_b128(rsW, vo, so + (unsigned)((f >> 3) * 64 + (f & 7)) * 16384u, 0));
        }
        unsigned m = 0;
#pragma unroll
        for (int f = 0; f < 16; ++f)
#pragma unroll
          for (int c = 0; c < 4; ++c) m = max(m, absbits(v[f][c]));
        const unsigned code = wave_code(m);
        const int sh = 15 + 128 - (int)(code * 8u);
        fval = (call << 8) | code;
        if (kind == 0) {
#pragma unroll
          for (int f = 0; f < 8; ++f) {
            h8 r;
#pragma unroll
            for (int j = 0; j < 8; ++j) r[j] = (_Float16)__builtin_ldexpf(v[2 * f + (j >> 2)][j & 3], sh);
            o[f] = __builtin_bit_cast(u4, r);
          }
        } else {
#pragma unroll
          for (int f = 0; f < 8; ++f) {
            h8 r;
#pragma unroll
            for (int j = 0; j < 8; ++j) r[j] = (_Float16)__builtin_ldexpf(v[(f >> 2) * 8 + j][f & 3], sh);
            o[f] = __builtin_bit_cast(u4, r);
          }
        }
      }
#pragma unroll
      for (int f = 0; f < 8; ++f) __builtin_amdgcn_raw_buffer_store_b128(o[f], rsD, lane * 16, doff + f * 1024, AUX_SYS);
      asm volatile("s_waitcnt vmcnt(0)" ::: "memory");
      if (lane == 0) __builtin_amdgcn_raw_buffer_store_b32(fval, rsD, 0, foff, AUX_SYS);
      asm volatile("" ::: "memory");
      STAMP(128 + nst); ++nst;
    }
  } else if (wv == 9) {
    __builtin_amdgcn_s_setprio(2);
    int u = 0;
    unsigned spins = 0;
    while (u < 9) {
      const int ul = min(u + (lane >> 5), 8);
      const unsigned line = ul == 0 ? rg : 32u + (unsigned)(ul - 1) * 8u + xc;
      const unsigned val = __builtin_amdgcn_raw_buffer_load_b32(rsMy, line * 128u + (lane & 31) * 4, fbase, AUX_SYS);
      lds_st(&sm.mirror[ul][lane & 31], val);
      const unsigned long long okm = __builtin_amdgcn_ballot_w64((val >> 8) == call);
      asm volatile("" ::: "memory");
      if ((unsigned)okm == 0xffffffffu) {
        STAMP(112 + u);
        u += (okm == ~0ull) ? 2 : 1;
        continue;
      }
      if ((++spins & 255u) == 0u) {
        if (now() - t0 > DEADLINE || __hip_atomic_load(tmo, __ATOMIC_RELAXED, __HIP_MEMORY_SCOPE_AGENT) == call) {
          __hip_atomic_store(tmo, call, __ATOMIC_RELAXED, __HIP_MEMORY_SCOPE_AGENT);
          lds_st(&sm.abort_, 1u);
          break;
        }
      }
      __builtin_amdgcn_s_sleep(1);
    }
  } else {
    __builtin_amdgcn_s_setprio(2);
    const bool own = rg < 16u;
    const unsigned o = own ? rank : ((4u * (h ^ 1u) + ((rg >> 2) & 3u)) & 7u);
    const unsigned row0 = own ? 32u * rg : 128u * (rank & 3u) + 32u * (rg & 3u);
    const rsrc_t rsO = mk(a.out[o], OUT_BYTES);
    const int vo = (lane >> 3) * 16384 + (lane & 7) * 16;
    for (int t = 0; t < 8; ++t) {
      f4 r[4];
#pragma unroll
      for (int i = 0; i < 4; ++i) r[i] = (f4){0.f, 0.f, 0.f, 0.f};
      int nfin = 0;
      while (nfin < 8) {
        const unsigned sv = __hip_atomic_load(&sm.st[lane & 7], __ATOMIC_RELAXED, __HIP_MEMORY_SCOPE_WORKGROUP);
        const unsigned m = (unsigned)__builtin_amdgcn_ballot_w64((sv >> 1) == (unsigned)t + 1u) & 255u;
        if (m == 0u) {
          if (now() - t0 > 3 * DEADLINE) { lds_st(&sm.abort_, 1u); break; }
          __builtin_amdgcn_s_sleep(1);
          continue;
        }
        const unsigned mf = (unsigned)__builtin_amdgcn_ballot_w64((sv & 1u) != 0u) & m;
        __builtin_amdgcn_fence(__ATOMIC_ACQUIRE, "workgroup");
        unsigned mm = m;
        while (mm != 0u) {
          f4 p[4][4];
          const int cntm = __builtin_popcount(mm);
#pragma unroll
          for (int j = 0; j < 4; ++j) {
            const int v = __builtin_ctz(mm | 128u);
            mm &= mm - 1u;
#pragma unroll
            for (int i = 0; i < 4; ++i) p[j][i] = ((const f4*)&sm.slot[v][0])[i * 64 + lane];
          }
#pragma unroll
          for (int j = 0; j < 4; ++j)
            if (j < cntm) {
#pragma unroll
              for (int i = 0; i < 4; ++i) r[i] += p[j][i];
              asm volatile("" ::: "memory");
            }
        }
        __builtin_amdgcn_fence(__ATOMIC_RELEASE, "workgroup");
        if (lane < 8 && ((m >> lane) & 1u)) lds_st(&sm.st[lane], 0u);
        nfin += __builtin_popcount(mf);
      }
      if (lds_ld(&sm.abort_) != 0u) {
        const float nan = __builtin_nanf("");
#pragma unroll
        for (int i = 0; i < 4; ++i) r[i] = (f4){nan, nan, nan, nan};
      }
      const unsigned so = row0 * 16384u + (2048u * h + 32u * ((unsigned)t * 8u + xc)) * 4u;
#pragma unroll
      for (int i = 0; i < 4; ++i) __builtin_amdgcn_raw_buffer_store_b128(__builtin_bit_cast(u4, r[i]), rsO, vo, so + (unsigned)i * 8u * 16384u, AUX_SYS);
      STAMP(96 + t);
    }
    asm volatile("s_waitcnt vmcnt(0)" ::: "memory");
    STAMP(104);
    if (!own) {
      const rsrc_t rsD = mk(a.ws[o], WS_USED);
      if (lane == 0) __builtin_amdgcn_raw_buffer_store_b32(call, rsD, 0, YFLAG_OFF + ((rank & 3u) * 32u + (rg & 3u) * 8u + xc) * 4u, AUX_SYS);
    } else if (b == 0) {
      unsigned spins = 0;
      while (true) {
        const unsigned v0 = __builtin_amdgcn_raw_buffer_load_b32(rsMy, lane * 4, YFLAG_OFF, AUX_SYS);
        const unsigned v1 = __builtin_amdgcn_raw_buffer_load_b32(rsMy, lane * 4 + 256, YFLAG_OFF, AUX_SYS);
        if (__builtin_amdgcn_ballot_w64(v0 == call && v1 == call) == ~0ull) break;
        if ((++spins & 255u) == 0u) {
          if (now() - t0 > DEADLINE || __hip_atomic_load(tmo, __ATOMIC_RELAXED, __HIP_MEMORY_SCOPE_AGENT) == call) {
            __hip_atomic_store(tmo, call, __ATOMIC_RELAXED, __HIP_MEMORY_SCOPE_AGENT);
            if (lane == 0) __builtin_amdgcn_raw_buffer_store_b32(0x7fc00000u, rsO, 0, 2048u * h * 4u, AUX_SYS);
            break;
          }
        }
        __builtin_amdgcn_s_sleep(1);
        asm volatile("" ::: "memory");
      }
      STAMP(105);
    }
  }
}

extern "C" void kernel_launch(int rank, int nranks, ncclComm_t comm,
                              hipStream_t stream,
                              void* const* d_in, const int* in_sizes, int n_in,
                              void* d_out, int out_size,
                              const int* in_dtypes, int out_dtype,
                              void* const* d_out_all,
                              void* const* d_ws_all, size_t ws_bytes) {
  (void)comm; (void)in_dtypes; (void)out_dtype; (void)d_out;
  if (nranks != NR || rank < 0 || rank >= NR || n_in < 2 || ws_bytes < WS_USED ||
      in_sizes[0] != 4096 * 512 || in_sizes[1] != 512 * 4096 || out_size != 512 * 4096) {
    fprintf(stderr, "kernel_launch: unsupported configuration\n");
    return;
  }
  Args a;
  a.x = (const float*)d_in[0];
  a.w = (const float*)d_in[1];
  for (int p = 0; p < NR; ++p) { a.ws[p] = (char*)d_ws_all[p]; a.out[p] = (float*)d_out_all[p]; }
  a.rank = rank;
  hipLaunchKernelGGL(fused_gemm_kernel, dim3(256), dim3(NTHREADS), 0, stream, a);
  HIP_CHECK(hipGetLastError());
}
```

```cpp
#include <hip/hip_runtime.h>
#include <rccl/rccl.h>
#include <cstdio>

#define HIP_CHECK(expr) do { hipError_t _e = (expr); if (_e != hipSuccess) { \
    fprintf(stderr, "HIP error %s at %s:%d\n", hipGetErrorString(_e), __FILE__, __LINE__); \
    return; } } while (0)

#define RELAY 1
#define EXP 0
#define PSLEEP 1
#define NCLS 6
#define NW 5
#define SPL 1
#define DYN 1
#define NTOKC NTOK
#define NYI 108u
#define AUX_ST 17
#define NTOK 2

typedef float f4 __attribute__((ext_vector_type(4)));
typedef float f16v __attribute__((ext_vector_type(16)));
typedef unsigned u4 __attribute__((ext_vector_type(4)));
typedef _Float16 h8 __attribute__((ext_vector_type(8)));
typedef __amdgpu_buffer_rsrc_t rsrc_t;

constexpr int NR = 8;
constexpr unsigned PIECE = 8192;
constexpr unsigned SRC_STRIDE = 384u * PIECE;
constexpr unsigned W_OFF = 128u * PIECE;
constexpr unsigned PAR_STRIDE = 24u << 20;
constexpr unsigned FLAG_OFF = 48u << 20;
constexpr unsigned FLAG_PAR = 16384;
constexpr unsigned YFLAG_OFF = FLAG_OFF + 32768;
constexpr unsigned CTR_OFF = FLAG_OFF + 36864;
constexpr unsigned TMO_OFF = FLAG_OFF + 40960;
constexpr unsigned QUEUE_OFF = FLAG_OFF + 45056;
constexpr unsigned YRDY_OFF = FLAG_OFF + 49152;
constexpr unsigned DBG_OFF = FLAG_OFF + 65536;
constexpr unsigned YST_OFF = FLAG_OFF + (1u << 20);
constexpr unsigned WS_USED = YST_OFF + (4u << 20);
constexpr unsigned OUT_BYTES = 512u * 4096u * 4u;
constexpr unsigned IN_BYTES = 512u * 4096u * 4u;
constexpr unsigned long long DEADLINE = 50000000ull;
constexpr int AUX_SYS = 17;
#define AUX_LD 16
constexpr int XR = 17;
constexpr int NTHREADS = 640;
constexpr int NT = (64 + NCLS - 1) / NCLS;
constexpr unsigned CPX = 4u * NCLS;

struct Args {
  const float* x;
  const float* w;
  char* ws[NR];
  float* out[NR];
  int rank;
};

struct __attribute__((aligned(16))) Smem {
  u4 xl[8][32 - XR][64];
  float slot[8][32 * 32];
  unsigned mirror[NT + 2][32];
  unsigned st[8];
  unsigned abort_;
  unsigned next, done, tick;
};

__device__ __forceinline__ rsrc_t mk(const void* p, unsigned bytes) {
  return __builtin_amdgcn_make_buffer_rsrc((void*)p, (short)0, (int)bytes, 0x00020000);
}
__device__ __forceinline__ unsigned lds_ld(const unsigned* p) {
  return __builtin_amdgcn_readfirstlane(__hip_atomic_load(p, __ATOMIC_RELAXED, __HIP_MEMORY_SCOPE_WORKGROUP));
}
__device__ __forceinline__ void lds_st(unsigned* p, unsigned v) {
  __hip_atomic_store(p, v, __ATOMIC_RELAXED, __HIP_MEMORY_SCOPE_WORKGROUP);
}
__device__ __forceinline__ unsigned long long now() { return __builtin_amdgcn_s_memrealtime(); }

#define STAMP(slot) do {} while (0)

__device__ __forceinline__ unsigned wait_mirror(Smem& sm, const unsigned* p, unsigned call, unsigned long long t0) {
  unsigned v = lds_ld(p);
  while ((v >> 8) != call) {
    if (lds_ld(&sm.abort_) != 0u) break;
    if (now() - t0 > 2 * DEADLINE) { lds_st(&sm.abort_, 1u); break; }
    __builtin_amdgcn_s_sleep(1);
    v = lds_ld(p);
  }
  asm volatile("" ::: "memory");
  return v;
}
__device__ __forceinline__ void wait_lds_zero(Smem& sm, const unsigned* p, unsigned long long t0) {
  while (lds_ld(p) != 0u) {
    if (now() - t0 > 3 * DEADLINE) { lds_st(&sm.abort_, 1u); break; }
    __builtin_amdgcn_s_sleep(1);
  }
  asm volatile("" ::: "memory");
}

__device__ __forceinline__ unsigned absbits(float v) { return __builtin_bit_cast(unsigned, v) & 0x7fffffffu; }

__device__ __forceinline__ unsigned wave_code(unsigned m) {
  unsigned c = max(((m >> 23) + 9u) >> 3, 3u);
  unsigned r = 0;
#pragma unroll
  for (unsigned b = 32; b; b >>= 1)
    if (__builtin_amdgcn_ballot_w64(c >= (r | b)) != 0ull) r |= b;
  return r;
}

#define WG_BARRIER() asm volatile("s_waitcnt lgkmcnt(0)\n\ts_barrier" ::: "memory")

__global__ void __launch_bounds__(NTHREADS) fused_gemm_kernel(Args a) {
  __shared__ Smem sm;
  const int tid = threadIdx.x;
  const int lane = tid & 63;
  const int wv = __builtin_amdgcn_readfirstlane(tid >> 6);
  const unsigned rank = (unsigned)a.rank & 7u;
  const unsigned h = rank >> 2;
  const unsigned b = blockIdx.x;
  const unsigned xc = b & 7u, jb = (b >> 3) & 31u;
  char* const myws = a.ws[rank];
  const rsrc_t rsMy = mk(myws, WS_USED);
  const unsigned long long t0 = now();
  unsigned* const tmo = (unsigned*)(myws + TMO_OFF);
  const unsigned ctr_raw = __builtin_amdgcn_raw_buffer_load_b32(rsMy, 0, CTR_OFF + b * 4u, AUX_SYS);
  STAMP(wv);
#define GET_CALL() \
  unsigned call = (__builtin_amdgcn_readfirstlane(ctr_raw) + 1u) & 0xffffffu; \
  if (call == 0u) call = 2u;     \
  asm volatile("" : "+s"(call) :: "memory"); \
  const unsigned dbase = (call & 1u) * PAR_STRIDE; \
  const unsigned fbase = FLAG_OFF + (call & 1u) * FLAG_PAR; \
  WG_BARRIER(); \
  if (tid == NTHREADS - 64) __builtin_amdgcn_raw_buffer_store_b32(call, rsMy, 0, CTR_OFF + b * 4u, AUX_SYS)

  if (jb < 8u * SPL) {
    if (tid == 0) { sm.next = NW; sm.done = 0u; sm.tick = 0u; sm.abort_ = 0u; }
    if (wv >= NW) {
      GET_CALL();
      (void)dbase; (void)fbase;
      if (b == 56u && wv == 9) {
        if (lane < 8) __builtin_amdgcn_raw_buffer_store_b32(0u, rsMy, lane * 128, QUEUE_OFF + ((call & 1u) ^ 1u) * 1024u, AUX_SYS);
        unsigned spins = 0;
        const unsigned w0 = (unsigned)(lane & 31) * 4u;
        bool live[4];
#pragma unroll
        for (int c = 0; c < 4; ++c) live[c] = w0 < 32u ? (!(EXP & 1) && ((w0 + c) & 7u) < (unsigned)NCLS) : w0 < 32u + 8u * NW;
        while (true) {
          const u4 v0 = __builtin_amdgcn_raw_buffer_load_b128(rsMy, lane * 16, YFLAG_OFF, AUX_SYS);
          const u4 v1 = __builtin_amdgcn_raw_buffer_load_b128(rsMy, lane * 16 + 1024, YFLAG_OFF, AUX_SYS);
          bool okk = true;
#pragma unroll
          for (int c = 0; c < 4; ++c) okk = okk && (!live[c] || (EXP ? (v0[c] >= call && v1[c] >= call) : (v0[c] == call && v1[c] == call)));
          if (__builtin_amdgcn_ballot_w64(okk) == ~0ull) break;
          if ((++spins & 255u) == 0u) {
            if (now() - t0 > DEADLINE || __hip_atomic_load(tmo, __ATOMIC_RELAXED, __HIP_MEMORY_SCOPE_AGENT) == call) {
              __hip_atomic_store(tmo, call, __ATOMIC_RELAXED, __HIP_MEMORY_SCOPE_AGENT);
              if (lane == 0) __builtin_amdgcn_raw_buffer_store_b32(0x7fc00000u, mk(a.out[rank], OUT_BYTES), 0, 2048u * (h ^ 1u) * 4u, AUX_SYS);
              break;
            }
          }
          __builtin_amdgcn_s_sleep(1);
          asm volatile("" ::: "memory");
        }
        STAMP(15);
      }
      return;
    }
    const unsigned d = (rank + 1u + jb) & 7u;
    const unsigned hd = d >> 2;
    const bool cross = hd != h, self = d == rank;
    const rsrc_t rsD = mk(a.ws[d], WS_USED);
    const rsrc_t rsX = mk(a.x, IN_BYTES);
    const rsrc_t rsW = mk(a.w, IN_BYTES);
    const unsigned nhop = (RELAY && cross) ? 16u : 0u;
    const unsigned per = !RELAY || self ? 4u : (cross ? 3u : 5u);
    const unsigned nin = 128u + nhop + 64u * per;
    const unsigned nitems = nin + ((cross && !(EXP & 1)) ? NYI : 0u);
    const rsrc_t rsO = mk(a.out[d], OUT_BYTES);
    constexpr unsigned NS = 8u * SPL;
    const unsigned sidx = (jb >> 3) * 8u + xc;
    unsigned i = (unsigned)wv * NS + sidx;
    unsigned call = 0;
#define S_DBASE ((call & 1u) * PAR_STRIDE)
#define S_FBASE (FLAG_OFF + (call & 1u) * FLAG_PAR)
    unsigned hopf = 0;
    bool bad = false;
    int nst = 0; (void)nst;
    while (true) {
      unsigned kind, idx, q, src = rank;
      if (i < nhop) { kind = 1; q = 0; idx = i * 4u + ((d - rank) & 3u); }
      else if (i < 128u + nhop) { const unsigned j = i - nhop; kind = 0; idx = ((j >> 2) + 16u) & 31u; q = j & 3u; }
      else if (i >= nin) { kind = 3; idx = (i - nin) >> 2; q = (i - nin) & 3u; }
      else {
        const unsigned j = i - 128u - nhop;
        const unsigned r = j / (NCLS * per), rem = j - r * (NCLS * per);
        const unsigned ng = min((unsigned)NCLS, 64u - r * NCLS);
        const unsigned k = rem / ng;
        idx = r * NCLS + rem - k * ng;
        kind = 1; q = k + 4u - per;
        if (per == 5u && k == 0u) { kind = 2; q = 0; src = 4u * (h ^ 1u) + ((rank - idx) & 3u); }
      }
      const unsigned foff = (kind == 0 ? idx : 32u + idx) * 128u + (src * 4u + q) * 4u;
      const unsigned doff = src * SRC_STRIDE + (kind == 0 ? 0u : W_OFF) + (idx * 4u + q) * PIECE;
      u4 o[8];
      unsigned fval;
      if (kind == 3) {
        const unsigned yi = (d & 3u) * 128u + idx * 4u + q;
        while (true) {
          const unsigned yv = __builtin_amdgcn_raw_buffer_load_b32(rsMy, (lane & 1) * 4, YRDY_OFF + yi * 8u, AUX_SYS);
          if (__builtin_amdgcn_ballot_w64(yv == call) == ~0ull) break;
          if (now() - t0 > DEADLINE || __hip_atomic_load(tmo, __ATOMIC_RELAXED, __HIP_MEMORY_SCOPE_AGENT) == call) {
            __hip_atomic_store(tmo, call, __ATOMIC_RELAXED, __HIP_MEMORY_SCOPE_AGENT); bad = true; break; }
          __builtin_amdgcn_s_sleep(4);
          asm volatile("" ::: "memory");
        }
        asm volatile("" ::: "memory");
        fval = 0;
#pragma unroll
        for (int f = 0; f < 8; ++f) o[f] = __builtin_amdgcn_raw_buffer_load_b128(rsMy, lane * 16, YST_OFF + yi * PIECE + f * 1024, AUX_SYS);
      } else if (kind == 2) {
        while (true) {
          fval = __builtin_amdgcn_readlane(hopf, idx);
          if ((fval >> 8) == call) break;
          if (now() - t0 > DEADLINE || __hip_atomic_load(tmo, __ATOMIC_RELAXED, __HIP_MEMORY_SCOPE_AGENT) == call) {
            __hip_atomic_store(tmo, call, __ATOMIC_RELAXED, __HIP_MEMORY_SCOPE_AGENT); break; }
          hopf = __builtin_amdgcn_raw_buffer_load_b32(rsMy, (32u + lane) * 128u + ((rank - lane) & 3u) * 16u, S_FBASE + (h ^ 1u) * 64u, AUX_SYS);
          asm volatile("" ::: "memory");
        }
        asm volatile("" ::: "memory");
#pragma unroll
        for (int f = 0; f < 8; ++f) o[f] = __builtin_amdgcn_raw_buffer_load_b128(rsMy, lane * 16, S_DBASE + doff + f * 1024, AUX_SYS);
      } else {
        f4 v[16];
        {
          const unsigned row0 = idx < 16u ? 512u * d + 32u * idx
                                          : 512u * (4u * (hd ^ 1u) + ((idx >> 2) & 3u)) + 128u * (d & 3u) + 32u * (idx & 3u);
          const bool isx = kind == 0;
          const rsrc_t rs = isx ? rsX : rsW;
          const int vo = isx ? (lane & 31) * 2048 + (lane >> 5) * 32 : (lane >> 3) * 8 * 16384 + (lane & 7) * 16;
          const unsigned so = isx ? row0 * 2048u + q * 512u : q * 128u * 16384u + (2048u * hd + 32u * idx) * 4u;
          const unsigned sa = isx ? 256u : 64u * 16384u, sb = isx ? 64u : 2u * 16384u, sc = isx ? 16u : 16384u;
#pragma unroll
          for (int f = 0; f < 16; ++f)
            v[f] = __builtin_bit_cast(f4, __builtin_amdgcn_raw_buffer_load_b128(rs, vo, so + (f >> 3) * sa + ((f >> 1) & 3) * sb + (f & 1) * sc, 0));
        }
        if (call == 0u) {
          call = (__builtin_amdgcn_readfirstlane(ctr_raw) + 1u) & 0xffffffu;
          if (call == 0u) call = 2u;
          asm volatile("" : "+s"(call) :: "memory");
          WG_BARRIER();
        }
        float m = 0.f;
#pragma unroll
        for (int f = 0; f < 16; ++f) {
          asm("v_max3_f32 %0, %1, |%2|, |%3|" : "=v"(m) : "v"(m), "v"(v[f][0]), "v"(v[f][1]));
          asm("v_max3_f32 %0, %1, |%2|, |%3|" : "=v"(m) : "v"(m), "v"(v[f][2]), "v"(v[f][3]));
        }
        const unsigned code = wave_code(__builtin_bit_cast(unsigned, m));
        const float scl = __builtin_bit_cast(float, (127u + 15u + 128u - code * 8u) << 23);
        fval = (call << 8) | code;
#pragma unroll
        for (int f = 0; f < 16; ++f) v[f] *= scl;
        if (kind == 0) {
#pragma unroll
          for (int f = 0; f < 8; ++f) {
            h8 r;
#pragma unroll
            for (int j = 0; j < 8; ++j) r[j] = (_Float16)v[2 * f + (j >> 2)][j & 3];
            o[f] = __builtin_bit_cast(u4, r);
          }
        } else {
#pragma unroll
          for (int f = 0; f < 8; ++f) {
            h8 r;
#pragma unroll
            for (int j = 0; j < 8; ++j) r[j] = (_Float16)v[(f >> 2) * 8 + j][f & 3];
            o[f] = __builtin_bit_cast(u4, r);
          }
        }
      }
#pragma unroll
      for (int f = 0; f < 8; ++f) asm volatile("" : "+v"(o[f]));
      unsigned tk = 0;
      if (lane == 0) tk = atomicAdd(&sm.tick, 1u);
      tk = __builtin_amdgcn_readfirstlane(tk);
      if (tk < 60) STAMP(16 + 4 * tk);
      if (!self) {
        const unsigned lim = (cross && kind != 0) ? NTOKC : NTOK;
        while (tk >= lds_ld(&sm.done) + lim) __builtin_amdgcn_s_sleep(1);
        asm volatile("" ::: "memory");
      }
      if (tk < 60) STAMP(16 + 4 * tk + 1);
      {
        const bool isy = kind == 3;
        const rsrc_t rs = isy ? rsO : rsD;
        const int vo = isy ? (lane >> 4) * 16384 + (lane & 15) * 16 : lane * 16;
        const unsigned so = isy ? (128u * (rank & 3u) + 32u * q) * 16384u + (2048u * h + 64u * idx) * 4u : S_DBASE + doff;
        const unsigned st = isy ? 4u * 16384u : 1024u;
#pragma unroll
        for (int f = 0; f < 8; ++f) __builtin_amdgcn_raw_buffer_store_b128(o[f], rs, vo, so + f * st, AUX_ST);
      }
      unsigned nn = 0;
      if (lane == 0) nn = __hip_atomic_fetch_add((unsigned*)(myws + QUEUE_OFF + (call & 1u) * 1024u + (jb & 7u) * 128u), 1u, __ATOMIC_RELAXED, __HIP_MEMORY_SCOPE_SYSTEM);
      asm volatile("s_waitcnt vmcnt(0)" ::: "memory");
      nn = __builtin_amdgcn_readfirstlane(nn) + NW * NS;
      if (lane == 0) {
        atomicAdd(&sm.done, 1u);
        if (kind != 3) __builtin_amdgcn_raw_buffer_store_b32(fval, rsD, 0, S_FBASE + foff, AUX_SYS);
      }
      asm volatile("" ::: "memory");
      if (tk < 60) STAMP(16 + 4 * tk + 2);
      i = nn;
      if (i >= nitems) break;
    }
    if (cross && lane == 0 && !bad)
      __builtin_amdgcn_raw_buffer_store_b32(call, rsD, 0, YFLAG_OFF + ((rank & 3u) * 128u + 32u + (unsigned)wv * 8u + xc) * 4u, AUX_SYS);
    return;
  }

  if (jb - 8u * SPL >= CPX) return;
  const unsigned cidx = xc * CPX + (jb - 8u * SPL);
  const unsigned cls = cidx >> 5;
  const unsigned rg = ((cidx & 1u) << 4) | ((cidx & 31u) >> 1);
  const int nt = (int)((64u - cls + NCLS - 1u) / NCLS);
  if (tid < (NT + 2) * 32) (&sm.mirror[0][0])[tid] = 0u;
  if (tid < 8) sm.st[tid] = 0u;
  if (tid == 0) sm.abort_ = 0u;
  GET_CALL();
  if (EXP & 1) { if (wv != 8 || (EXP & 2)) return; }

  if (wv < 8) {
    const unsigned v = (unsigned)wv;
    h8 xr[XR];
    u4 B0[8], B1[8];
    unsigned cx[4];
    {
      const unsigned xb = dbase + v * SRC_STRIDE + rg * 4u * PIECE;
#pragma unroll
      for (int q = 0; q < 4; ++q) cx[q] = wait_mirror(sm, &sm.mirror[0][v * 4 + q], call, t0) & 255u;
#pragma unroll
      for (int f = 0; f < 8; ++f) if (16 + f >= XR) B0[f] = __builtin_amdgcn_raw_buffer_load_b128(rsMy, lane * 16, xb + (16 + f) * 1024, AUX_LD);
#pragma unroll
      for (int f = 0; f < 8; ++f) B1[f] = __builtin_amdgcn_raw_buffer_load_b128(rsMy, lane * 16, xb + (24 + f) * 1024, AUX_LD);
#pragma unroll
      for (int f = 0; f < XR; ++f) xr[f] = __builtin_bit_cast(h8, __builtin_amdgcn_raw_buffer_load_b128(rsMy, lane * 16, xb + f * 1024, AUX_LD));
#pragma unroll
      for (int f = 0; f < 8; ++f) if (16 + f >= XR) sm.xl[v][16 + f - XR][lane] = B0[f];
#pragma unroll
      for (int f = 0; f < 8; ++f) sm.xl[v][24 + f - XR][lane] = B1[f];
    }
    STAMP(16 + wv);
    const int n = lane & 31, half = lane >> 5;
    const int voff = (n >> 3) * 1024 + half * 128 + (n & 7) * 16;
    const int col = 4 * (n & 7) + (n >> 3);
    const unsigned wb = dbase + v * SRC_STRIDE + W_OFF + cls * 4u * PIECE;
    f16v acc;
#pragma unroll
    for (int i = 0; i < 16; ++i) acc[i] = 0.f;
    int unit = 0;

#define LOADB(BUF, SO) do { const unsigned so_ = (SO); \
    _Pragma("unroll") for (int f = 0; f < 8; ++f) BUF[f] = __builtin_amdgcn_raw_buffer_load_b128(rsMy, voff, so_ + (f & 3) * 256 + (f >> 2) * 4096, AUX_LD); } while (0)
#define FLUSH(T, FINAL) do { wait_lds_zero(sm, &sm.st[v], t0); \
    float* tb_ = &sm.slot[v][4 * half * 32 + col]; \
    _Pragma("unroll") for (int i = 0; i < 16; ++i) { tb_[((i & 3) + 8 * (i >> 2)) * 32] = __builtin_ldexpf(acc[i], unit); acc[i] = 0.f; } \
    __builtin_amdgcn_fence(__ATOMIC_RELEASE, "workgroup"); \
    if (lane == 0) lds_st(&sm.st[v], 2u * (unsigned)(T) + 2u + (FINAL)); } while (0)
#define MMA(BUF, Q) do { _Pragma("unroll") for (int f = 0; f < 8; ++f) { \
      h8 av_; if ((Q) * 8 + f < XR) av_ = xr[(Q) * 8 + f < XR ? (Q) * 8 + f : 0]; else av_ = __builtin_bit_cast(h8, sm.xl[v][(Q) * 8 + f >= XR ? (Q) * 8 + f - XR : 0][lane]); \
      acc = __builtin_amdgcn_mfma_f32_32x32x16_f16(av_, __builtin_bit_cast(h8, BUF[f]), acc, 0, 0, 0); } } while (0)
#define STEP(CUR, NXT, Q) do { \
      const int u_ = (int)((cx[Q] + cw) * 8u) - 256 - 30; \
      if ((Q) == 0) unit = u_; \
      else if (u_ != unit) { FLUSH(t, 0u); unit = u_; } \
      const int tn_ = (Q) == 3 ? t + 1 : t; \
      const unsigned* mp_ = &sm.mirror[1 + tn_][v * 4 + (((Q) + 1) & 3)]; \
      const unsigned son_ = wb + (unsigned)tn_ * (NCLS * 4u * PIECE) + (((Q) + 1) & 3) * PIECE; \
      const bool more_ = tn_ < nt; \
      unsigned fw_ = lds_ld(mp_); \
      const bool rdy_ = more_ && (fw_ >> 8) == call; \
      asm volatile("" ::: "memory"); \
      LOADB(NXT, rdy_ ? son_ : WS_USED); \
      __builtin_amdgcn_sched_barrier(0); \
      MMA(CUR, Q); \
      if ((Q) == 3) FLUSH(t, 1u); \
      if (more_ && !rdy_) { fw_ = wait_mirror(sm, mp_, call, t0); LOADB(NXT, son_); } \
      cw = fw_ & 255u; \
    } while (0)

    unsigned cw = wait_mirror(sm, &sm.mirror[1][v * 4], call, t0) & 255u;
    LOADB(B0, wb);
    for (int t = 0; t < nt; ++t) {
      STEP(B0, B1, 0);
      STEP(B1, B0, 1);
      STEP(B0, B1, 2);
      STEP(B1, B0, 3);
      STAMP(128 + t * 8 + wv);
    }
  } else if (wv == 8) {
    __builtin_amdgcn_s_setprio(2);
    int u = 0;
    unsigned spins = 0;
    while (u <= nt) {
      const int ul = min(u + (lane >> 5), nt);
      const unsigned line = ul == 0 ? rg : 32u + (unsigned)(ul - 1) * NCLS + cls;
      const unsigned val = __builtin_amdgcn_raw_buffer_load_b32(rsMy, line * 128u + (lane & 31) * 4, fbase, AUX_SYS);
      lds_st(&sm.mirror[ul][lane & 31], val);
      const unsigned long long okm = __builtin_amdgcn_ballot_w64((val >> 8) == call);
      asm volatile("" ::: "memory");
      if ((unsigned)okm == 0xffffffffu) {
        STAMP(64 + u);
        if (okm == ~0ull) { STAMP(64 + u + 1); }
        u += (okm == ~0ull) ? 2 : 1;
        continue;
      }
      if ((++spins & 255u) == 0u) {
        if (now() - t0 > DEADLINE || __hip_atomic_load(tmo, __ATOMIC_RELAXED, __HIP_MEMORY_SCOPE_AGENT) == call) {
          __hip_atomic_store(tmo, call, __ATOMIC_RELAXED, __HIP_MEMORY_SCOPE_AGENT);
          lds_st(&sm.abort_, 1u);
          break;
        }
      }
      __builtin_amdgcn_s_sleep(PSLEEP);
    }
  } else {
    __builtin_amdgcn_s_setprio(2);
    const bool own = rg < 16u;
    const unsigned o = own ? rank : ((4u * (h ^ 1u) + ((rg >> 2) & 3u)) & 7u);
    const unsigned row0 = own ? 32u * rg : 128u * (rank & 3u) + 32u * (rg & 3u);
    const rsrc_t rsO = mk(a.out[o], OUT_BYTES);
    const int vo = (lane >> 3) * 16384 + (lane & 7) * 16;
    for (int t = 0; t < nt; ++t) {
      f4 r[4];
#pragma unroll
      for (int i = 0; i < 4; ++i) r[i] = (f4){0.f, 0.f, 0.f, 0.f};
      int nfin = 0;
      while (nfin < 8) {
        const unsigned sv = __hip_atomic_load(&sm.st[lane & 7], __ATOMIC_RELAXED, __HIP_MEMORY_SCOPE_WORKGROUP);
        const unsigned m = (unsigned)__builtin_amdgcn_ballot_w64((sv >> 1) == (unsigned)t + 1u) & 255u;
        if (m == 0u) {
          if (now() - t0 > 3 * DEADLINE) { lds_st(&sm.abort_, 1u); break; }
          __builtin_amdgcn_s_sleep(1);
          continue;
        }
        const unsigned mf = (unsigned)__builtin_amdgcn_ballot_w64((sv & 1u) != 0u) & m;
        __builtin_amdgcn_fence(__ATOMIC_ACQUIRE, "workgroup");
        unsigned mm = m;
        while (mm != 0u) {
          f4 p[4][4];
          const int cntm = __builtin_popcount(mm);
#pragma unroll
          for (int j = 0; j < 4; ++j) {
            const int v = __builtin_ctz(mm | 128u);
            mm &= mm - 1u;
#pragma unroll
            for (int i = 0; i < 4; ++i) p[j][i] = ((const f4*)&sm.slot[v][0])[i * 64 + lane];
          }
#pragma unroll
          for (int j = 0; j < 4; ++j)
            if (j < cntm) {
#pragma unroll
              for (int i = 0; i < 4; ++i) r[i] += p[j][i];
              asm volatile("" ::: "memory");
            }
        }
        __builtin_amdgcn_fence(__ATOMIC_RELEASE, "workgroup");
        if (lane < 8 && ((m >> lane) & 1u)) lds_st(&sm.st[lane], 0u);
        nfin += __builtin_popcount(mf);
      }
      if (lds_ld(&sm.abort_) != 0u) {
        const float nan = __builtin_nanf("");
#pragma unroll
        for (int i = 0; i < 4; ++i) r[i] = (f4){nan, nan, nan, nan};
      }
      const unsigned g = (unsigned)t * NCLS + cls;
      if (!own && g < NYI / 2u) {
        const unsigned yi = (o & 3u) * 128u + (g >> 1) * 4u + (rg & 3u);
#pragma unroll
        for (int i = 0; i < 4; ++i) __builtin_amdgcn_raw_buffer_store_b128(__builtin_bit_cast(u4, r[i]), rsMy, (lane >> 3) * 256 + (lane & 7) * 16, YST_OFF + yi * PIECE + (g & 1u) * 128u + (unsigned)i * 2048u, AUX_SYS);
        asm volatile("s_waitcnt vmcnt(0)" ::: "memory");
        if (lane == 0) __builtin_amdgcn_raw_buffer_store_b32(call, rsMy, 0, YRDY_OFF + yi * 8u + (g & 1u) * 4u, AUX_SYS);
      } else {
        const unsigned so = row0 * 16384u + (2048u * h + 32u * g) * 4u;
#pragma unroll
        for (int i = 0; i < 4; ++i) __builtin_amdgcn_raw_buffer_store_b128(__builtin_bit_cast(u4, r[i]), rsO, vo, so + (unsigned)i * 8u * 16384u, AUX_SYS);
      }
      STAMP(96 + t);
    }
    asm volatile("s_waitcnt vmcnt(0)" ::: "memory");
    STAMP(14);
    if (!own) {
      const rsrc_t rsD = mk(a.ws[o], WS_USED);
      if (lane == 0) __builtin_amdgcn_raw_buffer_store_b32(call, rsD, 0, YFLAG_OFF + ((rank & 3u) * 128u + (rg & 3u) * 8u + cls) * 4u, AUX_SYS);
    }
  }
}

extern "C" void kernel_launch(int rank, int nranks, ncclComm_t comm,
                              hipStream_t stream,
                              void* const* d_in, const int* in_sizes, int n_in,
                              void* d_out, int out_size,
                              const int* in_dtypes, int out_dtype,
                              void* const* d_out_all,
                              void* const* d_ws_all, size_t ws_bytes) {
  (void)comm; (void)in_dtypes; (void)out_dtype; (void)d_out;
  if (nranks != NR || rank < 0 || rank >= NR || n_in < 2 || ws_bytes < WS_USED ||
      in_sizes[0] != 4096 * 512 || in_sizes[1] != 512 * 4096 || out_size != 512 * 4096) {
    fprintf(stderr, "kernel_launch: unsupported configuration\n");
    return;
  }
  Args a;
  a.x = (const float*)d_in[0];
  a.w = (const float*)d_in[1];
  for (int p = 0; p < NR; ++p) { a.ws[p] = (char*)d_ws_all[p]; a.out[p] = (float*)d_out_all[p]; }
  a.rank = rank;
  hipLaunchKernelGGL(fused_gemm_kernel, dim3(256), dim3(NTHREADS), 0, stream, a);
  HIP_CHECK(hipGetLastError());
}
```

```cpp
#include <hip/hip_runtime.h>
#include <rccl/rccl.h>
#include <cstdio>

#define HIP_CHECK(expr) do { hipError_t _e = (expr); if (_e != hipSuccess) { \
    fprintf(stderr, "HIP error %s at %s:%d\n", hipGetErrorString(_e), __FILE__, __LINE__); \
    return; } } while (0)

#define RELAY 1
#define WARM 0
#define WIN 6
#define EXP 0
#define PSLEEP 1
#define NCLS 6
#define NW 4
#define SPL 1
#define DYN 1
#define NTOKC NTOK
#define NYI 108u
#define AUX_ST 17
#define NTOK 2

typedef float f4 __attribute__((ext_vector_type(4)));
typedef float f16v __attribute__((ext_vector_type(16)));
typedef unsigned u4 __attribute__((ext_vector_type(4)));
typedef _Float16 h8 __attribute__((ext_vector_type(8)));
typedef __amdgpu_buffer_rsrc_t rsrc_t;

constexpr int NR = 8;
constexpr unsigned PIECE = 8192;
constexpr unsigned SRC_STRIDE = 384u * PIECE;
constexpr unsigned W_OFF = 128u * PIECE;
constexpr unsigned PAR_STRIDE = 24u << 20;
constexpr unsigned FLAG_OFF = 48u << 20;
constexpr unsigned FLAG_PAR = 16384;
constexpr unsigned YFLAG_OFF = FLAG_OFF + 32768;
constexpr unsigned CTR_OFF = FLAG_OFF + 36864;
constexpr unsigned TMO_OFF = FLAG_OFF + 40960;
constexpr unsigned QUEUE_OFF = FLAG_OFF + 45056;
constexpr unsigned YRDY_OFF = FLAG_OFF + 49152;
constexpr unsigned CTR2_OFF = FLAG_OFF + 53248;
constexpr unsigned DBG_OFF = FLAG_OFF + 65536;
constexpr unsigned YST_OFF = FLAG_OFF + (1u << 20);
constexpr unsigned WS_USED = YST_OFF + (4u << 20);
constexpr unsigned OUT_BYTES = 512u * 4096u * 4u;
constexpr unsigned IN_BYTES = 512u * 4096u * 4u;
constexpr unsigned long long DEADLINE = 50000000ull;
constexpr int AUX_SYS = 17;
#define AUX_LD 16
constexpr int XR = 17;
constexpr int NTHREADS = 640;
constexpr int NT = (64 + NCLS - 1) / NCLS;
constexpr unsigned CPX = 4u * NCLS;

struct Args {
  const float* x;
  const float* w;
  char* myws;
  unsigned rank;
  unsigned pad_;
  char* wsd[NR];
  float* outd[NR];
};

struct __attribute__((aligned(16))) Smem {
  u4 xl[8][32 - XR][64];
  float slot[8][32 * 32];
  unsigned mirror[NT + 2][32];
  unsigned st[8];
  unsigned abort_;
  unsigned tick, pad_[2];
  u4 fifo[8];
};

__device__ __forceinline__ rsrc_t mk(const void* p, unsigned bytes) {
  return __builtin_amdgcn_make_buffer_rsrc((void*)p, (short)0, (int)bytes, 0x00020000);
}
__device__ __forceinline__ unsigned lds_ld(const unsigned* p) {
  return __builtin_amdgcn_readfirstlane(__hip_atomic_load(p, __ATOMIC_RELAXED, __HIP_MEMORY_SCOPE_WORKGROUP));
}
__device__ __forceinline__ void lds_st(unsigned* p, unsigned v) {
  __hip_atomic_store(p, v, __ATOMIC_RELAXED, __HIP_MEMORY_SCOPE_WORKGROUP);
}
__device__ __forceinline__ unsigned long long now() { return __builtin_amdgcn_s_memrealtime(); }

#define STAMP(slot) do {} while (0)

__device__ __forceinline__ unsigned wait_mirror(Smem& sm, const unsigned* p, unsigned call, unsigned long long t0) {
  unsigned v = lds_ld(p);
  while ((v >> 8) != call) {
    if (lds_ld(&sm.abort_) != 0u) break;
    if (now() - t0 > 2 * DEADLINE) { lds_st(&sm.abort_, 1u); break; }
    __builtin_amdgcn_s_sleep(1);
    v = lds_ld(p);
  }
  asm volatile("" ::: "memory");
  return v;
}
__device__ __forceinline__ void wait_lds_zero(Smem& sm, const unsigned* p, unsigned long long t0) {
  while (lds_ld(p) != 0u) {
    if (now() - t0 > 3 * DEADLINE) { lds_st(&sm.abort_, 1u); break; }
    __builtin_amdgcn_s_sleep(1);
  }
  asm volatile("" ::: "memory");
}

__device__ __forceinline__ unsigned absbits(float v) { return __builtin_bit_cast(unsigned, v) & 0x7fffffffu; }

__device__ __forceinline__ unsigned wave_code(unsigned m) {
  unsigned c = max(((m >> 23) + 9u) >> 3, 3u);
  unsigned r = 0;
#pragma unroll
  for (unsigned b = 32; b; b >>= 1)
    if (__builtin_amdgcn_ballot_w64(c >= (r | b)) != 0ull) r |= b;
  return r;
}

#define WG_BARRIER() asm volatile("s_waitcnt lgkmcnt(0)\n\ts_barrier" ::: "memory")

__device__ __forceinline__ unsigned next_call(unsigned raw) {
  unsigned call = (__builtin_amdgcn_readfirstlane(raw) + 1u) & 0xffffffu;
  if (call == 0u) call = 2u;
  asm volatile("" : "+s"(call) :: "memory");
  return call;
}

__global__ void __launch_bounds__(NTHREADS) fused_gemm_kernel(Args a) {
  __shared__ Smem sm;
#define STAMP_ENTRY() do {} while (0)
  const int tid = threadIdx.x;
  const int lane = tid & 63;
  const int wv = __builtin_amdgcn_readfirstlane(tid >> 6);
  const unsigned rank = a.rank & 7u;
  const unsigned h = rank >> 2;
  const unsigned b = blockIdx.x;
  const unsigned xc = b & 7u, jb = (b >> 3) & 31u;
  char* const myws = a.myws;
  const float* const px = a.x;
  const float* const pw = a.w;
  char* const pd = a.wsd[jb & 7u];
  float* const po = a.outd[jb & 7u];
  asm volatile("" :: "s"(px), "s"(pw), "s"(myws), "s"(rank), "s"(pd), "s"(po));
  const rsrc_t rsMy = mk(myws, WS_USED);
  unsigned* const tmo = (unsigned*)(myws + TMO_OFF);

  if (__builtin_expect(jb < 8u && wv < NW, 1)) {
    const unsigned d = (rank + 1u + jb) & 7u;
    const unsigned hd = d >> 2;
    const bool cross = hd != h, self = d == rank;
    const rsrc_t rsX = mk(px, IN_BYTES);
    const rsrc_t rsW = mk(pw, IN_BYTES);
    const unsigned nhop = (RELAY && cross) ? 16u : 0u;
    unsigned kind, idx, q, src = rank;
    f4 v[16];
#define DECODE_HX(I) do { \
      if ((I) < nhop) { kind = 1; q = 0; idx = (I) * 4u + ((jb + 1u) & 3u); } \
      else { const unsigned j_ = (I) - nhop; kind = 0; idx = ((j_ >> 2) + 16u) & 31u; q = j_ & 3u; } } while (0)
#define LOAD_XW() do { \
      const unsigned row0_ = idx < 16u ? 512u * d + 32u * idx \
                                       : 512u * (4u * (hd ^ 1u) + ((idx >> 2) & 3u)) + 128u * (d & 3u) + 32u * (idx & 3u); \
      const bool isx_ = kind == 0; \
      const rsrc_t rs_ = isx_ ? rsX : rsW; \
      const int vo_ = isx_ ? (lane >> 4) * 2048 + (lane & 15) * 16 : (lane >> 3) * 4 * 16384 + (lane & 7) * 16; \
      const unsigned so_ = isx_ ? row0_ * 2048u + q * 512u : q * 128u * 16384u + (2048u * hd + 32u * idx) * 4u; \
      const unsigned s0_ = isx_ ? 256u : 16384u, s1_ = isx_ ? 8192u : 32768u, s2_ = isx_ ? 16384u : 64u * 16384u, s3_ = isx_ ? 32768u : 32u * 16384u; \
      _Pragma("unroll") for (int f = 0; f < 16; ++f) \
        v[f] = __builtin_bit_cast(f4, __builtin_amdgcn_raw_buffer_load_b128(rs_, vo_, so_ + (f & 1) * s0_ + ((f >> 1) & 1) * s1_ + ((f >> 2) & 1) * s2_ + (f >> 3) * s3_, 0)); } while (0)
    unsigned i = (unsigned)wv * 8u + xc;
    DECODE_HX(i);
    const unsigned ctr_raw = __builtin_amdgcn_raw_buffer_load_b32(rsMy, 0, CTR2_OFF + b * 64u + (unsigned)wv * 4u, AUX_SYS);
    if (wv != 0) asm volatile("s_barrier" ::: "memory");
    LOAD_XW();
    __builtin_amdgcn_sched_barrier(0);
    asm volatile("" ::: "memory");
    if (wv == 0) asm volatile("s_barrier" ::: "memory");
    STAMP_ENTRY();
    if (wv == 0) STAMP(11);
    const unsigned long long t0 = now();
    const unsigned per = !RELAY || self ? 4u : (cross ? 3u : 5u);
    const unsigned rcp = per == 3u ? 3641u : per == 4u ? 2731u : 2185u;
    static_assert(NCLS == 6, "reciprocals");
    const unsigned nin = 128u + nhop + 64u * per;
    const unsigned nitems = nin + ((cross && !(EXP & 1)) ? NYI : 0u);
    u4* const tr = (u4*)&sm.xl[wv][0][0];
    const unsigned call = next_call(ctr_raw);
    if (wv == 0) STAMP(12);
    const unsigned dbase = (call & 1u) * PAR_STRIDE;
    const unsigned fbase = FLAG_OFF + (call & 1u) * FLAG_PAR;
    unsigned hopf = 0, fval = 0;
    bool bad = false;
    unsigned tk = (unsigned)wv, posted = 0;
    while (true) {
      const unsigned foff = (kind == 0 ? idx : 32u + idx) * 128u + (src * 4u + q) * 4u;
      const unsigned doff = src * SRC_STRIDE + (kind == 0 ? 0u : W_OFF) + (idx * 4u + q) * PIECE;
      const int lp = (lane >> 5) * 33 + (lane & 31);
      if (kind >= 2u) {
#pragma unroll
        for (int f = 0; f < 8; ++f) tr[66 * f + lp] = __builtin_bit_cast(u4, v[f]);
      } else {
        float m = 0.f;
#pragma unroll
        for (int f = 0; f < 16; ++f) {
          asm("v_max3_f32 %0, %1, |%2|, |%3|" : "=v"(m) : "v"(m), "v"(v[f][0]), "v"(v[f][1]));
          asm("v_max3_f32 %0, %1, |%2|, |%3|" : "=v"(m) : "v"(m), "v"(v[f][2]), "v"(v[f][3]));
        }
        const unsigned code = wave_code(__builtin_bit_cast(unsigned, m));
        const float scl = __builtin_bit_cast(float, (127u + 15u + 128u - code * 8u) << 23);
        fval = (call << 8) | code;
#pragma unroll
        for (int f = 0; f < 16; ++f) v[f] *= scl;
        if (kind == 0) {
#pragma unroll
          for (int f = 0; f < 8; ++f) {
            h8 r;
#pragma unroll
            for (int j = 0; j < 8; ++j) r[j] = (_Float16)v[2 * f + (j >> 2)][j & 3];
            tr[(lane & 15) * 33 + 4 * f + (lane >> 4)] = __builtin_bit_cast(u4, r);
          }
        } else {
#pragma unroll
          for (int f = 0; f < 8; ++f) {
            h8 r;
#pragma unroll
            for (int j = 0; j < 8; ++j) r[j] = (_Float16)v[(f >> 2) * 8 + j][f & 3];
            tr[66 * f + lp] = __builtin_bit_cast(u4, r);
          }
        }
      }
      asm volatile("" ::: "memory");
      if (posted != 0u) {
        if (lane == 0) tk = atomicAdd(&sm.tick, 1u);
        tk = __builtin_amdgcn_readfirstlane(tk);
      }
      {
        const bool isy = kind == 3;
        const unsigned so = isy ? (128u * (rank & 3u) + 32u * q) * 16384u + (2048u * h + 64u * idx) * 4u : dbase + doff;
        const u4 e = {((tk + 1u) << 8) | (isy ? 32u : 0u) | (unsigned)wv, so, isy ? WS_USED : fbase + foff, fval};
        asm volatile("" ::: "memory");
        if (lane == 0) sm.fifo[tk & 7u] = e;
        asm volatile("" ::: "memory");
      }
      if (tk < 60) STAMP(16 + 4 * tk);
      ++posted;
      unsigned nn = 0;
#define QFETCH() if (lane == 0) nn = __hip_atomic_fetch_add((unsigned*)(myws + QUEUE_OFF + (call & 1u) * 1024u + jb * 128u), 1u, __ATOMIC_RELAXED, __HIP_MEMORY_SCOPE_SYSTEM)
      if (DYN == 2) QFETCH();
      while (lds_ld(&sm.st[wv]) != posted) __builtin_amdgcn_s_sleep(1);
      asm volatile("" ::: "memory");
      if (DYN == 1) QFETCH();
      nn = DYN ? __builtin_amdgcn_readfirstlane(nn) + NW * 8u : i + NW * 8u;
      i = nn;
      if (i >= nitems) break;
      src = rank;
      if (i < 128u + nhop) DECODE_HX(i);
      else if (i >= nin) { kind = 3; idx = (i - nin) >> 2; q = (i - nin) & 3u; }
      else {
        const unsigned j = i - 128u - nhop;
        const unsigned r = (j * rcp) >> 16, rem = j - r * (NCLS * per);
        const unsigned ng = min((unsigned)NCLS, 64u - r * NCLS);
        const unsigned k = (rem * (ng == 6u ? 10923u : 16384u)) >> 16;
        idx = r * NCLS + rem - k * ng;
        kind = 1; q = k + 4u - per;
        if (per == 5u && k == 0u) { kind = 2; q = 0; src = 4u * (h ^ 1u) + ((rank - idx) & 3u); }
      }
      if (kind < 2u) LOAD_XW();
      else {
        unsigned so;
        if (kind == 3) {
          const unsigned yi = (d & 3u) * 128u + idx * 4u + q;
          while (true) {
            const unsigned yv = __builtin_amdgcn_raw_buffer_load_b32(rsMy, (lane & 1) * 4, YRDY_OFF + yi * 8u, AUX_SYS);
            if (__builtin_amdgcn_ballot_w64(yv == call) == ~0ull) break;
            if (now() - t0 > DEADLINE || __hip_atomic_load(tmo, __ATOMIC_RELAXED, __HIP_MEMORY_SCOPE_AGENT) == call) {
              __hip_atomic_store(tmo, call, __ATOMIC_RELAXED, __HIP_MEMORY_SCOPE_AGENT); bad = true; break; }
            __builtin_amdgcn_s_sleep(4);
            asm volatile("" ::: "memory");
          }
          so = YST_OFF + yi * PIECE;
        } else {
          while (true) {
            fval = __builtin_amdgcn_readlane(hopf, idx);
            if ((fval >> 8) == call) break;
            if (now() - t0 > DEADLINE || __hip_atomic_load(tmo, __ATOMIC_RELAXED, __HIP_MEMORY_SCOPE_AGENT) == call) {
              __hip_atomic_store(tmo, call, __ATOMIC_RELAXED, __HIP_MEMORY_SCOPE_AGENT); break; }
            hopf = __builtin_amdgcn_raw_buffer_load_b32(rsMy, (32u + lane) * 128u + ((rank - lane) & 3u) * 16u, fbase + (h ^ 1u) * 64u, AUX_SYS);
            asm volatile("" ::: "memory");
          }
          so = dbase + src * SRC_STRIDE + W_OFF + idx * 4u * PIECE;
        }
        asm volatile("" ::: "memory");
#pragma unroll
        for (int f = 0; f < 8; ++f) v[f] = __builtin_bit_cast(f4, __builtin_amdgcn_raw_buffer_load_b128(rsMy, lane * 16, so + f * 1024, AUX_SYS));
#pragma unroll
        for (int f = 8; f < 16; ++f) asm volatile("" : "=v"(v[f]));
      }
    }
    if (lane == 0) {
      tk = atomicAdd(&sm.tick, 1u);
      const u4 e = {((tk + 1u) << 8) | 128u | (bad ? 64u : 0u), 0u, 0u, call};
      sm.fifo[tk & 7u] = e;
      __builtin_amdgcn_raw_buffer_store_b32(call, rsMy, 0, CTR2_OFF + b * 64u + (unsigned)wv * 4u, AUX_SYS);
    }
    return;
  }
  if (jb < 8u && wv == NW) {
    if (lane < 8) { sm.fifo[lane] = (u4){0u, 0u, 0u, 0u}; sm.st[lane] = 0u; }
    if (lane == 0) sm.tick = NW;
    WG_BARRIER();
    __builtin_amdgcn_s_setprio(3);
    STAMP_ENTRY();
    const rsrc_t rsD = mk(pd, WS_USED);
    const rsrc_t rsO = mk(po, OUT_BYTES);
    const int lp = (lane >> 5) * 33 + (lane & 31);
    const int voy = (lane >> 4) * 16384 + (lane & 15) * 16;
    const unsigned fvo = lane == 0 ? 0u : WS_USED;
    unsigned pfo = WS_USED, pfv = 0, ends = 0, bad = 0, call = 0;
#define FLUSH_FLAG() do { if (pfo != WS_USED) { \
      asm volatile("s_waitcnt vmcnt(0)" ::: "memory"); \
      __builtin_amdgcn_raw_buffer_store_b32(pfv, rsD, fvo, pfo, AUX_SYS); \
      pfo = WS_USED; \
      if (pn < 60) STAMP(16 + 4 * pn + 2); } } while (0)
    unsigned pn = 0; (void)pn;
    for (unsigned n = 0; ends < NW; ++n) {
      asm volatile("" ::: "memory");
      u4 e = sm.fifo[n & 7u];
      unsigned e0 = __builtin_amdgcn_readfirstlane(e[0]);
      if ((e0 >> 8) != n + 1u) {
        FLUSH_FLAG();
        while ((e0 >> 8) != n + 1u) {
          __builtin_amdgcn_s_sleep(1);
          asm volatile("" ::: "memory");
          e = sm.fifo[n & 7u];
          e0 = __builtin_amdgcn_readfirstlane(e[0]);
        }
      }
      if ((e0 & 128u) != 0u) { ++ends; bad |= e0 & 64u; call = __builtin_amdgcn_readfirstlane(e[3]); continue; }
      asm volatile("" ::: "memory");
      const unsigned slot = e0 & 7u;
      const u4* const tr = (const u4*)&sm.xl[slot][0][0];
      u4 o[8];
#pragma unroll
      for (int f = 0; f < 8; ++f) o[f] = tr[66 * f + lp];
      asm volatile("" ::: "memory");
      if (lane == 0) atomicAdd(&sm.st[slot], 1u);
      asm volatile("" ::: "memory");
      const bool isy = (e0 & 32u) != 0u;
      const rsrc_t rs = isy ? rsO : rsD;
      const int vo = isy ? voy : lane * 16;
      const unsigned so = __builtin_amdgcn_readfirstlane(e[1]);
      const unsigned st = isy ? 4u * 16384u : 1024u;
#pragma unroll
      for (int f = 0; f < 8; ++f) {
        asm volatile("s_waitcnt vmcnt(%0)" :: "n"(((f + 8 - (WIN - 1)) % 8 != 0 && (f + 8 - (WIN - 1)) % 8 < WIN - 1) ? WIN : WIN - 1) : "memory");
        if (f == WIN - 1) {
          __builtin_amdgcn_raw_buffer_store_b32(pfv, rsD, fvo, pfo, AUX_SYS);
        }
        __builtin_amdgcn_raw_buffer_store_b128(o[f], rs, vo, so + f * st, AUX_ST);
        asm volatile("" ::: "memory");
      }
      pfo = __builtin_amdgcn_readfirstlane(e[2]);
      pfv = __builtin_amdgcn_readfirstlane(e[3]);
    }
    FLUSH_FLAG();
    asm volatile("s_waitcnt vmcnt(0)" ::: "memory");
    if (((rank + 1u + jb) ^ rank) & 4u) {
      if (lane == 0 && bad == 0u) __builtin_amdgcn_raw_buffer_store_b32(call, rsD, 0, YFLAG_OFF + ((rank & 3u) * 128u + 32u + xc) * 4u, AUX_SYS);
    }
    return;
  }
  STAMP_ENTRY();
  const unsigned long long t0 = now();
  if (jb < 8u) {
    if (b == 56u && wv == 9) {
      const unsigned ctr_raw = __builtin_amdgcn_raw_buffer_load_b32(rsMy, 0, CTR2_OFF + b * 64u + (unsigned)wv * 4u, AUX_SYS);
      asm volatile("s_barrier" ::: "memory");
      const unsigned call = next_call(ctr_raw);
      if (lane == 0) __builtin_amdgcn_raw_buffer_store_b32(call, rsMy, 0, CTR2_OFF + b * 64u + (unsigned)wv * 4u, AUX_SYS);
      if (lane < 8) __builtin_amdgcn_raw_buffer_store_b32(0u, rsMy, lane * 128, QUEUE_OFF + ((call & 1u) ^ 1u) * 1024u, AUX_SYS);
      unsigned spins = 0;
      const unsigned w0 = (unsigned)(lane & 31) * 4u;
      bool live[4];
#pragma unroll
      for (int c = 0; c < 4; ++c) live[c] = w0 < 32u ? (!(EXP & 1) && ((w0 + c) & 7u) < (unsigned)NCLS) : w0 < 40u;
      while (true) {
        const u4 v0 = __builtin_amdgcn_raw_buffer_load_b128(rsMy, lane * 16, YFLAG_OFF, AUX_SYS);
        const u4 v1 = __builtin_amdgcn_raw_buffer_load_b128(rsMy, lane * 16 + 1024, YFLAG_OFF, AUX_SYS);
        bool okk = true;
#pragma unroll
        for (int c = 0; c < 4; ++c) okk = okk && (!live[c] || (EXP ? (v0[c] >= call && v1[c] >= call) : (v0[c] == call && v1[c] == call)));
        if (__builtin_amdgcn_ballot_w64(okk) == ~0ull) break;
        if ((++spins & 255u) == 0u) {
          if (now() - t0 > DEADLINE || __hip_atomic_load(tmo, __ATOMIC_RELAXED, __HIP_MEMORY_SCOPE_AGENT) == call) {
            __hip_atomic_store(tmo, call, __ATOMIC_RELAXED, __HIP_MEMORY_SCOPE_AGENT);
            if (lane == 0) __builtin_amdgcn_raw_buffer_store_b32(0x7fc00000u, mk(a.outd[7], OUT_BYTES), 0, 2048u * (h ^ 1u) * 4u, AUX_SYS);
            break;
          }
        }
        __builtin_amdgcn_s_sleep(1);
        asm volatile("" ::: "memory");
      }
      STAMP(15);
    }
    return;
  }
  const unsigned ctr_raw = __builtin_amdgcn_raw_buffer_load_b32(rsMy, 0, CTR_OFF + b * 4u, AUX_SYS);
#define GET_CALL() \
  const unsigned call = next_call(ctr_raw); \
  const unsigned dbase = (call & 1u) * PAR_STRIDE; \
  const unsigned fbase = FLAG_OFF + (call & 1u) * FLAG_PAR; \
  WG_BARRIER(); \
  if (tid == NTHREADS - 64) __builtin_amdgcn_raw_buffer_store_b32(call, rsMy, 0, CTR_OFF + b * 4u, AUX_SYS)

  if (jb - 8u * SPL >= CPX) return;
  const unsigned cidx = xc * CPX + (jb - 8u * SPL);
  const unsigned cls = cidx >> 5;
  const unsigned rg = ((cidx & 1u) << 4) | ((cidx & 31u) >> 1);
  const int nt = (int)((64u - cls + NCLS - 1u) / NCLS);
  if (tid < (NT + 2) * 32) (&sm.mirror[0][0])[tid] = 0u;
  if (tid < 8) sm.st[tid] = 0u;
  if (tid == 0) sm.abort_ = 0u;
  GET_CALL();
  if (EXP & 1) { if (wv != 8 || (EXP & 2)) return; }

  if (wv < 8) {
    const unsigned v = (unsigned)wv;
    h8 xr[XR];
    u4 B0[8], B1[8];
    unsigned cx[4];
    {
      const unsigned xb = dbase + v * SRC_STRIDE + rg * 4u * PIECE;
#pragma unroll
      for (int q = 0; q < 4; ++q) cx[q] = wait_mirror(sm, &sm.mirror[0][v * 4 + q], call, t0) & 255u;
#pragma unroll
      for (int f = 0; f < 8; ++f) if (16 + f >= XR) B0[f] = __builtin_amdgcn_raw_buffer_load_b128(rsMy, lane * 16, xb + (16 + f) * 1024, AUX_LD);
#pragma unroll
      for (int f = 0; f < 8; ++f) B1[f] = __builtin_amdgcn_raw_buffer_load_b128(rsMy, lane * 16, xb + (24 + f) * 1024, AUX_LD);
#pragma unroll
      for (int f = 0; f < XR; ++f) xr[f] = __builtin_bit_cast(h8, __builtin_amdgcn_raw_buffer_load_b128(rsMy, lane * 16, xb + f * 1024, AUX_LD));
#pragma unroll
      for (int f = 0; f < 8; ++f) if (16 + f >= XR) sm.xl[v][16 + f - XR][lane] = B0[f];
#pragma unroll
      for (int f = 0; f < 8; ++f) sm.xl[v][24 + f - XR][lane] = B1[f];
    }
    STAMP(16 + wv);
    const int n = lane & 31, half = lane >> 5;
    const int voff = (n >> 3) * 1024 + half * 128 + (n & 7) * 16;
    const int col = 4 * (n & 7) + (n >> 3);
    const unsigned wb = dbase + v * SRC_STRIDE + W_OFF + cls * 4u * PIECE;
    f16v acc;
#pragma unroll
    for (int i = 0; i < 16; ++i) acc[i] = 0.f;
    int unit = 0;

#define LOADB(BUF, SO) do { const unsigned so_ = (SO); \
    _Pragma("unroll") for (int f = 0; f < 8; ++f) BUF[f] = __builtin_amdgcn_raw_buffer_load_b128(rsMy, voff, so_ + (f & 3) * 256 + (f >> 2) * 4096, AUX_LD); } while (0)
#define FLUSH(T, FINAL) do { wait_lds_zero(sm, &sm.st[v], t0); \
    float* tb_ = &sm.slot[v][4 * half * 32 + col]; \
    _Pragma("unroll") for (int i = 0; i < 16; ++i) { tb_[((i & 3) + 8 * (i >> 2)) * 32] = __builtin_ldexpf(acc[i], unit); acc[i] = 0.f; } \
    __builtin_amdgcn_fence(__ATOMIC_RELEASE, "workgroup"); \
    if (lane == 0) lds_st(&sm.st[v], 2u * (unsigned)(T) + 2u + (FINAL)); } while (0)
#define MMA(BUF, Q) do { _Pragma("unroll") for (int f = 0; f < 8; ++f) { \
      h8 av_; if ((Q) * 8 + f < XR) av_ = xr[(Q) * 8 + f < XR ? (Q) * 8 + f : 0]; else av_ = __builtin_bit_cast(h8, sm.xl[v][(Q) * 8 + f >= XR ? (Q) * 8 + f - XR : 0][lane]); \
      acc = __builtin_amdgcn_mfma_f32_32x32x16_f16(av_, __builtin_bit_cast(h8, BUF[f]), acc, 0, 0, 0); } } while (0)
#define STEP(CUR, NXT, Q) do { \
      const int u_ = (int)((cx[Q] + cw) * 8u) - 256 - 30; \
      if ((Q) == 0) unit = u_; \
      else if (u_ != unit) { FLUSH(t, 0u); unit = u_; } \
      const int tn_ = (Q) == 3 ? t + 1 : t; \
      const unsigned* mp_ = &sm.mirror[1 + tn_][v * 4 + (((Q) + 1) & 3)]; \
      const unsigned son_ = wb + (unsigned)tn_ * (NCLS * 4u * PIECE) + (((Q) + 1) & 3) * PIECE; \
      const bool more_ = tn_ < nt; \
      unsigned fw_ = lds_ld(mp_); \
      const bool rdy_ = more_ && (fw_ >> 8) == call; \
      asm volatile("" ::: "memory"); \
      LOADB(NXT, rdy_ ? son_ : WS_USED); \
      __builtin_amdgcn_sched_barrier(0); \
      MMA(CUR, Q); \
      if ((Q) == 3) FLUSH(t, 1u); \
      if (more_ && !rdy_) { fw_ = wait_mirror(sm, mp_, call, t0); LOADB(NXT, son_); } \
      cw = fw_ & 255u; \
    } while (0)

    unsigned cw = wait_mirror(sm, &sm.mirror[1][v * 4], call, t0) & 255u;
    LOADB(B0, wb);
    for (int t = 0; t < nt; ++t) {
      STEP(B0, B1, 0);
      STEP(B1, B0, 1);
      STEP(B0, B1, 2);
      STEP(B1, B0, 3);
      STAMP(128 + t * 8 + wv);
    }
  } else if (wv == 8) {
    __builtin_amdgcn_s_setprio(2);
    int u = 0;
    unsigned spins = 0;
    while (u <= nt) {
      const int ul = min(u + (lane >> 5), nt);
      const unsigned line = ul == 0 ? rg : 32u + (unsigned)(ul - 1) * NCLS + cls;
      const unsigned val = __builtin_amdgcn_raw_buffer_load_b32(rsMy, line * 128u + (lane & 31) * 4, fbase, AUX_SYS);
      lds_st(&sm.mirror[ul][lane & 31], val);
      const unsigned long long okm = __builtin_amdgcn_ballot_w64((val >> 8) == call);
      asm volatile("" ::: "memory");
      if ((unsigned)okm == 0xffffffffu) {
        STAMP(64 + u);
        if (okm == ~0ull) { STAMP(64 + u + 1); }
        u += (okm == ~0ull) ? 2 : 1;
        continue;
      }
      if ((++spins & 255u) == 0u) {
        if (now() - t0 > DEADLINE || __hip_atomic_load(tmo, __ATOMIC_RELAXED, __HIP_MEMORY_SCOPE_AGENT) == call) {
          __hip_atomic_store(tmo, call, __ATOMIC_RELAXED, __HIP_MEMORY_SCOPE_AGENT);
          lds_st(&sm.abort_, 1u);
          break;
        }
      }
      __builtin_amdgcn_s_sleep(PSLEEP);
    }
  } else {
    __builtin_amdgcn_s_setprio(2);
    const bool own = rg < 16u;
    const unsigned o = own ? rank : ((4u * (h ^ 1u) + ((rg >> 2) & 3u)) & 7u);
    const unsigned row0 = own ? 32u * rg : 128u * (rank & 3u) + 32u * (rg & 3u);
    const rsrc_t rsO = mk(a.outd[(o - rank - 1u) & 7u], OUT_BYTES);
    const int vo = (lane >> 3) * 16384 + (lane & 7) * 16;
    for (int t = 0; t < nt; ++t) {
      f4 r[4];
#pragma unroll
      for (int i = 0; i < 4; ++i) r[i] = (f4){0.f, 0.f, 0.f, 0.f};
      int nfin = 0;
      while (nfin < 8) {
        const unsigned sv = __hip_atomic_load(&sm.st[lane & 7], __ATOMIC_RELAXED, __HIP_MEMORY_SCOPE_WORKGROUP);
        const unsigned m = (unsigned)__builtin_amdgcn_ballot_w64((sv >> 1) == (unsigned)t + 1u) & 255u;
        if (m == 0u) {
          if (now() - t0 > 3 * DEADLINE) { lds_st(&sm.abort_, 1u); break; }
          __builtin_amdgcn_s_sleep(1);
          continue;
        }
        const unsigned mf = (unsigned)__builtin_amdgcn_ballot_w64((sv & 1u) != 0u) & m;
        __builtin_amdgcn_fence(__ATOMIC_ACQUIRE, "workgroup");
        unsigned mm = m;
        while (mm != 0u) {
          f4 p[4][4];
          const int cntm = __builtin_popcount(mm);
#pragma unroll
          for (int j = 0; j < 4; ++j) {
            const int v = __builtin_ctz(mm | 128u);
            mm &= mm - 1u;
#pragma unroll
            for (int i = 0; i < 4; ++i) p[j][i] = ((const f4*)&sm.slot[v][0])[i * 64 + lane];
          }
#pragma unroll
          for (int j = 0; j < 4; ++j)
            if (j < cntm) {
#pragma unroll
              for (int i = 0; i < 4; ++i) r[i] += p[j][i];
              asm volatile("" ::: "memory");
            }
        }
        __builtin_amdgcn_fence(__ATOMIC_RELEASE, "workgroup");
        if (lane < 8 && ((m >> lane) & 1u)) lds_st(&sm.st[lane], 0u);
        nfin += __builtin_popcount(mf);
      }
      if (lds_ld(&sm.abort_) != 0u) {
        const float nan = __builtin_nanf("");
#pragma unroll
        for (int i = 0; i < 4; ++i) r[i] = (f4){nan, nan, nan, nan};
      }
      const unsigned g = (unsigned)t * NCLS + cls;
      if (!own && g < NYI / 2u) {
        const unsigned yi = (o & 3u) * 128u + (g >> 1) * 4u + (rg & 3u);
#pragma unroll
        for (int i = 0; i < 4; ++i) __builtin_amdgcn_raw_buffer_store_b128(__builtin_bit_cast(u4, r[i]), rsMy, (lane >> 3) * 256 + (lane & 7) * 16, YST_OFF + yi * PIECE + (g & 1u) * 128u + (unsigned)i * 2048u, AUX_SYS);
        asm volatile("s_waitcnt vmcnt(0)" ::: "memory");
        if (lane == 0) __builtin_amdgcn_raw_buffer_store_b32(call, rsMy, 0, YRDY_OFF + yi * 8u + (g & 1u) * 4u, AUX_SYS);
      } else {
        const unsigned so = row0 * 16384u + (2048u * h + 32u * g) * 4u;
#pragma unroll
        for (int i = 0; i < 4; ++i) __builtin_amdgcn_raw_buffer_store_b128(__builtin_bit_cast(u4, r[i]), rsO, vo, so + (unsigned)i * 8u * 16384u, AUX_SYS);
      }
      STAMP(96 + t);
    }
    asm volatile("s_waitcnt vmcnt(0)" ::: "memory");
    STAMP(14);
    if (!own) {
      const rsrc_t rsD = mk(a.wsd[(o - rank - 1u) & 7u], WS_USED);
      if (lane == 0) __builtin_amdgcn_raw_buffer_store_b32(call, rsD, 0, YFLAG_OFF + ((rank & 3u) * 128u + (rg & 3u) * 8u + cls) * 4u, AUX_SYS);
    }
  }
}

extern "C" void kernel_launch(int rank, int nranks, ncclComm_t comm,
                              hipStream_t stream,
                              void* const* d_in, const int* in_sizes, int n_in,
                              void* d_out, int out_size,
                              const int* in_dtypes, int out_dtype,
                              void* const* d_out_all,
                              void* const* d_ws_all, size_t ws_bytes) {
  (void)comm; (void)in_dtypes; (void)out_dtype; (void)d_out;
  if (nranks != NR || rank < 0 || rank >= NR || n_in < 2 || ws_bytes < WS_USED ||
      in_sizes[0] != 4096 * 512 || in_sizes[1] != 512 * 4096 || out_size != 512 * 4096) {
    fprintf(stderr, "kernel_launch: unsupported configuration\n");
    return;
  }
  Args a;
  a.x = (const float*)d_in[0];
  a.w = (const float*)d_in[1];
  for (int p = 0; p < NR; ++p) { a.wsd[p] = (char*)d_ws_all[(rank + 1 + p) & 7]; a.outd[p] = (float*)d_out_all[(rank + 1 + p) & 7]; }
  a.myws = (char*)d_ws_all[rank];
  a.rank = (unsigned)rank;
  a.pad_ = 0u;
  hipLaunchKernelGGL(fused_gemm_kernel, dim3(256), dim3(NTHREADS), 0, stream, a);
  HIP_CHECK(hipGetLastError());
}
```

```cpp
#include <hip/hip_runtime.h>
#include <rccl/rccl.h>
#include <cstdio>

#define HIP_CHECK(expr) do { hipError_t _e = (expr); if (_e != hipSuccess) { \
    fprintf(stderr, "HIP error %s at %s:%d\n", hipGetErrorString(_e), __FILE__, __LINE__); \
    return; } } while (0)

#define WIN 6
#define PW 8
#define EXP 0
#define PSLEEP 1
#define NCLS 6
#define NW 2
#define NF 3
#define SPL 1
#define NREP 4
#define RR 11u
#define L3 4u
#define SHADOW 1
#define EPRIO 2
#define CSLEEP 1
#define FDELAY 64
#define YSLEEP 1
#define FSLEEP 8
#define NYI 120u
#define CDELAY 96

typedef float f4 __attribute__((ext_vector_type(4)));
typedef float f16v __attribute__((ext_vector_type(16)));
typedef unsigned u4 __attribute__((ext_vector_type(4)));
typedef _Float16 h8 __attribute__((ext_vector_type(8)));
typedef __amdgpu_buffer_rsrc_t rsrc_t;

constexpr int NR = 8;
constexpr unsigned PIECE = 8192;
constexpr unsigned SRC_STRIDE = 384u * PIECE;
constexpr unsigned W_OFF = 128u * PIECE;
constexpr unsigned PAR_STRIDE = 24u << 20;
constexpr unsigned FLAG_OFF = 48u << 20;
constexpr unsigned FL_OFF = FLAG_OFF + (704u << 10);
constexpr unsigned FX_REL = 6144;
constexpr unsigned FZ_REL = 10240;
constexpr unsigned REP_STRIDE = 32768u + 4352u;
constexpr unsigned FLAG_PAR = 16384;
constexpr unsigned CTR_OFF = FLAG_OFF + 36864;
constexpr unsigned TMO_OFF = FLAG_OFF + 40960;
constexpr unsigned QUEUE_OFF = FLAG_OFF + 45056;
constexpr unsigned YRDY_OFF = FLAG_OFF + 49152;
constexpr unsigned CTR2_OFF = FLAG_OFF + 53248;
constexpr unsigned HF_OFF = FLAG_OFF + 57344;
constexpr unsigned PRDY_OFF = FLAG_OFF + 61440;
constexpr unsigned AV_OFF = FLAG_OFF + (640u << 10);
constexpr unsigned DBG_OFF = FLAG_OFF + 65536;
constexpr unsigned YST_OFF = FLAG_OFF + (1u << 20);
constexpr unsigned OWN_OFF = FLAG_OFF + (6u << 20);
constexpr unsigned OWN_PAR = 8u << 20;
constexpr unsigned WS_USED = OWN_OFF + 2u * OWN_PAR;
constexpr unsigned OUT_BYTES = 512u * 4096u * 4u;
constexpr unsigned IN_BYTES = 512u * 4096u * 4u;
constexpr unsigned long long DEADLINE = 50000000ull;
constexpr int AUX_SYS = 17;
#define AUX_LD 16
constexpr int XR = 17;
constexpr int NTHREADS = 640;
constexpr int NT = (64 + NCLS - 1) / NCLS;
constexpr unsigned RG = RR * NCLS < 64u ? RR * NCLS : 64u;
constexpr unsigned NL = 2u * L3;
constexpr unsigned CPX = 4u * NCLS;

struct Args {
  const float* x;
  const float* w;
  char* myws;
  unsigned rank;
  unsigned pad_;
  char* wsd[NR];
  float* outd[NR];
};

struct __attribute__((aligned(128))) Slot {
  float d[32 * 32];
  unsigned st, pad_[31];
};
struct __attribute__((aligned(128))) Smem {
  u4 xl[8][32 - XR][64];
  Slot slot[8];
  unsigned st[8];
  unsigned abort_;
  unsigned tick, pad_[2];
  u4 fifo[8];
  u4 fifo2[8];
};

__device__ __forceinline__ rsrc_t mk(const void* p, unsigned bytes) {
  return __builtin_amdgcn_make_buffer_rsrc((void*)p, (short)0, (int)bytes, 0x00020000);
}
__device__ __forceinline__ unsigned lds_ld(const unsigned* p) {
  return __builtin_amdgcn_readfirstlane(__hip_atomic_load(p, __ATOMIC_RELAXED, __HIP_MEMORY_SCOPE_WORKGROUP));
}
__device__ __forceinline__ void lds_st(unsigned* p, unsigned v) {
  __hip_atomic_store(p, v, __ATOMIC_RELAXED, __HIP_MEMORY_SCOPE_WORKGROUP);
}
__device__ __forceinline__ unsigned lds_addr(const void* p) { return (unsigned)(size_t)(const __attribute__((address_space(3))) void*)p; }
__device__ __forceinline__ unsigned poll1(const char* p, unsigned off) {
  unsigned v;
  asm volatile("s_load_dword %0, %1, %2 glc\n\ts_waitcnt lgkmcnt(0)" : "=s"(v) : "s"(p), "s"(off) : "memory");
  return v;
}
__device__ __forceinline__ unsigned long long poll2(const char* p, unsigned off) {
  unsigned long long v;
  asm volatile("s_load_dwordx2 %0, %1, %2 glc\n\ts_waitcnt lgkmcnt(0)" : "=s"(v) : "s"(p), "s"(off) : "memory");
  return v;
}
__device__ __forceinline__ unsigned long long now() { return __builtin_amdgcn_s_memrealtime(); }

#define STAMP(slot) do {} while (0)

__device__ __forceinline__ unsigned wave_code(unsigned m) {
  const unsigned c = ((m >> 23) + 1u) >> 4;
  unsigned r = 0;
#pragma unroll
  for (unsigned b = 8; b; b >>= 1)
    if (__builtin_amdgcn_ballot_w64(c >= (r | b)) != 0ull) r |= b;
  return r;
}
__device__ __forceinline__ unsigned tag_of(unsigned call) { return (call % 15u + 1u) << 4; }
__device__ __forceinline__ unsigned wflag(unsigned g, unsigned sx) { const unsigned t = (g * 43u) >> 8; return ((g - t * NCLS) * 8u + sx) * 128u + t * 4u; }

#define WG_BARRIER() asm volatile("s_waitcnt lgkmcnt(0)\n\ts_barrier" ::: "memory")

__device__ __forceinline__ unsigned next_call(unsigned raw) {
  unsigned call = __builtin_amdgcn_readfirstlane(raw) + 1u;
  if (call > 0xfffffeu) call = 1u;
  asm volatile("" : "+s"(call) :: "memory");
  return call;
}
__device__ __forceinline__ char* lane_ws(unsigned r, unsigned rank) {
  typedef char* const __attribute__((address_space(4))) * kp_t;
  return ((kp_t)((const __attribute__((address_space(4))) char*)__builtin_amdgcn_kernarg_segment_ptr() + offsetof(Args, wsd)))[(r - rank - 1u) & 7u];
}
__device__ __forceinline__ void flag_st(char* p, unsigned v) {
  asm volatile("global_store_byte %0, %1, off sc0 sc1" :: "v"(p), "v"(v) : "memory");
}

__device__ __forceinline__ void load_piece(f4 (&v)[16], const float* px, const float* pw, bool isx, unsigned G, unsigned q, int lane) {
  const rsrc_t rs = mk(isx ? px : pw, IN_BYTES);
  const int vo = isx ? (lane >> 4) * 2048 + (lane & 15) * 16 : (lane >> 3) * 4 * 16384 + (lane & 7) * 16;
  const unsigned so = isx ? G * 65536u + q * 512u : q * 128u * 16384u + G * 128u;
  const unsigned s0 = isx ? 256u : 16384u, s1 = isx ? 8192u : 32768u, s2 = isx ? 16384u : 64u * 16384u, s3 = isx ? 32768u : 32u * 16384u;
#pragma unroll
  for (int f = 0; f < 16; ++f)
    v[f] = __builtin_bit_cast(f4, __builtin_amdgcn_raw_buffer_load_b128(rs, vo, so + (f & 1) * s0 + ((f >> 1) & 1) * s1 + ((f >> 2) & 1) * s2 + (f >> 3) * s3, 0));
}
__device__ __forceinline__ unsigned conv_piece(f4 (&v)[16], bool isx, u4* tr, int lane) {
  float m = 0.f, m2 = 0.f;
#pragma unroll
  for (int f = 0; f < 16; ++f) {
    asm("v_max3_f32 %0, %1, |%2|, |%3|" : "=v"(m) : "v"(m), "v"(v[f][0]), "v"(v[f][1]));
    asm("v_max3_f32 %0, %1, |%2|, |%3|" : "=v"(m2) : "v"(m2), "v"(v[f][2]), "v"(v[f][3]));
  }
  const unsigned code = wave_code(max(__builtin_bit_cast(unsigned, m), __builtin_bit_cast(unsigned, m2)));
  const float scl = __builtin_bit_cast(float, (127u + 15u + 112u - code * 16u) << 23);
#pragma unroll
  for (int f = 0; f < 16; ++f) v[f] *= scl;
  if (isx) {
#pragma unroll
    for (int f = 0; f < 8; ++f) {
      h8 r;
#pragma unroll
      for (int j = 0; j < 8; ++j) r[j] = (_Float16)v[2 * f + (j >> 2)][j & 3];
      tr[(lane & 15) * 33 + 4 * f + (lane >> 4)] = __builtin_bit_cast(u4, r);
    }
  } else {
#pragma unroll
    for (int f = 0; f < 8; ++f) {
      h8 r;
#pragma unroll
      for (int j = 0; j < 8; ++j) r[j] = (_Float16)v[(f >> 2) * 8 + j][f & 3];
      tr[((f >> 2) * 8 + (lane >> 3)) * 33 + 4 * (lane & 7) + (f & 3)] = __builtin_bit_cast(u4, r);
    }
  }
  asm volatile("" ::: "memory");
  return code;
}
__device__ __forceinline__ unsigned xgroup(unsigned d, unsigned rg) {
  return rg < 16u ? 16u * d + rg : 16u * (((d & 4u) ^ 4u) + ((rg >> 2) & 3u)) + 4u * (d & 3u) + (rg & 3u);
}

__device__ __forceinline__ unsigned xitem(unsigned d, unsigned j) {
  return ((j & 4u) ? 16u + 4u * ((d - (j >> 5)) & 3u) : 4u * ((d + (j >> 5)) & 3u)) + ((j >> 3) & 3u);
}

__global__ void __launch_bounds__(NTHREADS) fused_gemm_kernel(Args a) {
  __shared__ Smem sm;
#define STAMP_ENTRY() do {} while (0)
  const int tid = threadIdx.x;
  const int lane = tid & 63;
  const int wv = __builtin_amdgcn_readfirstlane(tid >> 6);
  const unsigned rank = a.rank & 7u;
  const unsigned h = rank >> 2;
  const unsigned b = blockIdx.x;
  const unsigned xc = b & 7u, jb = (b >> 3) & 31u;
  char* const myws = a.myws;
  const float* const px = a.x;
  const float* const pw = a.w;
  char* const pd = a.wsd[jb & 7u];
  asm volatile("" :: "s"(px), "s"(pw), "s"(myws), "s"(rank), "s"(pd));
  const rsrc_t rsMy = mk(myws, WS_USED);
  unsigned* const tmo = (unsigned*)(myws + TMO_OFF);
#define TIMED_OUT() (now() - t0 > DEADLINE || __hip_atomic_load(tmo, __ATOMIC_RELAXED, __HIP_MEMORY_SCOPE_AGENT) == call)
#define SET_TIMEOUT() __hip_atomic_store(tmo, call, __ATOMIC_RELAXED, __HIP_MEMORY_SCOPE_AGENT)

  if (__builtin_expect(jb < 7u && wv < NW, 1)) {
    static_assert(NW >= 2 && NW + 2 + NF <= 10, "hop pieces are pushed");
    const unsigned p = (rank + 1u + jb) & 7u;
    const unsigned nhop = ((p ^ rank) & 4u) * 4u;
    const unsigned ctr_raw = __builtin_amdgcn_raw_buffer_load_b32(rsMy, 0, CTR2_OFF + b * 64u + (unsigned)wv * 4u, AUX_SYS);
    const unsigned i = (unsigned)wv * 8u + xc;
    const bool isx = i >= nhop;
    const unsigned idx = isx ? xitem(p, i - nhop) : i * 4u + ((jb + 1u) & 3u), q = isx ? i & 3u : 0u;
    f4 v[16];
    if (wv != 0) asm volatile("s_barrier" ::: "memory");
    load_piece(v, px, pw, isx, isx ? xgroup(p, idx) : (p & 4u) * 16u + idx, q, lane);
    __builtin_amdgcn_sched_barrier(0);
    asm volatile("" ::: "memory");
    if (wv == 0) asm volatile("s_barrier" ::: "memory");
    STAMP_ENTRY();
    const unsigned call = next_call(ctr_raw);
    const unsigned code = conv_piece(v, isx, (u4*)&sm.xl[wv][0][0], lane);
    const u4 e = {isx ? 1u : HF_OFF + (call & 1u) * 1024u + (p & 3u) * 256u + idx,
                  (call & 1u) * PAR_STRIDE + (rank ^ p) * SRC_STRIDE + (isx ? 0u : W_OFF) + (idx * 4u + q) * PIECE,
                  FL_OFF + (call & 1u) * FLAG_PAR + (isx ? FX_REL + idx * 128u + (rank ^ p) * 4u : wflag(idx, rank ^ p)) + q, tag_of(call) | code};
    if (lane == 0) sm.fifo[wv] = e;
    asm volatile("" ::: "memory");
    STAMP(16 + 4 * wv);
    if (lane == 0) __builtin_amdgcn_raw_buffer_store_b32(call, rsMy, 0, CTR2_OFF + b * 64u + (unsigned)wv * 4u, AUX_SYS);
    return;
  }
  if (jb < 7u && wv >= NW + 2 && wv < NW + 2 + NF) {
    const unsigned p = (rank + 1u + jb) & 7u;
    const bool cross = ((p ^ rank) & 4u) != 0u;
    const unsigned nhop = cross ? 16u : 0u;
    const unsigned fi = (unsigned)wv - (NW + 2);
    asm volatile("s_barrier" ::: "memory");
    const unsigned ctr_raw = __builtin_amdgcn_raw_buffer_load_b32(rsMy, 0, CTR2_OFF + b * 64u + (unsigned)wv * 4u, AUX_SYS);
    STAMP_ENTRY();
    const unsigned long long t0 = now();
    const unsigned call = next_call(ctr_raw);
    if (lane == 0) __builtin_amdgcn_raw_buffer_store_b32(call, rsMy, 0, CTR2_OFF + b * 64u + (unsigned)wv * 4u, AUX_SYS);
    const unsigned per = cross ? 3u : 5u;
    const unsigned rcp = cross ? 3641u : 2185u;
    const unsigned nin = 128u + nhop + RG * per + (64u - RG) * 4u;
    const unsigned nx = cross ? 144u : 128u - 4u * NL, nl = cross ? nin : nin - 4u * NL;
    const unsigned nitems = nin + ((cross && !(EXP & 1)) ? NYI : 0u);
    const unsigned dbase = (call & 1u) * PAR_STRIDE;
    const unsigned frel = (call & 1u) * FLAG_PAR;
    const unsigned obase = OWN_OFF + (call & 1u) * OWN_PAR;
    const unsigned avb = AV_OFF + frel + p * 512u, hfb = HF_OFF + (call & 1u) * 1024u + (p & 3u) * 256u;
    const unsigned tag = tag_of(call);
    unsigned posted = 0, spins = 0;
    unsigned i = (NW + fi) * 8u + xc;
    __builtin_amdgcn_s_sleep(FDELAY);
    while (i < nitems) {
      unsigned so, dof, fo = 0, fv = 0, isy = 0;
      bool bad = false;
      if (i >= nin) {
        const unsigned j = i - nin;
        so = YST_OFF + ((rank & 3u) * 128u + j) * PIECE;
        dof = (128u * (p & 3u) + 32u * (j & 3u)) * 16384u + (512u * (p & 4u) + 64u * (j >> 2)) * 4u;
        isy = 32u;
        while (!(EXP & 8)) {
          if (poll2(myws, YRDY_OFF + ((p & 3u) * 128u + j) * 8u) == call * 0x100000001ull) break;
          if ((++spins & 63u) == 0u && TIMED_OUT()) { SET_TIMEOUT(); bad = true; break; }
          __builtin_amdgcn_s_sleep(YSLEEP);
          asm volatile("" ::: "memory");
        }
        if (bad && lane == 0) __builtin_amdgcn_raw_buffer_store_b32(0x7fc00000u, mk(a.outd[7], OUT_BYTES), 0, dof, AUX_SYS);
      } else {
        unsigned idx, q;
        const bool isx = i < nx || i >= nl;
        bool fwd = false;
        if (isx) {
          unsigned j = i - nhop;
          if (i >= nl) { const unsigned e = i - nl; j = 8u * (16u - NL + 2u * (e >> 3) + (e & 1u)) + (((e >> 1) + (p ^ rank)) & 3u); }
          else if (!cross && j >= 8u * (16u - NL)) j = 2u * (j & ~3u) - 8u * (16u - NL) + 4u + (j & 3u);
          idx = xitem(rank, j); q = j & 3u;
        } else {
          unsigned j = i - nx;
          const bool rel = j < RG * per;
          if (!rel) j -= RG * per;
          const unsigned pe = rel ? per : 4u;
          const unsigned r = (j * (rel ? rcp : 2731u)) >> 16, rem = j - r * (NCLS * pe);
          const unsigned g0 = r * NCLS + (rel ? 0u : RG);
          const unsigned ng = min((unsigned)NCLS, 64u - g0);
          const unsigned k = (rem * (ng == 6u ? 10923u : 16384u)) >> 16;
          idx = g0 + rem - k * ng;
          fwd = pe == 5u && k == 0u;
          q = fwd ? 0u : k + 4u - pe;
          bad = cross && q == 0u && ((idx - rank + p) & 3u) == 0u;
        }
        const unsigned src = fwd ? 4u * (h ^ 1u) + ((p - idx) & 3u) : p;
        const unsigned po = dbase + (isx ? 0u : W_OFF) + (idx * 4u + q) * PIECE;
        dof = po + (src ^ rank) * SRC_STRIDE;
        fo = frel + (isx ? FX_REL + idx * 128u + (src ^ rank) * 4u : wflag(idx, src ^ rank)) + q;
        if (i >= nl) {
          const unsigned e = i - nl, hq = (e & 1u) * 4u + q, ij = (e >> 3) * 4u + (p ^ rank);
          dof = dbase + (hq * 4u * L3 + ij) * PIECE;
          fo = frel + FZ_REL + hq * 128u + ij;
        }
        so = fwd ? po + (src ^ p) * SRC_STRIDE : obase + ((isx ? xgroup(rank, idx) : 128u + 64u * h + idx) * 4u + q) * PIECE;
        const unsigned ao = fwd ? hfb + idx : avb + (isx ? idx : 32u + idx) * 4u + q;
        while (!bad) {
          fv = (poll1(myws, ao & ~3u) >> (8u * (ao & 3u))) & 255u;
          if ((fv & 0xf0u) == tag) break;
          if ((++spins & 63u) == 0u && TIMED_OUT()) { SET_TIMEOUT(); bad = true; break; }
          __builtin_amdgcn_s_sleep(FSLEEP);
          asm volatile("" ::: "memory");
        }
      }
      if (!bad) {
        unsigned tk = 0;
        if (lane == 0) tk = atomicAdd(&sm.tick, 1u);
        tk = __builtin_amdgcn_readfirstlane(tk);
        const u4 e = {((tk + 1u) << 24) | (fo << 6) | isy | fi, so, dof, fv};
        asm volatile("" ::: "memory");
        if (lane == 0) sm.fifo2[tk & 7u] = e;
        asm volatile("" ::: "memory");
        if (tk + NW < 60) STAMP(16 + 4 * (tk + NW));
        ++posted;
        while (lds_ld(&sm.st[fi]) != posted) __builtin_amdgcn_s_sleep(1);
        asm volatile("" ::: "memory");
      }
      i = 0;
      if (lane == 0) i = __hip_atomic_fetch_add((unsigned*)(myws + QUEUE_OFF + (call & 1u) * 1024u + jb * 128u), 1u, __ATOMIC_RELAXED, __HIP_MEMORY_SCOPE_SYSTEM);
      i = __builtin_amdgcn_readfirstlane(i) + (NW + NF) * 8u;
    }
    if (lane == 0) {
      const unsigned tk = atomicAdd(&sm.tick, 1u);
      sm.fifo2[tk & 7u] = (u4){((tk + 1u) << 24) | 16u, 0u, 0u, 0u};
    }
    return;
  }
  if (jb < 7u && wv == NW) {
    if (lane < 8) { sm.fifo[lane] = (u4){0u, 0u, 0u, 0u}; sm.fifo2[lane] = (u4){0u, 0u, 0u, 0u}; sm.st[lane] = 0u; }
    if (lane == 0) sm.tick = 0u;
    WG_BARRIER();
    __builtin_amdgcn_s_setprio(3);
    STAMP_ENTRY();
    const rsrc_t rsD = mk(pd, WS_USED);
    const int lp = (lane >> 5) * 33 + (lane & 31);
    const unsigned p = (rank + 1u + jb) & 7u;
    char* const fws = lane_ws((p & 4u) | ((p + (unsigned)lane) & 3u), rank);
    unsigned pfo = TMO_OFF + 64u, pfv = 0;
    bool pfl = false;
    for (unsigned n = 0; n < NW; ++n) {
      u4 e;
      while (true) {
        asm volatile("" ::: "memory");
        e = sm.fifo[n];
        if (__builtin_amdgcn_readfirstlane(e[0]) != 0u) break;
      }
      asm volatile("" ::: "memory");
      const u4* const tr = (const u4*)&sm.xl[n][0][0];
      u4 o[8];
#pragma unroll
      for (int f = 0; f < 8; ++f) o[f] = tr[66 * f + lp];
      const unsigned so = __builtin_amdgcn_readfirstlane(e[1]);
#pragma unroll
      for (int f = 0; f < 8; ++f) {
        asm volatile("s_waitcnt vmcnt(%0)" :: "n"(((f + 8 - (WIN - 1)) % 8 != 0 && (f + 8 - (WIN - 1)) % 8 < WIN - 1) ? WIN : WIN - 1) : "memory");
        if (f == WIN - 1 && n != 0) { if (pfl) flag_st(fws + pfo, pfv); }
        __builtin_amdgcn_raw_buffer_store_b128(o[f], rsD, lane * 16, so + f * 1024, AUX_SYS);
        asm volatile("" ::: "memory");
      }
      pfo = (lane & 3) == 0 ? e[2] + (unsigned)(lane >> 2) * REP_STRIDE : e[0];
      pfv = e[3];
      pfl = (lane & 3) == 0 ? lane < 4 * NREP : (lane < 4 && e[0] != 1u);
    }
    asm volatile("s_waitcnt vmcnt(0)" ::: "memory");
    if (pfl) flag_st(fws + pfo, pfv);
    STAMP(12);
    return;
  }
  if (jb < 7u && wv == NW + 1) {
    asm volatile("s_barrier" ::: "memory");
    __builtin_amdgcn_s_setprio(3);
    STAMP_ENTRY();
    const rsrc_t rsP = mk(pd, WS_USED);
    const rsrc_t rsO = mk(a.outd[7], OUT_BYTES);
    const int voy = (lane >> 4) * 16384 + (lane & 15) * 16;
    const int vol = lane * 16;
    const unsigned fvo = lane < NREP ? (unsigned)lane * REP_STRIDE : WS_USED;
    unsigned d1 = WS_USED, y1 = 0, fo1 = WS_USED, fv1 = 0, fo2 = WS_USED, fv2 = 0, ends = 0, n = 0;
    u4 e = {0u, 0u, 0u, 0u};
    while (true) {
      unsigned e0 = __builtin_amdgcn_readfirstlane(e[0]);
      if ((e0 >> 24) != ((n + 1u) & 255u)) {
        asm volatile("" ::: "memory");
        e = sm.fifo2[n & 7u];
        e0 = __builtin_amdgcn_readfirstlane(e[0]);
      }
      unsigned so = WS_USED, d0 = WS_USED, y0 = 0, fo0 = WS_USED, fv0 = 0;
      if ((e0 >> 24) == ((n + 1u) & 255u)) {
        ++n;
        if ((e0 & 16u) != 0u) { ++ends; continue; }
        so = __builtin_amdgcn_readfirstlane(e[1]);
        d0 = __builtin_amdgcn_readfirstlane(e[2]);
        fv0 = __builtin_amdgcn_readfirstlane(e[3]);
        y0 = e0 & 32u;
        fo0 = y0 ? WS_USED : FL_OFF + ((e0 >> 6) & 0x7fffu);
        asm volatile("" ::: "memory");
        if (lane == 0) __hip_atomic_fetch_add(&sm.st[e0 & 7u], 1u, __ATOMIC_RELAXED, __HIP_MEMORY_SCOPE_WORKGROUP);
      } else if (d1 == WS_USED && fo2 == WS_USED) {
        if (ends == NF) break;
        __builtin_amdgcn_s_sleep(1);
        continue;
      }
      const rsrc_t rs = y1 ? rsO : rsMy;
      const int vo = y1 ? voy : vol;
      const unsigned st = y1 ? 4u * 16384u : 1024u;
      {
        unsigned da = d1, sb = so + 4096u;
#define PSTEP(F, R, SO, OFS) "s_waitcnt vmcnt(%[w" #F "])\n\t" PFLAG##F \
        "buffer_store_dwordx4 " R ", %[vo], %[rs], %[da] offen sc0 sc1\n\t" \
        "buffer_load_dwordx4 " R ", %[vol], %[rp], %[" SO "] offen offset:" #OFS " sc0 sc1\n\t" \
        "s_add_u32 %[da], %[da], %[st]\n\t"
#define PFLAG0
#define PFLAG1
#define PFLAG2
#define PFLAG3
#define PFLAG4
#define PFLAG5
#define PFLAG6
#define PFLAG7 "buffer_store_byte %[fv], %[fvo], %[rm], %[fo] offen sc0 sc1\n\t"
#define PWAIT(F) [w##F] "n"(14 - 2 * (8 - PW) + (F + 8 - PW < 7 ? 1 : 0))
#define POPS : [da] "+s"(da) \
          : [vo] "v"(vo), [vol] "v"(vol), [fv] "v"(fv2), [fvo] "v"(fvo), [rs] "s"(rs), [rp] "s"(rsP), [rm] "s"(rsMy), \
            [sa] "s"(so), [sb] "s"(sb), [st] "s"(st), [fo] "s"(fo2), \
            PWAIT(0), PWAIT(1), PWAIT(2), PWAIT(3), PWAIT(4), PWAIT(5), PWAIT(6), PWAIT(7) \
          : "memory", "scc", "v136", "v137", "v138", "v139", "v140", "v141", "v142", "v143", "v144", "v145", "v146", "v147", "v148", "v149", "v150", "v151", \
            "v152", "v153", "v154", "v155", "v156", "v157", "v158", "v159", "v160", "v161", "v162", "v163", "v164", "v165", "v166", "v167"
        asm volatile("s_nop 4\n\t"
          PSTEP(0, "v[136:139]", "sa", 0) PSTEP(1, "v[140:143]", "sa", 1024) PSTEP(2, "v[144:147]", "sa", 2048) PSTEP(3, "v[148:151]", "sa", 3072)
          PSTEP(4, "v[152:155]", "sb", 0) PSTEP(5, "v[156:159]", "sb", 1024) PSTEP(6, "v[160:163]", "sb", 2048) POPS);
        e = sm.fifo2[n & 7u];
        asm volatile(PSTEP(7, "v[164:167]", "sb", 3072) POPS);
      }
      fo2 = fo1; fv2 = fv1;
      d1 = d0; y1 = y0; fo1 = fo0; fv1 = fv0;
    }
    asm volatile("s_waitcnt vmcnt(0)" ::: "memory");
    return;
  }
  STAMP_ENTRY();
  const unsigned long long t0 = now();
  if (jb < 8u) {
    if (b == 56u && wv == 0) {
      const unsigned ctr_raw = __builtin_amdgcn_raw_buffer_load_b32(rsMy, 0, CTR2_OFF + b * 64u, AUX_SYS);
      const unsigned call = next_call(ctr_raw);
      if (lane == 0) __builtin_amdgcn_raw_buffer_store_b32(call, rsMy, 0, CTR2_OFF + b * 64u, AUX_SYS);
      if (lane < 8) __builtin_amdgcn_raw_buffer_store_b32(0u, rsMy, lane * 128, QUEUE_OFF + ((call & 1u) ^ 1u) * 1024u, AUX_SYS);
      static_assert(NYI >= 96u && NYI % 4u == 0u, "");
      unsigned spins = 0;
      const bool live = !(EXP & 1) && 96u + 2u * (lane & 15) >= NYI;
      const unsigned ytag = call;
      while (true) {
        const u4 yv = __builtin_amdgcn_raw_buffer_load_b128(rsMy, (lane >> 4) * 1024 + (lane & 15) * 16, YRDY_OFF + 768u, AUX_SYS);
        if (__builtin_amdgcn_ballot_w64(!live || (yv[0] == ytag && yv[1] == ytag && yv[2] == ytag && yv[3] == ytag)) == ~0ull) break;
        if ((++spins & 255u) == 0u) {
          if (TIMED_OUT()) {
            SET_TIMEOUT();
            if (lane == 0) __builtin_amdgcn_raw_buffer_store_b32(0x7fc00000u, mk(a.outd[7], OUT_BYTES), 0, (2048u * (h ^ 1u) + 2047u) * 4u, AUX_SYS);
            break;
          }
        }
        __builtin_amdgcn_s_sleep(1);
        asm volatile("" ::: "memory");
      }
      STAMP(15);
    }
    return;
  }

  const unsigned ctr_raw = __builtin_amdgcn_raw_buffer_load_b32(rsMy, 0, CTR_OFF + b * 4u, AUX_SYS);
  const unsigned cidx = xc * CPX + (jb - 8u);
  const unsigned cls = cidx >> 5;
  const unsigned rg = ((cidx & 1u) << 4) | ((cidx & 31u) >> 1);
  static_assert(NCLS == 6, "");
  const int nt = cidx < 128u ? NT : NT - 1;
  if (tid < 8) sm.slot[tid].st = 0u;
  if (tid == 0) sm.abort_ = 0u;
  unsigned call;
  {
    const unsigned c = (unsigned)wv * 192u + (b - 64u);
    const bool isx = c < 512u;
    const bool cv = c < 1024u - RG;
    const unsigned cg = c < 960u ? ((c - 512u) * 9363u) >> 16 : (c - 960u + RG) & 63u, cr = c < 960u ? c - 512u - cg * 7u : 7u;
    const unsigned G = isx ? ((c >> 2) & 7u) * 16u + (((c >> 2) + (c >> 7)) & 3u) * 4u + ((c >> 5) & 3u) : (cr < 4u ? h : h ^ 1u) * 64u + cg;
    const unsigned q = isx ? c & 3u : cr < 4u ? cr : (cr - 3u) & 3u;
    const unsigned cr_ = !isx ? (G >> 6) * 4u + ((unsigned)lane & 3u) : (lane & 3) == 0 ? G >> 4 : (((G >> 4) & 4u) ^ 4u) + ((G >> 2) & 3u);
    const unsigned cl_ = !isx ? 32u + (G & 63u) : (lane & 3) == 0 ? G & 15u : 16u + ((G >> 4) & 3u) * 4u + (G & 3u);
    f4 v[16];
    if (wv != 0) {
      call = next_call(ctr_raw);
      WG_BARRIER();
      for (int i = 0; i < wv && cv; ++i) __builtin_amdgcn_s_sleep(CDELAY);
    }
    if (cv) load_piece(v, px, pw, isx, G, q, lane);
    char* const cws = lane_ws(cr_, rank);
    if (wv == 0) {
      call = next_call(ctr_raw);
      WG_BARRIER();
    }
    if (tid == NTHREADS - 64) __builtin_amdgcn_raw_buffer_store_b32(call, rsMy, 0, CTR_OFF + b * 4u, AUX_SYS);
    if (cv) {
      u4* const tr = (u4*)&sm.xl[wv & 7][0][0];
      const unsigned code = conv_piece(v, isx, tr, lane);
      const unsigned pi = ((isx ? 0u : 128u) + G) * 4u + q;
#pragma unroll
      for (int f = 0; f < 8; ++f)
        __builtin_amdgcn_raw_buffer_store_b128(tr[66 * f + (lane >> 5) * 33 + (lane & 31)], rsMy, lane * 16, OWN_OFF + (call & 1u) * OWN_PAR + pi * PIECE + f * 1024, AUX_SYS);
      asm volatile("s_waitcnt vmcnt(0)" ::: "memory");
      if ((lane & 3) < (isx ? 2 : 4) && lane < (cr_ == rank ? 4 * NREP : 4))
        flag_st(cws + ((call & 1u) * FLAG_PAR + q + (cr_ == rank ? FL_OFF + (unsigned)(lane >> 2) * REP_STRIDE + (isx ? FX_REL + cl_ * 128u : wflag(G & 63u, 0u)) : AV_OFF + rank * 512u + cl_ * 4u)), tag_of(call) | code);
      STAMP(24 + wv);
    }
  }
  const unsigned dbase = (call & 1u) * PAR_STRIDE;
  const unsigned fbase = FL_OFF + (call & 1u) * FLAG_PAR + (cidx % NREP) * REP_STRIDE;
  const unsigned obase = OWN_OFF + (call & 1u) * OWN_PAR;
  if (EXP & 1) return;
#define LNN(x) ((x) > nt ? (x) - nt + NT : (x))
  const bool two = cidx >= 128u || (cidx & 1u) == 0u;
  const unsigned g2 = cidx < 128u ? cidx >> 1 : cidx - 128u, half = cidx >> 7, r2 = 4u * rank + 16u - NL + half;
  const unsigned lp = ((rg - 4u * rank) & 15u) - (16u - NL);
  const bool latecu = rg < 16u && lp < NL;
  const int vo2 = (lane >> 3) * 16384 + ((lane ^ (lane >> 3)) & 7) * 16;

  if (wv < 8) {
    const unsigned v = (unsigned)wv;
    if (latecu && v - 1u < 3u) return;
    const unsigned tag4 = tag_of(call) * 0x01010101u;
    const unsigned lb = lds_addr(&sm.xl[v][0][0]);
    const rsrc_t rsX = mk(myws - lb, WS_USED + lb);
    const unsigned rb = v == 0u ? obase + xgroup(rank, rg) * 4u * PIECE : dbase + v * SRC_STRIDE + rg * 4u * PIECE;
    const unsigned ab = (v == 0u ? obase + (128u + 64u * h) * 4u * PIECE : dbase + v * SRC_STRIDE + W_OFF) + cls * 4u * PIECE;
    const unsigned hq = half * 4u + (v & 3u);
#define VR(B, I) "v[" B "+4*(" I "):" B "+4*(" I ")+3]"
#define LD1(R, S, F) "buffer_load_dwordx4 " R ", v1, %[rs], " S " offen offset:1024*" F " sc1\n\t"
#define LDI(X, F) LD1(VR(X, F), "s[74+" F "/4]", "(" F "&3)")
#define LDN(X, F)
#define LOAD8(X) LDI(X, "0") LDI(X, "1") LDI(X, "2") LDI(X, "3") LDI(X, "4") LDI(X, "5") LDI(X, "6") LDI(X, "7")
#define RES(Q, S, R7) LD1(VR("100", "4*" Q), S, "0") LD1(VR("100", "4*" Q "+1"), S, "1") LD1(VR("100", "4*" Q "+2"), S, "2") LD1(VR("100", "4*" Q "+3"), S, "3") \
      "s_addk_i32 " S ", 0x1000\n\t" LD1(VR("36", "4*" Q), S, "0") LD1(VR("36", "4*" Q "+1"), S, "1") LD1(VR("36", "4*" Q "+2"), S, "2") LD1(R7, S, "3")
#define XW(I) "ds_write_b128 v1, " VR("36", I) " offset:1024*" I "\n\t"
#define XRD(Q, I) "ds_read_b128 " VR("20", I) ", v1 offset:4096*" Q "+1024*" I "\n\t"
#define POLL "s_load_dwordx4 s[64:67], %[ws], s70 glc\n\t"
#define ZACC "v_mov_b64 v[4:5], 0\n\tv_mov_b64 v[6:7], 0\n\tv_mov_b64 v[8:9], 0\n\tv_mov_b64 v[10:11], 0\n\t" \
      "v_mov_b64 v[12:13], 0\n\tv_mov_b64 v[14:15], 0\n\tv_mov_b64 v[16:17], 0\n\tv_mov_b64 v[18:19], 0\n\t"
#define PEEK "s_xor_b32 s80, s64, %[tag]\n\ts_quadmask_b32 s80, s80\n\ts_andn2_b32 s80, s68, s80\n\t"
#define PICK(N, NONE) PEEK "s_cbranch_scc0 " NONE "\n\t" \
      "s_ff1_i32_b32 s80, s80\n\ts_lshl_b32 s74, s80, 12\n\ts_add_u32 s74, s74, s69\n\ts_add_u32 s75, s74, 0x1000\n\t" \
      "s_and_b32 s81, s64, 0x0f0f0f0f\n\ts_add_u32 s81, s81, s94\n\ts_lshl2_add_u32 s82, s80, 0x7fffc\n\ts_bfe_u32 s81, s81, s82\n\t" \
      "s_lshl_b32 s81, s81, 8\n\ts_or_b32 s73, s72, s80\n\ts_or_b32 s73, s73, s81\n\t" \
      "s_bitset0_b32 s68, s80\n\ts_cmp_eq_u32 s68, 0\n\ts_cbranch_scc1 La" N "%=\nLb" N "%=:\n\t"
#define DBGST(S, SH, O)
#define ADV(N) "La" N "%=:\n\t" DBGST("s72", "16", "0x4f0000") "s_bitset1_b32 s73, 7\n\ts_mov_b32 s64, s65\n\ts_mov_b32 s65, s66\n\ts_mov_b32 s66, s67\n\ts_mov_b32 s67, 0\n\t" \
      "s_add_u32 s69, s69, s90\n\ts_add_u32 s70, s70, 4\n\ts_add_u32 s72, s72, 0x80000\n\ts_sub_u32 s71, s71, 1\n\t" \
      "s_cmp_lg_u32 s71, 0\n\ts_cselect_b32 s68, s91, 0\n\ts_branch Lb" N "%=\n"
#define MF(W, XA, X, F, LD) "s_waitcnt vmcnt(" W ")\n\tv_mfma_f32_32x32x16_f16 v[4:19], " XA ", " VR(X, F) ", v[4:19]\n\t" LD(X, F)
#define WL(F) "15"
#define WM(F) "15-" F
#define WS(F) "7-" F
#define BODY(X, Q, W, LD, MID, R3, X7, END) XRD(Q, "0") XRD(Q, "1") XRD(Q, "2") R3 "ds_read_b32 v3, v2 offset:4096\n\t" \
      MF(W("0"), VR("100", "4*" Q), X, "0", LD) MF(W("1"), VR("100", "4*" Q "+1"), X, "1", LD) MF(W("2"), VR("100", "4*" Q "+2"), X, "2", LD) MF(W("3"), VR("100", "4*" Q "+3"), X, "3", LD) \
      "s_waitcnt lgkmcnt(0)\n\t" MID \
      MF(W("4"), "v[20:23]", X, "4", LD) MF(W("5"), "v[24:27]", X, "5", LD) MF(W("6"), "v[28:31]", X, "6", LD) MF(W("7"), X7, X, "7", LD) \
      "s_branch " END "\n"
#define TREE(N, X, D, W, LD, MID, END) "s_bitcmp1_b32 " D ", 2\n\ts_cbranch_scc1 Lh" N "%=\n\ts_bitcmp1_b32 " D ", 1\n\ts_cbranch_scc1 Li" N "%=\n\t" \
      BODY(X, "0", W, LD, MID, XRD("0", "3"), "v[32:35]", END) "Li" N "%=:\n\t" BODY(X, "1", W, LD, MID, XRD("1", "3"), "v[32:35]", END) \
      "Lh" N "%=:\n\ts_bitcmp1_b32 " D ", 1\n\ts_cbranch_scc1 Lj" N "%=\n\t" \
      BODY(X, "2", W, LD, MID, XRD("2", "3"), "v[32:35]", END) "Lj" N "%=:\n\t" BODY(X, "3", W, LD, MID, "", "v[164:167]", END)
#define TAIL(D, NEW, NEXT) "s_add_u32 s79, s79, 1\n\ts_bitcmp1_b32 " D ", 7\n\ts_mov_b32 " D ", " NEW "\n\ts_cbranch_scc0 " NEXT "\n\ts_call_b64 s[84:85], Lflush%=\n\ts_branch " NEXT "\n"
#define STEP(X, Y, DX, DY) "Ls" X "%=:\n\t" \
      "s_lshr_b32 s80, " DX ", 8\n\ts_cmp_eq_u32 s79, 0\n\ts_cselect_b32 s78, s80, s78\n\ts_cmp_lg_u32 s78, s80\n\ts_cbranch_scc1 Lk" X "%=\n" \
      "Lp" X "%=:\n\t" PICK(X, "Ln" X "%=") \
      "s_cmp_eq_u32 " DY ", 0\n\ts_cbranch_scc1 Ly" X "%=\n\t" \
      TREE("l" X, X, DX, WL, LDI, "", "Lt" X "%=") \
      "Ly" X "%=:\n\t" LOAD8(Y) "s_mov_b32 " DY ", s73\n\ts_branch Lp" X "%=\n" \
      "Lk" X "%=:\n\ts_call_b64 s[84:85], Lflush%=\n\ts_branch Ls" X "%=\n" \
      ADV(X) \
      "Ln" X "%=:\n\ts_mov_b32 s73, 0\n\ts_cmp_lg_u32 " DY ", 0\n\ts_cbranch_scc1 Lm" X "%=\n\t" \
      TREE("s" X, X, DX, WS, LDN, POLL, "Lu" X "%=") \
      "Lm" X "%=:\n\t" TREE("m" X, X, DX, WM, LDN, "", "Lt" X "%=") \
      "Lu" X "%=:\n\t" TAIL(DX, "0", "Lidle%=") \
      "Lt" X "%=:\n\t" TAIL(DX, "s73", "Ls" Y "%=")
#define LDX(I) "v_ldexp_f32 v[4+" I "], v[4+" I "], s82\n\t"
#define LDX4(I) LDX("4*" I) LDX("4*" I "+1") LDX("4*" I "+2") LDX("4*" I "+3")
#define FW1(I) LDX4(I) "ds_write2st64_b32 v2, v[4+4*" I "], v[5+4*" I "] offset0:4*" I " offset1:4*" I "+1\n\t" \
      "ds_write2st64_b32 v2, v[6+4*" I "], v[7+4*" I "] offset0:4*" I "+2 offset1:4*" I "+3\n\t"
#define FW2(I) LDX4(I) "v_xor_b32 v20, 32*" I ", v2\n\tds_write_b128 v20, " VR("4", I) "\n\t"
#define ABORT "v_mov_b32 v20, %[ex]\n\tv_mov_b32 v21, 1\n\tds_write_b32 v20, v21\n\t"
#define STR_(X) #X
#define STR(X) STR_(X)
#define RWAIT(L) "s_xor_b32 s81, s94, %[tag]\n\ts_and_b32 s81, s81, 0xf0f0f0f0\n\ts_cbranch_scc0 Lgo%=\n\t" \
      "s_sleep 4\n\ts_add_u32 s86, s86, 1\n\ts_cmp_le_u32 s86, 0x200000\n\ts_cbranch_scc1 " L "\n\t" ABORT
#define DBGI(X)
      asm volatile(
        "v_mbcnt_lo_u32_b32 v0, -1, 0\n\tv_mbcnt_hi_u32_b32 v0, -1, v0\n\tv_lshl_add_u32 v1, v0, 4, %[lb]\n\tv_lshl_add_u32 v2, v0, 2, %[sl]\n\t"
        "s_mov_b32 s70, %[fo]\n\ts_mov_b32 s80, %[rb]\n\ts_mov_b32 s69, %[ab]\n\ts_mov_b32 s71, %[tl]\n\ts_mov_b32 s72, 0x80000\n\t"
        "s_mov_b32 s90, %[ts1]\n\ts_movk_i32 s91, 0xaa\n\ts_movk_i32 s92, 0x2000\n\ts_mov_b32 s93, 0\n\ts_mov_b64 s[86:87], 0\n\t" DBGI("s_mov_b32 s95, %[dbg]\n")
        "Lq%=:\n\ts_load_dword s94, %[ws], %[fx] glc\n\ts_waitcnt lgkmcnt(0)\n\t" RWAIT("Lq%=")
        "Lgo%=:\n\ts_and_b32 s94, s94, 0x0f0f0f0f\n\ts_add_u32 s81, s80, s92\n\ts_add_u32 s82, s81, s92\n\ts_add_u32 s83, s82, s92\n\t"
        "s_cmp_lg_u32 s93, 0\n\ts_cbranch_scc1 Lr%=\n\t"
        RES("0", "s80", VR("36", "3")) "Lr%=:\n\t" RES("1", "s81", VR("36", "7")) RES("2", "s82", VR("36", "11")) RES("3", "s83", "v[164:167]")
        POLL
        "s_mov_b32 s68, s91\n\ts_mov_b64 s[76:77], 0\n\ts_mov_b64 s[78:79], 0\n\t" ZACC
        "s_waitcnt vmcnt(0)\n\t"
        XW("0") XW("1") XW("2") XW("3") XW("4") XW("5") XW("6") XW("7") XW("8") XW("9") XW("10") XW("11") XW("12") XW("13") XW("14")
        DBGST("s92", "17", "-832")
        "Lidle%=:\n\ts_waitcnt lgkmcnt(0)\n"
        "Lc%=:\n\t" PICK("0", "Lnone%=") LOAD8("36") "s_mov_b32 s76, s73\n\t"
        PEEK "s_cbranch_scc1 Ls36%=\n\ts_cmp_eq_u32 s68, 0\n\ts_cbranch_scc1 Ls36%=\n\t" POLL "s_waitcnt lgkmcnt(0)\n"
        STEP("36", "68", "s76", "s77")
        STEP("68", "36", "s77", "s76")
        ADV("0")
        "Lflush%=:\n\tv_readfirstlane_b32 s81, v3\n\ts_and_b32 s82, s78, 0xff\n\ts_lshl4_add_u32 s82, s82, -254\n\ts_cmp_lg_u32 s81, 0\n\ts_cbranch_scc1 Lw%=\n"
        "Lg%=:\n\ts_lshr_b32 s81, s78, 8\n\ts_add_u32 s81, s81, s79\n\ts_cmp_lg_u32 s93, 0\n\ts_cbranch_scc1 Lx%=\n\t"
        FW1("0") FW1("1") FW1("2") FW1("3") "s_branch Lz%=\n"
        "Lx%=:\n\t" FW2("0") FW2("1") FW2("2") FW2("3")
        "Lz%=:\n\tv_mov_b32 v3, s81\n\ts_mov_b64 exec, 1\n\tds_write_b32 v2, v3 offset:4096\n\ts_mov_b64 exec, -1\n\t"
        DBGST("s78", "8", "0") ZACC "s_mov_b32 s79, 0\n\ts_setpc_b64 s[84:85]\n"
        "Lw%=:\n\ts_sleep " STR(CSLEEP) "\n\tds_read_b32 v3, v2 offset:4096\n\ts_add_u32 s87, s87, 1\n\ts_waitcnt lgkmcnt(0)\n\tv_readfirstlane_b32 s81, v3\n\t"
        "s_cmp_eq_u32 s81, 0\n\ts_cbranch_scc1 Lg%=\n\ts_cmp_lt_u32 s87, 0x2000000\n\ts_cbranch_scc1 Lw%=\n\t" ABORT "s_branch Lg%=\n"
        "Lnone%=:\n\ts_cmp_eq_u32 s68, 0\n\ts_cbranch_scc1 Lend%=\n\ts_add_u32 s86, s86, 1\n\ts_cmp_gt_u32 s86, 0x200000\n\ts_cbranch_scc1 Lo%=\n\t"
        "s_sleep " STR(PSLEEP) "\n\t" POLL "s_branch Lidle%=\n"
        "Lo%=:\n\t" ABORT "s_mov_b32 s64, %[tag]\n\ts_branch Lc%=\n"
        "Lend%=:\n\ts_cmp_eq_u32 s93, %[rb2]\n\ts_cbranch_scc1 Lfin%=\n\t"
        "s_mov_b32 s93, %[rb2]\n\ts_mov_b32 s80, %[rb2]\n\ts_mov_b32 s69, %[ab2]\n\ts_mov_b32 s70, %[fo2]\n\ts_mov_b32 s71, %[l3]\n\t"
        "s_mov_b32 s90, 0x8000\n\ts_movk_i32 s91, 0xa8\n\ts_mov_b32 s92, %[ss]\n\t" DBGI("s_mov_b32 s95, %[dbg2]\n\t")
        "v_lshrrev_b32 v2, 5, v0\n\tv_xor_b32 v2, v2, v0\n\tv_and_b32 v2, 7, v2\n\tv_lshl_add_u32 v2, v2, 4, %[sl]\n\t"
        "v_and_b32 v3, 31, v0\n\tv_lshl_add_u32 v2, v3, 7, v2\n"
        "Lv%=:\n\ts_load_dword s81, %[ws], %[fw] offset:128 glc\n\ts_load_dword s82, %[ws], %[fw] offset:256 glc\n\ts_load_dword s83, %[ws], %[fw] offset:384 glc\n\t"
        "s_and_b32 s94, %[tag], 0xff\n\ts_waitcnt lgkmcnt(0)\n\ts_bfe_u32 s81, s81, %[sh]\n\ts_bfe_u32 s82, s82, %[sh]\n\ts_bfe_u32 s83, s83, %[sh]\n\t"
        "s_lshl_b32 s81, s81, 8\n\ts_lshl_b32 s83, s83, 8\n\ts_or_b32 s94, s94, s81\n\ts_or_b32 s82, s82, s83\n\ts_pack_ll_b32_b16 s94, s94, s82\n\t" RWAIT("Lv%=")
        "s_branch Lgo%=\n"
        "Lfin%=:\n\ts_endpgm"
        :
        : [rs] "s"(rsX), [ws] "s"(myws), [tag] "s"(tag4), [ex] "s"(lds_addr(&sm.abort_)), [lb] "s"(lb), [sl] "s"(lds_addr(&sm.slot[v])),
          [fx] "s"(fbase + FX_REL + rg * 128u + v * 4u), [rb] "s"(rb), [fo] "s"(fbase + (cls * 8u + v) * 128u), [ab] "s"(ab - 4096u), [tl] "s"(nt),
          [fw] "s"(fbase + wflag(g2, 0u)), [sh] "s"(0x80000u + 8u * (v & 3u)), [rb2] "s"(v >= 4u && two ? dbase + W_OFF + (g2 * 4u + (v & 3u)) * PIECE : 0u),
          [fo2] "s"(fbase + FZ_REL + hq * 128u), [ab2] "s"(dbase + hq * 4u * L3 * PIECE - 4096u),
          [ts1] "n"(NCLS * 4u * PIECE), [l3] "n"(L3), [ss] "n"(SRC_STRIDE)
        : "memory", "scc", "v167",
          "s64", "s65", "s66", "s67", "s68", "s69", "s70", "s71", "s72", "s73", "s74", "s75", "s76", "s77", "s78", "s79",
          "s80", "s81", "s82", "s83", "s84", "s85", "s86", "s87", "s88", "s89", "s90", "s91", "s92", "s93", "s94", "s95");
  } else {
    const bool own = rg < 16u;
    const unsigned o = own ? rank : ((4u * (h ^ 1u) + ((rg >> 2) & 3u)) & 7u);
    const unsigned row0 = own ? 32u * rg : 128u * (rank & 3u) + 32u * (rg & 3u);
    const rsrc_t rsO = mk(a.outd[(o - rank - 1u) & 7u], OUT_BYTES);
    const rsrc_t rsD = mk(a.wsd[(o - rank - 1u) & 7u], WS_USED);
    const int lr = (lane >> 4) + ((lane >> 3) & 1) * 4;
    unsigned* const sp = lane < 8 ? &sm.slot[lane].st : &sm.abort_;
    const unsigned nl = (unsigned)nt + (two ? L3 : 0u);
    unsigned spins = 0;
    for (unsigned ts = (unsigned)wv - 7u; ts <= nl; ts += 2u) {
      f4 s[4];
#pragma unroll
      for (int i = 0; i < 4; ++i) s[i] = (f4){0.f, 0.f, 0.f, 0.f};
      const unsigned so2 = ((r2 + 2u * (ts - 1u - (unsigned)nt)) & 15u) * 32u * 16384u + (2048u * h + 32u * g2) * 4u;
      if (ts > (unsigned)nt) {
        while (poll1(myws, PRDY_OFF + ((2u * (ts - 1u - (unsigned)nt) + half) * 64u + g2) * 4u) != call) {
          if ((++spins & 63u) == 0u && now() - t0 > 3 * DEADLINE) { lds_st(&sm.abort_, 1u); break; }
          __builtin_amdgcn_s_sleep(4);
        }
#pragma unroll
        for (int i = 0; i < 4; ++i) s[i] = __builtin_bit_cast(f4, __builtin_amdgcn_raw_buffer_load_b128(mk(a.outd[7], OUT_BYTES), vo2, so2 + (unsigned)i * 8u * 16384u, AUX_LD));
      }
      unsigned need = ts > (unsigned)nt ? 12u : latecu ? 20u : 32u, sv;
      do {
        unsigned m;
        __builtin_amdgcn_s_setprio(0);
        while (true) {
          sv = __hip_atomic_load(sp, __ATOMIC_RELAXED, __HIP_MEMORY_SCOPE_WORKGROUP);
          m = (unsigned)__builtin_amdgcn_ballot_w64((sv >> 3) == ts);
          if (m != 0u) break;
          if ((++spins & 1023u) == 0u && now() - t0 > 3 * DEADLINE) {
            lds_st(&sm.abort_, 1u);
            if (lane == 0) __builtin_amdgcn_raw_buffer_store_b32(0x7fc00000u, mk(a.outd[7], OUT_BYTES), 0, 0, AUX_SYS);
            __builtin_amdgcn_endpgm();
          }
          __builtin_amdgcn_s_sleep(CSLEEP);
        }
        __builtin_amdgcn_s_setprio(EPRIO);
        asm volatile("" ::: "memory");
        unsigned mm = m;
        do {
          f4 p[2][4];
          const int v1 = __builtin_ctz(mm);
          mm &= mm - 1u;
#pragma unroll
          for (int i = 0; i < 4; ++i) p[0][i] = ((const f4*)sm.slot[v1].d)[i * 64 + lane];
          need -= __builtin_amdgcn_readlane(sv, v1) & 7u;
          if (mm != 0u) {
            const int v2 = __builtin_ctz(mm);
            mm &= mm - 1u;
#pragma unroll
            for (int i = 0; i < 4; ++i) p[1][i] = ((const f4*)sm.slot[v2].d)[i * 64 + lane];
            need -= __builtin_amdgcn_readlane(sv, v2) & 7u;
#pragma unroll
            for (int i = 0; i < 4; ++i) p[0][i] += p[1][i];
          }
#pragma unroll
          for (int i = 0; i < 4; ++i) s[i] += p[0][i];
        } while (mm != 0u);
        asm volatile("" ::: "memory");
        if ((sv >> 3) == ts) lds_st(sp, 0u);
      } while (need != 0u);
      if (__builtin_amdgcn_readlane(sv, 8) != 0u) {
        const float nan = __builtin_nanf("");
#pragma unroll
        for (int i = 0; i < 4; ++i) s[i] = (f4){nan, nan, nan, nan};
      }
      const unsigned g = (ts - 1u) * NCLS + cls;
      unsigned yfo = WS_USED;
      if (ts > (unsigned)nt) {
#pragma unroll
        for (int i = 0; i < 4; ++i) __builtin_amdgcn_raw_buffer_store_b128(__builtin_bit_cast(u4, s[i]), mk(a.outd[7], OUT_BYTES), vo2, so2 + (unsigned)i * 8u * 16384u, AUX_SYS);
      } else if (!own && g < NYI / 2u) {
        const unsigned yi = (o & 3u) * 128u + (g >> 1) * 4u + (rg & 3u);
#pragma unroll
        for (int i = 0; i < 4; ++i) __builtin_amdgcn_raw_buffer_store_b128(__builtin_bit_cast(u4, s[i]), rsMy, lr * 256 + (lane & 7) * 16, YST_OFF + yi * PIECE + (g & 1u) * 128u + (unsigned)i * 2048u, AUX_SYS);
        if (!(EXP & 8)) yfo = YRDY_OFF + ((rank & 3u) * 128u + (yi & 127u)) * 8u + (g & 1u) * 4u;
      } else {
        const unsigned so = row0 * 16384u + (2048u * h + 32u * g) * 4u;
#pragma unroll
        for (int i = 0; i < 4; ++i) __builtin_amdgcn_raw_buffer_store_b128(__builtin_bit_cast(u4, s[i]), rsO, lr * 16384 + (lane & 7) * 16, so + (unsigned)i * 8u * 16384u, AUX_SYS);
        if (latecu) yfo = PRDY_OFF + (lp * 64u + g) * 4u;
        if (!own) yfo = YRDY_OFF + ((rank & 3u) * 128u + (g >> 1) * 4u + (rg & 3u)) * 8u + (g & 1u) * 4u;
      }
      STAMP(95 + LNN((int)ts));
      if (yfo != WS_USED) {
        asm volatile("s_waitcnt vmcnt(0)" ::: "memory");
        if (lane == 0) __builtin_amdgcn_raw_buffer_store_b32(call, rsD, 0, yfo, AUX_SYS);
      }
    }
  }
}

extern "C" void kernel_launch(int rank, int nranks, ncclComm_t comm,
                              hipStream_t stream,
                              void* const* d_in, const int* in_sizes, int n_in,
                              void* d_out, int out_size,
                              const int* in_dtypes, int out_dtype,
                              void* const* d_out_all,
                              void* const* d_ws_all, size_t ws_bytes) {
  (void)comm; (void)in_dtypes; (void)out_dtype; (void)d_out;
  if (nranks != NR || rank < 0 || rank >= NR || n_in < 2 || ws_bytes < WS_USED ||
      in_sizes[0] != 4096 * 512 || in_sizes[1] != 512 * 4096 || out_size != 512 * 4096) {
    fprintf(stderr, "kernel_launch: unsupported configuration\n");
    return;
  }
  Args a;
  a.x = (const float*)d_in[0];
  a.w = (const float*)d_in[1];
  for (int p = 0; p < NR; ++p) { a.wsd[p] = (char*)d_ws_all[(rank + 1 + p) & 7]; a.outd[p] = (float*)d_out_all[(rank + 1 + p) & 7]; }
  a.myws = (char*)d_ws_all[rank];
  a.rank = (unsigned)rank;
  a.pad_ = 0u;
  hipLaunchKernelGGL(fused_gemm_kernel, dim3(256), dim3(NTHREADS), 0, stream, a);
  HIP_CHECK(hipGetLastError());
}
```

```cpp
#include <hip/hip_runtime.h>
#include <rccl/rccl.h>
#include <cstdio>

#define HIP_CHECK(expr) do { hipError_t _e = (expr); if (_e != hipSuccess) { \
    fprintf(stderr, "HIP error %s at %s:%d\n", hipGetErrorString(_e), __FILE__, __LINE__); \
    return; } } while (0)

#define WIN 6
#define PW 8
#define EXP 0
#define PSLEEP 1
#define NCLS 6
#define NW 2
#define NF 3
#define SPL 1
#define NREP 4
#define RR 9u
#define L3 4u
#define LROT 5u
#define OOO 1
#define D0 0
#define D0FROM 128u
#define SHADOW 1
#define EPRIO 2
#define CSLEEP 1
#define FDELAY 64
#define YSLEEP 1
#define FSLEEP 8
#define CDELAY 96

typedef float f4 __attribute__((ext_vector_type(4)));
typedef float f16v __attribute__((ext_vector_type(16)));
typedef unsigned u4 __attribute__((ext_vector_type(4)));
typedef _Float16 h8 __attribute__((ext_vector_type(8)));
typedef __amdgpu_buffer_rsrc_t rsrc_t;

constexpr int NR = 8;
constexpr unsigned PIECE = 8192;
constexpr unsigned SRC_STRIDE = 384u * PIECE;
constexpr unsigned W_OFF = 128u * PIECE;
constexpr unsigned PAR_STRIDE = 24u << 20;
constexpr unsigned FLAG_OFF = 48u << 20;
constexpr unsigned FL_OFF = FLAG_OFF + (704u << 10);
constexpr unsigned FX_REL = 6144;
constexpr unsigned FZ_REL = 10240;
constexpr unsigned FE_REL = 12288;
constexpr unsigned REP_STRIDE = 32768u + 4352u;
constexpr unsigned FLAG_PAR = 16384;
constexpr unsigned CTR_OFF = FLAG_OFF + 36864;
constexpr unsigned TMO_OFF = FLAG_OFF + 40960;
constexpr unsigned QUEUE_OFF = FLAG_OFF + 45056;
constexpr unsigned YRDY_OFF = FLAG_OFF;
constexpr unsigned CTR2_OFF = FLAG_OFF + 53248;
constexpr unsigned HF_OFF = FLAG_OFF + 57344;
constexpr unsigned PRDY_OFF = FLAG_OFF + 61440;
constexpr unsigned AV_OFF = FLAG_OFF + (640u << 10);
constexpr unsigned DBG_OFF = FLAG_OFF + 65536;
constexpr unsigned OWN_OFF = FLAG_OFF + (6u << 20);
constexpr unsigned OWN_PAR = 8u << 20;
constexpr unsigned YST_OFF = OWN_OFF + 2u * OWN_PAR;
constexpr unsigned WS_USED = YST_OFF + (8u << 20);
constexpr unsigned OUT_BYTES = 512u * 4096u * 4u;
constexpr unsigned IN_BYTES = 512u * 4096u * 4u;
constexpr unsigned long long DEADLINE = 50000000ull;
constexpr int AUX_SYS = 17;
#define AUX_LD 16
constexpr int XR = 17;
constexpr int NTHREADS = 640;
constexpr int NT = (64 + NCLS - 1) / NCLS;
constexpr unsigned RG = RR * NCLS < 64u ? RR * NCLS : 64u;
constexpr unsigned NL = 2u * L3;
constexpr unsigned LH = L3 / 2u;
static_assert(L3 % 2u == 0u, "");
constexpr unsigned CPX = 4u * NCLS;

struct Args {
  const float* x;
  const float* w;
  char* myws;
  unsigned rank;
  unsigned pad_;
  char* wsd[NR];
  float* outd[NR];
};

struct __attribute__((aligned(128))) Slot {
  float d[32 * 32];
  unsigned st, pad_[31];
};
struct __attribute__((aligned(128))) Smem {
  u4 xl[8][32 - XR][64];
  Slot slot[8];
  unsigned st[8];
  unsigned abort_;
  unsigned tick, pad_[2];
  u4 fifo[8];
  u4 fifo2[8];
};

__device__ __forceinline__ rsrc_t mk(const void* p, unsigned bytes) {
  return __builtin_amdgcn_make_buffer_rsrc((void*)p, (short)0, (int)bytes, 0x00020000);
}
__device__ __forceinline__ unsigned lds_ld(const unsigned* p) {
  return __builtin_amdgcn_readfirstlane(__hip_atomic_load(p, __ATOMIC_RELAXED, __HIP_MEMORY_SCOPE_WORKGROUP));
}
__device__ __forceinline__ void lds_st(unsigned* p, unsigned v) {
  __hip_atomic_store(p, v, __ATOMIC_RELAXED, __HIP_MEMORY_SCOPE_WORKGROUP);
}
__device__ __forceinline__ unsigned lds_addr(const void* p) { return (unsigned)(size_t)(const __attribute__((address_space(3))) void*)p; }
__device__ __forceinline__ unsigned poll1(const char* p, unsigned off) {
  unsigned v;
  asm volatile("s_load_dword %0, %1, %2 glc\n\ts_waitcnt lgkmcnt(0)" : "=s"(v) : "s"(p), "s"(off) : "memory");
  return v;
}
__device__ __forceinline__ u4 poll4(const char* p, unsigned off) {
  u4 v;
  asm volatile("s_load_dwordx4 %0, %1, %2 glc\n\ts_waitcnt lgkmcnt(0)" : "=s"(v) : "s"(p), "s"(off) : "memory");
  return v;
}
__device__ __forceinline__ unsigned long long now() { return __builtin_amdgcn_s_memrealtime(); }

#define STAMP(slot) do {} while (0)

__device__ __forceinline__ unsigned wave_code(unsigned m) {
  const unsigned c = ((m >> 23) + 1u) >> 4;
  unsigned r = 0;
#pragma unroll
  for (unsigned b = 8; b; b >>= 1)
    if (__builtin_amdgcn_ballot_w64(c >= (r | b)) != 0ull) r |= b;
  return r;
}
__device__ __forceinline__ unsigned tag_of(unsigned call) { return (call % 15u + 1u) << 4; }
__device__ __forceinline__ unsigned wflag(unsigned g, unsigned sx) { const unsigned t = (g * 43u) >> 8; return ((g - t * NCLS) * 8u + sx) * 128u + t * 4u; }

#define WG_BARRIER() asm volatile("s_waitcnt lgkmcnt(0)\n\ts_barrier" ::: "memory")

__device__ __forceinline__ unsigned next_call(unsigned raw) {
  unsigned call = __builtin_amdgcn_readfirstlane(raw) + 1u;
  if (call > 0xfffffeu) call = 1u;
  asm volatile("" : "+s"(call) :: "memory");
  return call;
}
__device__ __forceinline__ char* lane_ws(unsigned r, unsigned rank) {
  typedef char* const __attribute__((address_space(4))) * kp_t;
  return ((kp_t)((const __attribute__((address_space(4))) char*)__builtin_amdgcn_kernarg_segment_ptr() + offsetof(Args, wsd)))[(r - rank - 1u) & 7u];
}
__device__ __forceinline__ void flag_st(char* p, unsigned v) {
  asm volatile("global_store_byte %0, %1, off sc0 sc1" :: "v"(p), "v"(v) : "memory");
}

__device__ __forceinline__ void load_piece(f4 (&v)[16], const float* px, const float* pw, bool isx, unsigned G, unsigned q, int lane) {
  const rsrc_t rs = mk(isx ? px : pw, IN_BYTES);
  const int vo = isx ? (lane >> 4) * 2048 + (lane & 15) * 16 : (lane >> 3) * 4 * 16384 + (lane & 7) * 16;
  const unsigned so = isx ? G * 65536u + q * 512u : q * 128u * 16384u + G * 128u;
  const unsigned s0 = isx ? 256u : 16384u, s1 = isx ? 8192u : 32768u, s2 = isx ? 16384u : 64u * 16384u, s3 = isx ? 32768u : 32u * 16384u;
#pragma unroll
  for (int f = 0; f < 16; ++f)
    v[f] = __builtin_bit_cast(f4, __builtin_amdgcn_raw_buffer_load_b128(rs, vo, so + (f & 1) * s0 + ((f >> 1) & 1) * s1 + ((f >> 2) & 1) * s2 + (f >> 3) * s3, 0));
}
__device__ __forceinline__ unsigned conv_piece(f4 (&v)[16], bool isx, u4* tr, int lane) {
  float m = 0.f, m2 = 0.f;
#pragma unroll
  for (int f = 0; f < 16; ++f) {
    asm("v_max3_f32 %0, %1, |%2|, |%3|" : "=v"(m) : "v"(m), "v"(v[f][0]), "v"(v[f][1]));
    asm("v_max3_f32 %0, %1, |%2|, |%3|" : "=v"(m2) : "v"(m2), "v"(v[f][2]), "v"(v[f][3]));
  }
  const unsigned code = wave_code(max(__builtin_bit_cast(unsigned, m), __builtin_bit_cast(unsigned, m2)));
  const float scl = __builtin_bit_cast(float, (127u + 15u + 112u - code * 16u) << 23);
#pragma unroll
  for (int f = 0; f < 16; ++f) v[f] *= scl;
  if (isx) {
#pragma unroll
    for (int f = 0; f < 8; ++f) {
      h8 r;
#pragma unroll
      for (int j = 0; j < 8; ++j) r[j] = (_Float16)v[2 * f + (j >> 2)][j & 3];
      tr[(lane & 15) * 33 + 4 * f + (lane >> 4)] = __builtin_bit_cast(u4, r);
    }
  } else {
#pragma unroll
    for (int f = 0; f < 8; ++f) {
      h8 r;
#pragma unroll
      for (int j = 0; j < 8; ++j) r[j] = (_Float16)v[(f >> 2) * 8 + j][f & 3];
      tr[((f >> 2) * 8 + (lane >> 3)) * 33 + 4 * (lane & 7) + (f & 3)] = __builtin_bit_cast(u4, r);
    }
  }
  asm volatile("" ::: "memory");
  return code;
}
__device__ __forceinline__ unsigned xgroup(unsigned d, unsigned rg) {
  return rg < 16u ? 16u * d + rg : 16u * (((d & 4u) ^ 4u) + ((rg >> 2) & 3u)) + 4u * (d & 3u) + (rg & 3u);
}

__device__ __forceinline__ unsigned xitem(unsigned d, unsigned j) {
  return ((j & 4u) ? 16u + 4u * ((d - (j >> 5)) & 3u) : 4u * ((d + (j >> 5)) & 3u)) + ((j >> 3) & 3u);
}

__global__ void __launch_bounds__(NTHREADS) fused_gemm_kernel(Args a) {
  __shared__ Smem sm;
#define STAMP_ENTRY() do {} while (0)
  const int tid = threadIdx.x;
  const int lane = tid & 63;
  const int wv = __builtin_amdgcn_readfirstlane(tid >> 6);
  const unsigned rank = a.rank & 7u;
  const unsigned h = rank >> 2;
  const unsigned b = blockIdx.x;
  const unsigned xc = b & 7u, jb = (b >> 3) & 31u;
  char* const myws = a.myws;
  const float* const px = a.x;
  const float* const pw = a.w;
  char* const pd = a.wsd[jb & 7u];
  asm volatile("" :: "s"(px), "s"(pw), "s"(myws), "s"(rank), "s"(pd));
  const rsrc_t rsMy = mk(myws, WS_USED);
  unsigned* const tmo = (unsigned*)(myws + TMO_OFF);
#define TIMED_OUT() (now() - t0 > DEADLINE || __hip_atomic_load(tmo, __ATOMIC_RELAXED, __HIP_MEMORY_SCOPE_AGENT) == call)
#define SET_TIMEOUT() __hip_atomic_store(tmo, call, __ATOMIC_RELAXED, __HIP_MEMORY_SCOPE_AGENT)

  if (__builtin_expect(jb < 7u && wv < NW, 1)) {
    static_assert(NW >= 2 && NW + 2 + NF <= 10, "hop pieces are pushed");
    const unsigned p = (rank + 1u + jb) & 7u;
    const unsigned nhop = ((p ^ rank) & 4u) * 4u;
    const unsigned ctr_raw = __builtin_amdgcn_raw_buffer_load_b32(rsMy, 0, CTR2_OFF + b * 64u + (unsigned)wv * 4u, AUX_SYS);
    const unsigned i = (unsigned)wv * 8u + xc;
    const bool isx = i >= nhop;
    static_assert(NW <= 4, "");
    const unsigned idx = isx ? xitem(p, i - nhop + (jb == 3u ? i & 12u : 0u)) : i * 4u + ((jb + 1u) & 3u), q = isx ? i & 3u : 0u;
    f4 v[16];
    if (wv != 0) asm volatile("s_barrier" ::: "memory");
    load_piece(v, px, pw, isx, isx ? xgroup(p, idx) : (p & 4u) * 16u + idx, q, lane);
    __builtin_amdgcn_sched_barrier(0);
    asm volatile("" ::: "memory");
    if (wv == 0) asm volatile("s_barrier" ::: "memory");
    STAMP_ENTRY();
    const unsigned call = next_call(ctr_raw);
    const unsigned code = conv_piece(v, isx, (u4*)&sm.xl[wv][0][0], lane);
    const u4 e = {isx ? 1u : HF_OFF + (call & 1u) * 1024u + (p & 3u) * 256u + idx,
                  (call & 1u) * PAR_STRIDE + (rank ^ p) * SRC_STRIDE + (isx ? 0u : W_OFF) + (idx * 4u + q) * PIECE,
                  FL_OFF + (call & 1u) * FLAG_PAR + (isx ? FX_REL + idx * 128u + (rank ^ p) * 4u : wflag(idx, rank ^ p)) + q, tag_of(call) | code};
    if (lane == 0) sm.fifo[wv] = e;
    asm volatile("" ::: "memory");
    STAMP(16 + 4 * wv);
    if (lane == 0) __builtin_amdgcn_raw_buffer_store_b32(call, rsMy, 0, CTR2_OFF + b * 64u + (unsigned)wv * 4u, AUX_SYS);
    return;
  }
  if (jb < 7u && wv >= NW + 2 && wv < NW + 2 + NF) {
    const unsigned p = (rank + 1u + jb) & 7u;
    const bool cross = ((p ^ rank) & 4u) != 0u;
    const unsigned nhop = cross ? 16u : 0u;
    const unsigned fi = (unsigned)wv - (NW + 2);
    asm volatile("s_barrier" ::: "memory");
    const unsigned ctr_raw = __builtin_amdgcn_raw_buffer_load_b32(rsMy, 0, CTR2_OFF + b * 64u + (unsigned)wv * 4u, AUX_SYS);
    STAMP_ENTRY();
    const unsigned long long t0 = now();
    const unsigned call = next_call(ctr_raw);
    if (lane == 0) __builtin_amdgcn_raw_buffer_store_b32(call, rsMy, 0, CTR2_OFF + b * 64u + (unsigned)wv * 4u, AUX_SYS);
    const unsigned per = cross ? 3u : 5u;
    const unsigned nin = 128u + RG * per + (64u - RG) * 4u;
    const unsigned nx = cross ? 128u : 128u - 4u * NL, nl = cross ? nin : nin - 4u * NL;
    const unsigned nitems = nin + ((cross && !(EXP & 1)) ? 128u : 0u);
    const unsigned dbase = (call & 1u) * PAR_STRIDE;
    const unsigned frel = (call & 1u) * FLAG_PAR;
    const unsigned obase = OWN_OFF + (call & 1u) * OWN_PAR;
    const unsigned avb = AV_OFF + frel + p * 512u, hfb = HF_OFF + (call & 1u) * 1024u + (p & 3u) * 256u;
    const unsigned tag = tag_of(call);
    unsigned posted = 0, spins = 0;
    unsigned i = (NW + fi) * 8u + xc;
    __builtin_amdgcn_s_sleep(FDELAY);
    while (i < nitems) {
      unsigned so, dof, fo = 0, fv = 0, isy = 0;
      bool bad = false;
      if (i >= nin) {
        const unsigned j = i - nin;
        so = YST_OFF + ((call & 1u) * 512u + (rank & 3u) * 128u + j) * PIECE;
        dof = (128u * (p & 3u) + 32u * (j & 3u)) * 16384u + (512u * (p & 4u) + 64u * (j >> 2)) * 4u;
        isy = 32u;
        while (true) {
          const u4 yv = poll4(myws, YRDY_OFF + ((call & 1u) * 512u + (p & 3u) * 128u + j) * 16u);
          if (yv[0] == call && yv[1] == call && yv[2] == call && yv[3] == call) break;
          if ((++spins & 63u) == 0u && TIMED_OUT()) { SET_TIMEOUT(); bad = true; break; }
          __builtin_amdgcn_s_sleep(YSLEEP);
          asm volatile("" ::: "memory");
        }
        if (bad && lane == 0) __builtin_amdgcn_raw_buffer_store_b32(0x7fc00000u, mk(a.outd[7], OUT_BYTES), 0, dof, AUX_SYS);
      } else {
        unsigned idx, q;
        const bool isx = i < nx || i >= nl;
        bool fwd = false;
        const unsigned lt = (i - nl + LROT * (p ^ rank)) & 15u;
        if (isx) {
          unsigned j = i - nhop;
          if (i >= nl) j = 8u * (16u - NL + ((i - nl) >> 4) * 4u + (lt & 3u)) + (lt >> 2);
          else if (!cross && j >= 8u * (16u - NL)) j = 2u * (j & ~3u) - 8u * (16u - NL) + 4u + (j & 3u);
          else if (cross) {
            const unsigned t = j - 32u * ((rank - p) & 3u);
            if ((int)t >= 0) j += t < 16u ? t & 12u : 16u;
          }
          idx = xitem(rank, j); q = j & 3u;
        } else {
          unsigned j = i - nx;
          const unsigned n1 = cross ? RG * 3u : RG;
          const bool rel = j < n1;
          if (!rel) j -= n1;
          const unsigned pe = rel ? 3u : 4u;
          const unsigned r = (j * (rel ? 3641u : 2731u)) >> 16, rem = j - r * (NCLS * pe);
          const unsigned g0 = r * NCLS + (!rel && cross ? RG : 0u);
          const unsigned ng = min((unsigned)NCLS, 64u - g0);
          const unsigned k = (rem * (ng == 6u ? 10923u : 16384u)) >> 16;
          fwd = rel && !cross;
          idx = fwd ? j : g0 + rem - k * ng;
          q = fwd ? 0u : k + 4u - pe;
          bad = cross && q == 0u && ((idx - rank + p) & 3u) == 0u;
        }
        const unsigned src = fwd ? 4u * (h ^ 1u) + ((p - idx) & 3u) : p;
        const unsigned po = dbase + (isx ? 0u : W_OFF) + (idx * 4u + q) * PIECE;
        dof = po + (src ^ rank) * SRC_STRIDE;
        fo = frel + (isx ? FX_REL + idx * 128u + (src ^ rank) * 4u : wflag(idx, src ^ rank)) + q;
        if (i >= nl) {
          const unsigned hq = (lt & 3u) * 4u + q, ij = ((i - nl) >> 4) * 4u + (p ^ rank);
          dof = dbase + (hq * 4u * LH + ij) * PIECE;
          fo = frel + FZ_REL + hq * 128u + ij;
        }
        so = fwd ? po + (src ^ p) * SRC_STRIDE : obase + ((isx ? xgroup(rank, idx) : 128u + 64u * h + idx) * 4u + q) * PIECE;
        const unsigned ao = fwd ? hfb + idx : avb + (isx ? idx : 32u + idx) * 4u + q;
        while (!bad) {
          fv = (poll1(myws, ao & ~3u) >> (8u * (ao & 3u))) & 255u;
          if ((fv & 0xf0u) == tag) break;
          if ((++spins & 63u) == 0u && TIMED_OUT()) { SET_TIMEOUT(); bad = true; break; }
          __builtin_amdgcn_s_sleep(FSLEEP);
          asm volatile("" ::: "memory");
        }
      }
      if (!bad) {
        unsigned tk = 0;
        if (lane == 0) tk = atomicAdd(&sm.tick, 1u);
        tk = __builtin_amdgcn_readfirstlane(tk);
        const u4 e = {((tk + 1u) << 24) | (fo << 6) | isy | fi, so, dof, fv};
        asm volatile("" ::: "memory");
        if (lane == 0) sm.fifo2[tk & 7u] = e;
        asm volatile("" ::: "memory");
        if (tk + NW < 60) STAMP(16 + 4 * (tk + NW));
        ++posted;
        while (lds_ld(&sm.st[fi]) != posted) __builtin_amdgcn_s_sleep(1);
        asm volatile("" ::: "memory");
      }
      i = 0;
      if (lane == 0) i = __hip_atomic_fetch_add((unsigned*)(myws + QUEUE_OFF + (call & 1u) * 1024u + jb * 128u), 1u, __ATOMIC_RELAXED, __HIP_MEMORY_SCOPE_SYSTEM);
      i = __builtin_amdgcn_readfirstlane(i) + (NW + NF) * 8u;
    }
    if (lane == 0) {
      const unsigned tk = atomicAdd(&sm.tick, 1u);
      sm.fifo2[tk & 7u] = (u4){((tk + 1u) << 24) | 16u, 0u, 0u, 0u};
    }
    return;
  }
  if (jb < 7u && wv == NW) {
    __builtin_amdgcn_s_waitcnt(0x0f70);
    if (lane < 8) { sm.fifo[lane] = (u4){0u, 0u, 0u, 0u}; sm.fifo2[lane] = (u4){0u, 0u, 0u, 0u}; sm.st[lane] = 0u; }
    if (lane == 0) sm.tick = 0u;
    WG_BARRIER();
    __builtin_amdgcn_s_setprio(3);
    STAMP_ENTRY();
    const rsrc_t rsD = mk(pd, WS_USED);
    const int lp = (lane >> 5) * 33 + (lane & 31);
    const unsigned p = (rank + 1u + jb) & 7u;
    char* const fws = lane_ws((p & 4u) | ((p + (unsigned)lane) & 3u), rank);
    unsigned pfo = TMO_OFF + 64u, pfv = 0;
    bool pfl = false;
    for (unsigned n = 0; n < NW; ++n) {
      u4 e;
      while (true) {
        asm volatile("" ::: "memory");
        e = sm.fifo[n];
        if (__builtin_amdgcn_readfirstlane(e[0]) != 0u) break;
      }
      asm volatile("" ::: "memory");
      const u4* const tr = (const u4*)&sm.xl[n][0][0];
      u4 o[8];
#pragma unroll
      for (int f = 0; f < 8; ++f) o[f] = tr[66 * f + lp];
      const unsigned so = __builtin_amdgcn_readfirstlane(e[1]);
#pragma unroll
      for (int f = 0; f < 8; ++f) {
        asm volatile("s_waitcnt vmcnt(%0)" :: "n"(((f + 8 - (WIN - 1)) % 8 != 0 && (f + 8 - (WIN - 1)) % 8 < WIN - 1) ? WIN : WIN - 1) : "memory");
        if (f == WIN - 1 && n != 0) { if (pfl) flag_st(fws + pfo, pfv); }
        __builtin_amdgcn_raw_buffer_store_b128(o[f], rsD, lane * 16, so + f * 1024, AUX_SYS);
        asm volatile("" ::: "memory");
      }
      pfo = (lane & 3) == 0 ? e[2] + (unsigned)(lane >> 2) * REP_STRIDE : e[0];
      pfv = e[3];
      pfl = (lane & 3) == 0 ? lane < 4 * NREP : (lane < 4 && e[0] != 1u);
    }
    asm volatile("s_waitcnt vmcnt(0)" ::: "memory");
    if (pfl) flag_st(fws + pfo, pfv);
    STAMP(12);
    return;
  }
  if (jb < 7u && wv == NW + 1) {
    __builtin_amdgcn_s_waitcnt(0x0f70);
    asm volatile("s_barrier" ::: "memory");
    __builtin_amdgcn_s_setprio(3);
    STAMP_ENTRY();
    const rsrc_t rsP = mk(pd, WS_USED);
    const rsrc_t rsO = mk(a.outd[7], OUT_BYTES);
    const int voy = (lane >> 4) * 16384 + (lane & 15) * 16;
    const int vol = lane * 16;
    const unsigned fvo = lane < NREP ? (unsigned)lane * REP_STRIDE : WS_USED;
    unsigned d1 = WS_USED, y1 = 0, fo1 = WS_USED, fv1 = 0, fo2 = WS_USED, fv2 = 0, ends = 0, n = 0;
    bool ym = false;
    u4 e = {0u, 0u, 0u, 0u};
    while (true) {
      unsigned e0 = __builtin_amdgcn_readfirstlane(e[0]);
      if ((e0 >> 24) != ((n + 1u) & 255u)) {
        asm volatile("" ::: "memory");
        e = sm.fifo2[n & 7u];
        e0 = __builtin_amdgcn_readfirstlane(e[0]);
      }
      unsigned so = WS_USED, d0 = WS_USED, y0 = 0, fo0 = WS_USED, fv0 = 0;
      if ((e0 >> 24) == ((n + 1u) & 255u)) {
        ++n;
        if ((e0 & 16u) != 0u) { ++ends; continue; }
        so = __builtin_amdgcn_readfirstlane(e[1]);
        d0 = __builtin_amdgcn_readfirstlane(e[2]);
        fv0 = __builtin_amdgcn_readfirstlane(e[3]);
        y0 = e0 & 32u;
        fo0 = y0 ? WS_USED : FL_OFF + ((e0 >> 6) & 0x7fffu);
        asm volatile("" ::: "memory");
        if (lane == 0) __hip_atomic_fetch_add(&sm.st[e0 & 7u], 1u, __ATOMIC_RELAXED, __HIP_MEMORY_SCOPE_WORKGROUP);
      } else if (d1 == WS_USED && fo2 == WS_USED) {
        if (ends == NF) break;
        __builtin_amdgcn_s_sleep(1);
        continue;
      }
      const rsrc_t rs = y1 ? rsO : rsMy;
      const int vo = y1 ? voy : vol;
      const unsigned st = y1 ? 4u * 16384u : 1024u;
      if (y0 != 0u && !ym) { ym = true; asm volatile("s_waitcnt vmcnt(0)" ::: "memory"); }
      if (ym) {
        unsigned da = d1, sb = so + 4096u, pa = y0 ? d0 : WS_USED;
#define YSTEP(F, W, FL, SO, OFS) "s_waitcnt vmcnt(" W ")\n\t" FL "s_cbranch_vccz Lpy" #F "_%=\n\t" \
        "v_pk_add_f32 v[136+4*" #F ":137+4*" #F "], v[136+4*" #F ":137+4*" #F "], v[104+4*" #F ":105+4*" #F "]\n\t" \
        "v_pk_add_f32 v[138+4*" #F ":139+4*" #F "], v[138+4*" #F ":139+4*" #F "], v[106+4*" #F ":107+4*" #F "]\n" \
        "Lpy" #F "_%=:\n\tbuffer_store_dwordx4 v[136+4*" #F ":139+4*" #F "], %[vo], %[rs], %[da] offen sc0 sc1\n\t" \
        "buffer_load_dwordx4 v[104+4*" #F ":107+4*" #F "], %[voy], %[ro], %[pa] offen sc0 sc1\n\t" \
        "buffer_load_dwordx4 v[136+4*" #F ":139+4*" #F "], %[vol], %[rp], %[" SO "] offen offset:" #OFS " sc0 sc1\n\t" \
        "s_add_u32 %[da], %[da], %[st]\n\ts_add_u32 %[pa], %[pa], 0x10000\n\t"
#define YOPS : [da] "+s"(da), [pa] "+s"(pa) \
          : [vo] "v"(vo), [vol] "v"(vol), [voy] "v"(voy), [fv] "v"(fv2), [fvo] "v"(fvo), [rs] "s"(rs), [rp] "s"(rsP), [rm] "s"(rsMy), [ro] "s"(rsO), \
            [sa] "s"(so), [sb] "s"(sb), [st] "s"(st), [fo] "s"(fo2), [y1] "s"(y1) : "memory", "scc", "vcc", "v104", "v167"
        asm volatile("s_cmp_lg_u32 %[y1], 0\n\ts_cselect_b64 vcc, -1, 0\n\ts_nop 4\n\t"
          YSTEP(0, "22", "", "sa", 0) YSTEP(1, "22", "", "sa", 1024) YSTEP(2, "22", "", "sa", 2048) YSTEP(3, "22", "", "sa", 3072)
          YSTEP(4, "22", "", "sb", 0) YSTEP(5, "22", "", "sb", 1024) YSTEP(6, "22", "", "sb", 2048) YOPS);
        e = sm.fifo2[n & 7u];
        asm volatile("s_cmp_lg_u32 %[y1], 0\n\ts_cselect_b64 vcc, -1, 0\n\ts_nop 4\n\t"
          YSTEP(7, "21", "buffer_store_byte %[fv], %[fvo], %[rm], %[fo] offen sc0 sc1\n\t", "sb", 3072) YOPS);
      } else {
        unsigned da = d1, sb = so + 4096u;
#define PSTEP(F, R, SO, OFS) "s_waitcnt vmcnt(%[w" #F "])\n\t" PFLAG##F \
        "buffer_store_dwordx4 " R ", %[vo], %[rs], %[da] offen sc0 sc1\n\t" \
        "buffer_load_dwordx4 " R ", %[vol], %[rp], %[" SO "] offen offset:" #OFS " sc0 sc1\n\t" \
        "s_add_u32 %[da], %[da], %[st]\n\t"
#define PFLAG0
#define PFLAG1
#define PFLAG2
#define PFLAG3
#define PFLAG4
#define PFLAG5
#define PFLAG6
#define PFLAG7 "buffer_store_byte %[fv], %[fvo], %[rm], %[fo] offen sc0 sc1\n\t"
#define PWAIT(F) [w##F] "n"(14 - 2 * (8 - PW) + (F + 8 - PW < 7 ? 1 : 0))
#define POPS : [da] "+s"(da) \
          : [vo] "v"(vo), [vol] "v"(vol), [fv] "v"(fv2), [fvo] "v"(fvo), [rs] "s"(rs), [rp] "s"(rsP), [rm] "s"(rsMy), \
            [sa] "s"(so), [sb] "s"(sb), [st] "s"(st), [fo] "s"(fo2), \
            PWAIT(0), PWAIT(1), PWAIT(2), PWAIT(3), PWAIT(4), PWAIT(5), PWAIT(6), PWAIT(7) \
          : "memory", "scc", "v136", "v137", "v138", "v139", "v140", "v141", "v142", "v143", "v144", "v145", "v146", "v147", "v148", "v149", "v150", "v151", \
            "v152", "v153", "v154", "v155", "v156", "v157", "v158", "v159", "v160", "v161", "v162", "v163", "v164", "v165", "v166", "v167"
        asm volatile("s_nop 4\n\t"
          PSTEP(0, "v[136:139]", "sa", 0) PSTEP(1, "v[140:143]", "sa", 1024) PSTEP(2, "v[144:147]", "sa", 2048) PSTEP(3, "v[148:151]", "sa", 3072)
          PSTEP(4, "v[152:155]", "sb", 0) PSTEP(5, "v[156:159]", "sb", 1024) PSTEP(6, "v[160:163]", "sb", 2048) POPS);
        e = sm.fifo2[n & 7u];
        asm volatile(PSTEP(7, "v[164:167]", "sb", 3072) POPS);
      }
      fo2 = fo1; fv2 = fv1;
      d1 = d0; y1 = y0; fo1 = fo0; fv1 = fv0;
    }
    asm volatile("s_waitcnt vmcnt(0)" ::: "memory");
    return;
  }
  STAMP_ENTRY();
  const unsigned long long t0 = now();
  if (jb < 8u) {
    if (b == 56u && wv == 0) {
      const unsigned ctr_raw = __builtin_amdgcn_raw_buffer_load_b32(rsMy, 0, CTR2_OFF + b * 64u, AUX_SYS);
      const unsigned call = next_call(ctr_raw);
      if (lane == 0) __builtin_amdgcn_raw_buffer_store_b32(call, rsMy, 0, CTR2_OFF + b * 64u, AUX_SYS);
      if (lane < 8) __builtin_amdgcn_raw_buffer_store_b32(0u, rsMy, lane * 128, QUEUE_OFF + ((call & 1u) ^ 1u) * 1024u, AUX_SYS);
      STAMP(15);
    }
    return;
  }

  const unsigned ctr_raw = __builtin_amdgcn_raw_buffer_load_b32(rsMy, 0, CTR_OFF + b * 4u, AUX_SYS);
  f4 v[16];
#define XG(c) ((((c) >> 2) & 7u) * 16u + ((((c) >> 2) + ((c) >> 7)) & 3u) * 4u + (((c) >> 5) & 3u))
  if (wv == 0) {
    if (b >= 64u + D0FROM) __builtin_amdgcn_s_sleep(D0);
    load_piece(v, px, pw, true, XG(b - 64u), b & 3u, lane);
  }
  __builtin_amdgcn_sched_barrier(0);
  asm volatile("" ::: "memory");
  const unsigned cidx = xc * CPX + (jb - 8u);
  const unsigned cls = cidx >> 5;
  const unsigned rg = ((cidx & 1u) << 4) | ((cidx & 31u) >> 1);
  static_assert(NCLS == 6, "");
  const int nt = cidx < 128u ? NT : NT - 1;
  if (tid < 8) sm.slot[tid].st = 0u;
  if (tid == 0) sm.abort_ = 0u;
  unsigned call;
  {
    const unsigned c = (unsigned)wv * 192u + (b - 64u);
    const bool isx = c < 512u;
    const bool cv = c < 1024u;
    const unsigned cg = c < 960u ? ((c - 512u) * 9363u) >> 16 : (c - 960u + RG) & 63u, cr = c < 960u ? c - 512u - cg * 7u : 7u;
    const unsigned G = isx ? XG(c) : (cr < 4u ? h : h ^ 1u) * 64u + cg;
    const unsigned q = isx ? c & 3u : cr < 4u ? cr : (cr - 3u) & 3u;
    const unsigned cr_ = !isx ? (G >> 6) * 4u + ((unsigned)lane & 3u) : (lane & 3) == 0 ? G >> 4 : (((G >> 4) & 4u) ^ 4u) + ((G >> 2) & 3u);
    const unsigned cl_ = !isx ? 32u + (G & 63u) : (lane & 3) == 0 ? G & 15u : 16u + ((G >> 4) & 3u) * 4u + (G & 3u);
    char* fp = lane_ws(lane < 16 ? cr_ : rank, rank);
    static_assert(NREP <= 4, "");
    const unsigned wf = wflag(G & 63u, 0u);
    unsigned fofs = q + (lane >= 16 ? FL_OFF + ((unsigned)lane & 3u) * REP_STRIDE + FE_REL + (wf >> 10) * 128u + (wf & 127u) :
                    cr_ == rank ? FL_OFF + (unsigned)(lane >> 2) * REP_STRIDE + (isx ? FX_REL + cl_ * 128u : wf) : AV_OFF + rank * 512u + cl_ * 4u);
    if (!(lane < 16 ? (lane & 3) < (isx ? ((G >> 4) == rank ? 1 : 2) : c < 1024u - RG ? 4 : 0) && lane < (cr_ == rank ? 4 * NREP : 4) : !isx && cr >= 4u && lane < 16 + NREP)) fofs = 0u;
    unsigned pi = ((isx ? 0u : 128u) + G) * 4u + q;
    asm volatile("" : "+v"(fofs) :: "memory");
    asm volatile("" : "+s"(pi) :: "memory");
    call = next_call(ctr_raw);
    WG_BARRIER();
    if (wv != 0 && cv) {
      for (int i = 0; i < wv; ++i) __builtin_amdgcn_s_sleep(CDELAY);
      load_piece(v, px, pw, isx, G, q, lane);
    }
    if (tid == NTHREADS - 64) __builtin_amdgcn_raw_buffer_store_b32(call, rsMy, 0, CTR_OFF + b * 4u, AUX_SYS);
    if (cv) {
      u4* const tr = (u4*)&sm.xl[wv & 7][0][0];
      const unsigned code = conv_piece(v, isx, tr, lane);
      fp += fofs + (call & 1u) * FLAG_PAR;
      asm volatile("" : "+v"(fp));
      u4 o[8];
#pragma unroll
      for (int f = 0; f < 8; ++f) o[f] = tr[66 * f + (lane >> 5) * 33 + (lane & 31)];
      __builtin_amdgcn_sched_barrier(0);
#pragma unroll
      for (int f = 0; f < 8; ++f)
        __builtin_amdgcn_raw_buffer_store_b128(o[f], rsMy, lane * 16, OWN_OFF + (call & 1u) * OWN_PAR + pi * PIECE + f * 1024, AUX_SYS);
      asm volatile("s_waitcnt vmcnt(0)" ::: "memory");
      if (fofs != 0u) flag_st(fp, tag_of(call) | code);
      STAMP(24 + wv);
    }
  }
  const unsigned dbase = (call & 1u) * PAR_STRIDE;
  const unsigned fbase = FL_OFF + (call & 1u) * FLAG_PAR + (cidx % NREP) * REP_STRIDE;
  const unsigned obase = OWN_OFF + (call & 1u) * OWN_PAR;
  if (EXP & 1) return;
#define LNN(x) ((x) > nt ? (x) - nt + NT : (x))
  const unsigned g2 = cidx < 128u ? cidx >> 1 : cidx - 128u, half = cidx >> 7;
  const unsigned lp = ((rg - 4u * rank) & 15u) - (16u - NL);
  const bool latecu = rg < 16u && lp < NL;
  const unsigned ve = rg < 16u ? 8u : 4u + ((rank ^ (rg >> 2)) & 3u);
  const int vo2 = (lane >> 3) * 16384 + ((lane ^ (lane >> 3)) & 7) * 16;

  if (wv < 8) {
    const unsigned v = (unsigned)wv;
    const unsigned tag4 = tag_of(call) * 0x01010101u;
    const unsigned lb = lds_addr(&sm.xl[v][0][0]);
    const rsrc_t rsX = mk(myws - lb, WS_USED + lb);
    const bool ew = v == ve;
    const unsigned rb = v == 0u || ew ? obase + xgroup(rank, ew ? rg - 16u : rg) * 4u * PIECE : dbase + v * SRC_STRIDE + rg * 4u * PIECE;
    const unsigned ab = (v == 0u || ew ? obase + (128u + 64u * (ew ? h ^ 1u : h)) * 4u * PIECE : dbase + v * SRC_STRIDE + W_OFF) + cls * 4u * PIECE;
    const unsigned og = (v >> 2) ^ 1u, hq = (half ? 2u + og : cidx & 1u) * 4u + (v & 3u);
    if (latecu && v - 1u < 3u && !half) return;
#define VR(B, I) "v[" B "+4*(" I "):" B "+4*(" I ")+3]"
#define LD1(R, S, F) "buffer_load_dwordx4 " R ", v1, %[rs], " S " offen offset:1024*" F " sc1\n\t"
#define LDI(X, F) LD1(VR(X, F), "s[74+" F "/4]", "(" F "&3)")
#define LDN(X, F)
#define LOAD8(X) LDI(X, "0") LDI(X, "1") LDI(X, "2") LDI(X, "3") LDI(X, "4") LDI(X, "5") LDI(X, "6") LDI(X, "7")
#define RES(Q, S, R7) LD1(VR("100", "4*" Q), S, "0") LD1(VR("100", "4*" Q "+1"), S, "1") LD1(VR("100", "4*" Q "+2"), S, "2") LD1(VR("100", "4*" Q "+3"), S, "3") \
      "s_addk_i32 " S ", 0x1000\n\t" LD1(VR("36", "4*" Q), S, "0") LD1(VR("36", "4*" Q "+1"), S, "1") LD1(VR("36", "4*" Q "+2"), S, "2") LD1(R7, S, "3")
#define XW(I) "ds_write_b128 v1, " VR("36", I) " offset:1024*" I "\n\t"
#define XRD(Q, I) "ds_read_b128 " VR("20", I) ", v1 offset:4096*" Q "+1024*" I "\n\t"
#define POLL "s_load_dwordx4 s[64:67], %[ws], s70 glc\n\t"
#define ZACC "v_mov_b64 v[4:5], 0\n\tv_mov_b64 v[6:7], 0\n\tv_mov_b64 v[8:9], 0\n\tv_mov_b64 v[10:11], 0\n\t" \
      "v_mov_b64 v[12:13], 0\n\tv_mov_b64 v[14:15], 0\n\tv_mov_b64 v[16:17], 0\n\tv_mov_b64 v[18:19], 0\n\t"
#define PEEK "s_xor_b64 s[80:81], s[64:65], s[98:99]\n\ts_quadmask_b64 s[80:81], s[80:81]\n\ts_andn2_b32 s80, s68, s80\n\t"
#define PICK(N, NONE) PEEK "s_cbranch_scc0 " NONE "\n\t" \
      "s_ff1_i32_b32 s80, s80\n\ts_lshl_b32 s74, s80, 12\n\ts_add_u32 s74, s74, s69\n\ts_add_u32 s75, s74, 0x1000\n\t" \
      "s_and_b32 s83, s80, 8\n\ts_cselect_b32 s81, s65, s64\n\ts_cselect_b32 s73, s96, 0\n\ts_add_u32 s73, s73, s72\n\ts_lshl_b32 s83, 0xff, s83\n\t" \
      "s_and_b32 s81, s81, 0x0f0f0f0f\n\ts_add_u32 s81, s81, s94\n\ts_lshl2_add_u32 s82, s80, 0x7fffc\n\ts_bfe_u32 s81, s81, s82\n\t" \
      "s_lshl_b32 s81, s81, 8\n\ts_or_b32 s73, s73, s80\n\ts_or_b32 s73, s73, s81\n\t" \
      "s_bitset0_b32 s68, s80\n\ts_and_b32 s83, s83, s68\n\ts_cbranch_scc0 La" N "%=\nLb" N "%=:\n\t"
#define DBGST(S, SH, O)
#define ADV(N) "La" N "%=:\n\t" DBGST("s73", "16", "0x4f0000") "s_bitset1_b32 s73, 7\n" \
      "Ld" N "%=:\n\ts_and_b32 s81, s68, 0xff\n\ts_cbranch_scc1 Lb" N "%=\n\t" \
      "s_mov_b32 s64, s65\n\ts_mov_b32 s65, s66\n\ts_mov_b32 s66, s67\n\ts_mov_b32 s67, 0\n\t" \
      "s_add_u32 s69, s69, s90\n\ts_add_u32 s70, s70, 4\n\ts_add_u32 s72, s72, s96\n\ts_lshr_b32 s68, s68, 8\n\ts_sub_u32 s71, s71, 1\n\t" \
      "s_cmp_gt_i32 s71, 0\n\ts_cselect_b32 s81, s91, 0\n\ts_or_b32 s68, s68, s81\n\ts_cbranch_scc1 Ld" N "%=\n\ts_branch Lb" N "%=\n"
#define MF(W, XA, X, F, LD) "s_waitcnt vmcnt(" W ")\n\tv_mfma_f32_32x32x16_f16 v[4:19], " XA ", " VR(X, F) ", v[4:19]\n\t" LD(X, F)
#define WL(F) "15"
#define WM(F) "15-" F
#define WS(F) "7-" F
#define BODY(X, Q, W, LD, MID, R3, X7, END) XRD(Q, "0") XRD(Q, "1") XRD(Q, "2") R3 "ds_read_b32 v3, v2 offset:4096\n\t" \
      MF(W("0"), VR("100", "4*" Q), X, "0", LD) MF(W("1"), VR("100", "4*" Q "+1"), X, "1", LD) MF(W("2"), VR("100", "4*" Q "+2"), X, "2", LD) MF(W("3"), VR("100", "4*" Q "+3"), X, "3", LD) \
      "s_waitcnt lgkmcnt(0)\n\t" MID \
      MF(W("4"), "v[20:23]", X, "4", LD) MF(W("5"), "v[24:27]", X, "5", LD) MF(W("6"), "v[28:31]", X, "6", LD) MF(W("7"), X7, X, "7", LD) \
      "s_branch " END "\n"
#define TREE(N, X, D, W, LD, MID, END) "s_bitcmp1_b32 " D ", 2\n\ts_cbranch_scc1 Lh" N "%=\n\ts_bitcmp1_b32 " D ", 1\n\ts_cbranch_scc1 Li" N "%=\n\t" \
      BODY(X, "0", W, LD, MID, XRD("0", "3"), "v[32:35]", END) "Li" N "%=:\n\t" BODY(X, "1", W, LD, MID, XRD("1", "3"), "v[32:35]", END) \
      "Lh" N "%=:\n\ts_bitcmp1_b32 " D ", 1\n\ts_cbranch_scc1 Lj" N "%=\n\t" \
      BODY(X, "2", W, LD, MID, XRD("2", "3"), "v[32:35]", END) "Lj" N "%=:\n\t" BODY(X, "3", W, LD, MID, "", "v[164:167]", END)
#define TAIL(D, NEW, NEXT) "s_add_u32 s79, s79, 1\n\ts_bitcmp1_b32 " D ", 7\n\ts_mov_b32 " D ", " NEW "\n\ts_cbranch_scc0 " NEXT "\n\ts_call_b64 s[84:85], Lflush%=\n\ts_branch " NEXT "\n"
#define STEP(X, Y, DX, DY) "Ls" X "%=:\n\t" \
      "s_lshr_b32 s80, " DX ", 8\n\ts_cmp_eq_u32 s79, 0\n\ts_cselect_b32 s78, s80, s78\n\ts_cmp_lg_u32 s78, s80\n\ts_cbranch_scc1 Lk" X "%=\n" \
      "Lp" X "%=:\n\t" PICK(X, "Ln" X "%=") \
      "s_cmp_eq_u32 " DY ", 0\n\ts_cbranch_scc1 Ly" X "%=\n\t" \
      TREE("l" X, X, DX, WL, LDI, "", "Lt" X "%=") \
      "Ly" X "%=:\n\t" LOAD8(Y) "s_mov_b32 " DY ", s73\n\ts_branch Lp" X "%=\n" \
      "Lk" X "%=:\n\ts_call_b64 s[84:85], Lflush%=\n\ts_branch Ls" X "%=\n" \
      ADV(X) \
      "Ln" X "%=:\n\ts_mov_b32 s73, 0\n\ts_cmp_lg_u32 " DY ", 0\n\ts_cbranch_scc1 Lm" X "%=\n\t" \
      TREE("s" X, X, DX, WS, LDN, POLL, "Lu" X "%=") \
      "Lm" X "%=:\n\t" TREE("m" X, X, DX, WM, LDN, "", "Lt" X "%=") \
      "Lu" X "%=:\n\t" TAIL(DX, "0", "Lidle%=") \
      "Lt" X "%=:\n\t" TAIL(DX, "s73", "Ls" Y "%=")
#define LDX(I) "v_ldexp_f32 v[4+" I "], v[4+" I "], s82\n\t"
#define LDX4(I) LDX("4*" I) LDX("4*" I "+1") LDX("4*" I "+2") LDX("4*" I "+3")
#define FW1(I) LDX4(I) "ds_write2st64_b32 v2, v[4+4*" I "], v[5+4*" I "] offset0:4*" I " offset1:4*" I "+1\n\t" \
      "ds_write2st64_b32 v2, v[6+4*" I "], v[7+4*" I "] offset0:4*" I "+2 offset1:4*" I "+3\n\t"
#define FW2(I) LDX4(I) "v_xor_b32 v20, 32*" I ", v2\n\tds_write_b128 v20, " VR("4", I) "\n\t"
#define ABORT "v_mov_b32 v20, %[ex]\n\tv_mov_b32 v21, 1\n\tds_write_b32 v20, v21\n\t"
#define STR_(X) #X
#define STR(X) STR_(X)
#define RWAIT(L) "s_xor_b32 s81, s94, %[tag]\n\ts_and_b32 s81, s81, 0xf0f0f0f0\n\ts_cbranch_scc0 Lgo%=\n\t" \
      "s_sleep 4\n\ts_add_u32 s86, s86, 1\n\ts_cmp_le_u32 s86, 0x200000\n\ts_cbranch_scc1 " L "\n\t" ABORT
#define DBGI(X)
      asm volatile(
        "v_mbcnt_lo_u32_b32 v0, -1, 0\n\tv_mbcnt_hi_u32_b32 v0, -1, v0\n\tv_lshl_add_u32 v1, v0, 4, %[lb]\n\tv_lshl_add_u32 v2, v0, 2, %[sl]\n\t"
        "s_mov_b32 s70, %[fo]\n\ts_mov_b32 s80, %[rb]\n\ts_mov_b32 s69, %[ab]\n\ts_mov_b32 s71, %[tl]\n\ts_mov_b32 s72, 0x80000\n\t"
        "s_mov_b32 s90, %[ts1]\n\ts_movk_i32 s91, 0xaa\n\ts_movk_i32 s92, 0x2000\n\ts_mov_b32 s93, 0\n\ts_mov_b64 s[86:87], 0\n\t" DBGI("s_mov_b32 s95, %[dbg]\n\t")
        "s_mov_b32 s96, s72\n\ts_mov_b32 s98, %[tag]\n\ts_mov_b32 s99, %[tag]\n\ts_cmp_eq_u32 s71, 0\n\ts_cbranch_scc1 Lend%=\n"
        "Lq%=:\n\ts_load_dword s94, %[ws], %[fx] glc\n\ts_waitcnt lgkmcnt(0)\n\t" RWAIT("Lq%=")
        "Lgo%=:\n\ts_and_b32 s94, s94, 0x0f0f0f0f\n\ts_add_u32 s81, s80, s92\n\ts_add_u32 s82, s81, s92\n\ts_add_u32 s83, s82, s92\n\t"
        "s_cmp_lg_u32 s93, 0\n\ts_cbranch_scc1 Lr%=\n\t"
        RES("0", "s80", VR("36", "3")) "Lr%=:\n\t" RES("1", "s81", VR("36", "7")) RES("2", "s82", VR("36", "11")) RES("3", "s83", "v[164:167]")
        POLL
        "s_lshr_b32 s68, s91, 8\n\ts_or_b32 s68, s68, s91\n\ts_mov_b64 s[76:77], 0\n\ts_mov_b64 s[78:79], 0\n\t" ZACC
        "s_waitcnt vmcnt(0)\n\t"
        XW("0") XW("1") XW("2") XW("3") XW("4") XW("5") XW("6") XW("7") XW("8") XW("9") XW("10") XW("11") XW("12") XW("13") XW("14")
        DBGST("s92", "17", "-832")
        "Lidle%=:\n\ts_waitcnt lgkmcnt(0)\n"
        "Lc%=:\n\t" PICK("0", "Lnone%=") LOAD8("36") "s_mov_b32 s76, s73\n\t"
        PEEK "s_cbranch_scc1 Ls36%=\n\ts_cmp_eq_u32 s68, 0\n\ts_cbranch_scc1 Ls36%=\n\t" POLL "s_waitcnt lgkmcnt(0)\n"
        STEP("36", "68", "s76", "s77")
        STEP("68", "36", "s77", "s76")
        ADV("0")
        "Lflush%=:\n\tv_readfirstlane_b32 s81, v3\n\ts_and_b32 s82, s78, 0xff\n\ts_lshl4_add_u32 s82, s82, -254\n\ts_cmp_lg_u32 s81, 0\n\ts_cbranch_scc1 Lw%=\n"
        "Lg%=:\n\ts_lshr_b32 s81, s78, 8\n\ts_add_u32 s81, s81, s79\n\ts_cmp_lg_u32 s93, 0\n\ts_cbranch_scc1 Lx%=\n\t"
        FW1("0") FW1("1") FW1("2") FW1("3") "s_branch Lz%=\n"
        "Lx%=:\n\t" FW2("0") FW2("1") FW2("2") FW2("3")
        "Lz%=:\n\tv_mov_b32 v3, s81\n\ts_mov_b64 exec, 1\n\tds_write_b32 v2, v3 offset:4096\n\ts_mov_b64 exec, -1\n\t"
        DBGST("s78", "8", "0") ZACC "s_mov_b32 s79, 0\n\ts_setpc_b64 s[84:85]\n"
        "Lw%=:\n\ts_sleep " STR(CSLEEP) "\n\tds_read_b32 v3, v2 offset:4096\n\ts_add_u32 s87, s87, 1\n\ts_waitcnt lgkmcnt(0)\n\tv_readfirstlane_b32 s81, v3\n\t"
        "s_cmp_eq_u32 s81, 0\n\ts_cbranch_scc1 Lg%=\n\ts_cmp_lt_u32 s87, 0x2000000\n\ts_cbranch_scc1 Lw%=\n\t" ABORT "s_branch Lg%=\n"
        "Lnone%=:\n\ts_cmp_eq_u32 s68, 0\n\ts_cbranch_scc1 Lend%=\n\ts_add_u32 s86, s86, 1\n\ts_cmp_gt_u32 s86, 0x200000\n\ts_cbranch_scc1 Lo%=\n\t"
        "s_sleep " STR(PSLEEP) "\n\t" POLL "s_branch Lidle%=\n"
        "Lo%=:\n\t" ABORT "s_mov_b64 s[64:65], s[98:99]\n\ts_branch Lc%=\n"
        "Lend%=:\n\ts_cmp_eq_u32 s93, %[rb2]\n\ts_cbranch_scc1 Lfin%=\n\t"
        "s_mov_b32 s93, %[rb2]\n\ts_mov_b32 s80, %[rb2]\n\ts_mov_b32 s69, %[ab2]\n\ts_mov_b32 s70, %[fo2]\n\ts_and_b32 s71, %[t2], 15\n\t"
        "s_andn2_b32 s72, %[t2], 15\n\ts_mov_b32 s96, %[ti]\n\t"
        "s_mov_b32 s90, 0x8000\n\ts_mov_b32 s91, %[m2]\n\ts_mov_b32 s92, %[ss]\n\t" DBGI("s_mov_b32 s95, %[dbg2]\n\t")
        "v_lshrrev_b32 v2, 5, v0\n\tv_xor_b32 v2, v2, v0\n\tv_and_b32 v2, 7, v2\n\tv_lshl_add_u32 v2, v2, 4, %[sl]\n\t"
        "v_and_b32 v3, 31, v0\n\tv_lshl_add_u32 v2, v3, 7, v2\n"
        "Lv%=:\n\ts_load_dword s81, %[ws], %[fw] offset:128 glc\n\ts_load_dword s82, %[ws], %[fw] offset:256 glc\n\ts_load_dword s83, %[ws], %[fw] offset:384 glc\n\t"
        "s_and_b32 s94, %[tag], 0xff\n\ts_waitcnt lgkmcnt(0)\n\ts_bfe_u32 s81, s81, %[sh]\n\ts_bfe_u32 s82, s82, %[sh]\n\ts_bfe_u32 s83, s83, %[sh]\n\t"
        "s_lshl_b32 s81, s81, 8\n\ts_lshl_b32 s83, s83, 8\n\ts_or_b32 s94, s94, s81\n\ts_or_b32 s82, s82, s83\n\ts_pack_ll_b32_b16 s94, s94, s82\n\t" RWAIT("Lv%=")
        "s_branch Lgo%=\n"
        "Lfin%=:\n\ts_endpgm"
        :
        : [rs] "s"(rsX), [ws] "s"(myws), [tag] "s"(tag4), [ex] "s"(lds_addr(&sm.abort_)), [lb] "s"(lb), [sl] "s"(lds_addr(&sm.slot[v])),
          [fx] "s"(fbase + FX_REL + (ew ? (rg - 16u) * 128u : rg * 128u + v * 4u)), [rb] "s"(rb), [fo] "s"(fbase + (ew ? FE_REL + cls * 128u : (cls * 8u + v) * 128u)), [ab] "s"(ab - 4096u), [tl] "s"(latecu && v - 1u < 3u ? 0 : nt),
          [fw] "s"(fbase + wflag(g2, 0u)), [sh] "s"(0x80000u + 8u * (v & 3u)), [rb2] "s"(half >= og ? dbase + W_OFF + (g2 * 4u + (v & 3u)) * PIECE : 0u),
          [fo2] "s"(fbase + FZ_REL + hq * 128u), [ab2] "s"(dbase + hq * 4u * LH * PIECE - 4096u),
          [t2] "s"((((unsigned)nt + 1u + og) << 19) + LH - OOO), [m2] "n"(0xa8 << (8 * OOO)), [ti] "s"(0x80000u << half),
          [ts1] "n"(NCLS * 4u * PIECE), [ss] "n"(SRC_STRIDE)
        : "memory", "scc", "v167",
          "s64", "s65", "s66", "s67", "s68", "s69", "s70", "s71", "s72", "s73", "s74", "s75", "s76", "s77", "s78", "s79",
          "s80", "s81", "s82", "s83", "s84", "s85", "s86", "s87", "s88", "s89", "s90", "s91", "s92", "s93", "s94", "s95", "s96", "s98", "s99");
  } else {
    const bool own = rg < 16u;
    const unsigned o = own ? rank : ((4u * (h ^ 1u) + ((rg >> 2) & 3u)) & 7u);
    const rsrc_t rsO = mk(a.outd[7], OUT_BYTES);
    const rsrc_t rsD = mk(a.wsd[(o - rank - 1u) & 7u], WS_USED);
    const int lr = (lane >> 4) + ((lane >> 3) & 1) * 4;
    unsigned* const sp = lane < 8 ? &sm.slot[lane].st : &sm.abort_;
    unsigned spins = 0, sv;
    auto await = [&](unsigned lo, unsigned hi) {
      __builtin_amdgcn_s_setprio(0);
      while (true) {
        sv = __hip_atomic_load(sp, __ATOMIC_RELAXED, __HIP_MEMORY_SCOPE_WORKGROUP);
        if (__builtin_amdgcn_ballot_w64((sv >> 3) - lo <= hi - lo && (((sv >> 3) ^ lo) & 1u) == 0u) != 0ull) break;
        if ((++spins & 1023u) == 0u && now() - t0 > 3 * DEADLINE) {
          lds_st(&sm.abort_, 1u);
          if (lane == 0) __builtin_amdgcn_raw_buffer_store_b32(0x7fc00000u, mk(a.outd[7], OUT_BYTES), 0, 0, AUX_SYS);
          __builtin_amdgcn_endpgm();
        }
        __builtin_amdgcn_s_sleep(CSLEEP);
      }
      __builtin_amdgcn_s_setprio(EPRIO);
      asm volatile("" ::: "memory");
    };
    auto absorb = [&](bool mine, f4 (&s)[4], unsigned& need) {
      unsigned mm = (unsigned)__builtin_amdgcn_ballot_w64(mine);
      while (mm != 0u) {
        f4 p[2][4];
        const int v1 = __builtin_ctz(mm);
        mm &= mm - 1u;
#pragma unroll
        for (int i = 0; i < 4; ++i) p[0][i] = ((const f4*)sm.slot[v1].d)[i * 64 + lane];
        need -= __builtin_amdgcn_readlane(sv, v1) & 7u;
        if (mm != 0u) {
          const int v2 = __builtin_ctz(mm);
          mm &= mm - 1u;
#pragma unroll
          for (int i = 0; i < 4; ++i) p[1][i] = ((const f4*)sm.slot[v2].d)[i * 64 + lane];
          need -= __builtin_amdgcn_readlane(sv, v2) & 7u;
#pragma unroll
          for (int i = 0; i < 4; ++i) p[0][i] += p[1][i];
        }
#pragma unroll
        for (int i = 0; i < 4; ++i) s[i] += p[0][i];
      }
      asm volatile("" ::: "memory");
      if (mine) lds_st(sp, 0u);
      if (__builtin_amdgcn_readlane(sv, 8) != 0u) s[0] = (f4){__builtin_nanf(""), 0.f, 0.f, 0.f};
    };
    unsigned ts = (unsigned)wv - 7u;
    for (; ts <= (unsigned)nt; ts += 2u) {
      f4 s[4], se[4];
#pragma unroll
      for (int i = 0; i < 4; ++i) s[i] = se[i] = (f4){0.f, 0.f, 0.f, 0.f};
      const unsigned g = (ts - 1u) * NCLS + cls;
      const unsigned yi = (call & 1u) * 512u + (o & 3u) * 128u + (g >> 1) * 4u + (rg & 3u);
      unsigned need = latecu ? 20u : own ? 32u : 28u, neede = own ? 0u : 4u, efo = WS_USED;
      do {
        await(ts, ts);
        absorb((sv >> 3) == ts && (unsigned)lane != ve, s, need);
        if (efo != WS_USED && need != 0u) {
          asm volatile("s_waitcnt vmcnt(0)" ::: "memory");
          if (lane == 0) __builtin_amdgcn_raw_buffer_store_b32(call, rsMy, 0, efo, AUX_SYS);
          efo = WS_USED;
        }
        if ((__builtin_amdgcn_readlane(sv, ve) >> 3) == ts) {
          absorb((unsigned)lane == ve, se, neede);
          if (neede == 0u) {
#pragma unroll
            for (int i = 0; i < 4; ++i) __builtin_amdgcn_raw_buffer_store_b128(__builtin_bit_cast(u4, se[i]), rsO, lr * 16384 + (lane & 7) * 16, (32u * (rg - 16u) + (unsigned)i * 8u) * 16384u + (2048u * (h ^ 1u) + 32u * g) * 4u, AUX_SYS);
            efo = YRDY_OFF + yi * 16u + 8u + (g & 1u) * 4u;
          }
        }
      } while ((need | neede) != 0u);
      unsigned yfo = WS_USED;
      if (!own) {
#pragma unroll
        for (int i = 0; i < 4; ++i) __builtin_amdgcn_raw_buffer_store_b128(__builtin_bit_cast(u4, s[i]), rsMy, lr * 256 + (lane & 7) * 16, YST_OFF + yi * PIECE + (g & 1u) * 128u + (unsigned)i * 2048u, AUX_SYS);
        yfo = YRDY_OFF + ((rank & 3u) * 128u + (yi & 0x27fu)) * 16u + (g & 1u) * 4u;
      } else {
        const unsigned so = 32u * rg * 16384u + (2048u * h + 32u * g) * 4u;
#pragma unroll
        for (int i = 0; i < 4; ++i) __builtin_amdgcn_raw_buffer_store_b128(__builtin_bit_cast(u4, s[i]), rsO, lr * 16384 + (lane & 7) * 16, so + (unsigned)i * 8u * 16384u, AUX_SYS);
        if (latecu) yfo = PRDY_OFF + (lp * 64u + g) * 4u;
      }
      STAMP(95 + ts);
      if (yfo != WS_USED) {
        asm volatile("s_waitcnt vmcnt(0)" ::: "memory");
        if (lane == 0) __builtin_amdgcn_raw_buffer_store_b32(call, rsD, 0, yfo, AUX_SYS);
        if (lane == 0 && efo != WS_USED) __builtin_amdgcn_raw_buffer_store_b32(call, rsMy, 0, efo, AUX_SYS);
      }
    }
    const unsigned u = ts - 1u - (unsigned)nt, l0 = half ? 2u + u : 4u * u + (cidx & 1u), ls = 8u >> half;
    f4 s[LH][4];
    unsigned need[LH], left = 0;
#define SO2(K) (((4u * rank + 16u - NL + l0 + ls * (K)) & 15u) * 32u * 16384u + (2048u * h + 32u * g2) * 4u)
#pragma unroll
    for (unsigned k = 0; k < LH; ++k) {
      need[k] = 12u;
      if (l0 + ls * k < NL) {
        ++left;
        while (poll1(myws, PRDY_OFF + ((l0 + ls * k) * 64u + g2) * 4u) != call) {
          if ((++spins & 63u) == 0u && now() - t0 > 3 * DEADLINE) { lds_st(&sm.abort_, 1u); break; }
          __builtin_amdgcn_s_sleep(4);
        }
      }
#pragma unroll
      for (int i = 0; i < 4; ++i) s[k][i] = __builtin_bit_cast(f4, __builtin_amdgcn_raw_buffer_load_b128(rsO, vo2, l0 + ls * k < NL ? SO2(k) + (unsigned)i * 8u * 16384u : OUT_BYTES, AUX_LD));
    }
    while (left != 0u) {
      await(ts, ts + 2u * LH);
#pragma unroll
      for (unsigned k = 0; k < LH; ++k) {
        if (__builtin_amdgcn_ballot_w64((sv >> 3) == ts + 2u * k) == 0ull) continue;
        absorb((sv >> 3) == ts + 2u * k, s[k], need[k]);
        if (need[k] != 0u) continue;
#pragma unroll
        for (int i = 0; i < 4; ++i) __builtin_amdgcn_raw_buffer_store_b128(__builtin_bit_cast(u4, s[k][i]), rsO, vo2, SO2(k) + (unsigned)i * 8u * 16384u, AUX_SYS);
        --left;
        STAMP(95 + LNN((int)(ts + 2u * k)));
      }
    }
  }
}

extern "C" void kernel_launch(int rank, int nranks, ncclComm_t comm,
                              hipStream_t stream,
                              void* const* d_in, const int* in_sizes, int n_in,
                              void* d_out, int out_size,
                              const int* in_dtypes, int out_dtype,
                              void* const* d_out_all,
                              void* const* d_ws_all, size_t ws_bytes) {
  (void)comm; (void)in_dtypes; (void)out_dtype; (void)d_out;
  if (nranks != NR || rank < 0 || rank >= NR || n_in < 2 || ws_bytes < WS_USED ||
      in_sizes[0] != 4096 * 512 || in_sizes[1] != 512 * 4096 || out_size != 512 * 4096) {
    fprintf(stderr, "kernel_launch: unsupported configuration\n");
    return;
  }
  Args a;
  a.x = (const float*)d_in[0];
  a.w = (const float*)d_in[1];
  for (int p = 0; p < NR; ++p) { a.wsd[p] = (char*)d_ws_all[(rank + 1 + p) & 7]; a.outd[p] = (float*)d_out_all[(rank + 1 + p) & 7]; }
  a.myws = (char*)d_ws_all[rank];
  a.rank = (unsigned)rank;
  a.pad_ = 0u;
  hipLaunchKernelGGL(fused_gemm_kernel, dim3(256), dim3(NTHREADS), 0, stream, a);
  HIP_CHECK(hipGetLastError());
}
```

```cpp
#include <hip/hip_runtime.h>
#include <rccl/rccl.h>
#include <cstdio>

#define HIP_CHECK(expr) do { hipError_t _e = (expr); if (_e != hipSuccess) { \
    fprintf(stderr, "HIP error %s at %s:%d\n", hipGetErrorString(_e), __FILE__, __LINE__); \
    return; } } while (0)

#define WIN 6
#define PW 8
#define EXP 0
#define PSLEEP 1
#define NCLS 6
#define NW 2
#define NF 3
#define SPL 1
#define NREP 4
#define RG 54u
#define L3 4u
#define LROT 5u
#define OOO 1
#define D0 24
#define CD0 40
#define JD 96u
#define FWS 8
#define SHADOW 1
#define EPRIO 2
#define CSLEEP 1
#define FDELAY 40
#define YSLEEP 1
#define FSLEEP 8
#define CDELAY 96

typedef float f4 __attribute__((ext_vector_type(4)));
typedef float f16v __attribute__((ext_vector_type(16)));
typedef unsigned u4 __attribute__((ext_vector_type(4)));
typedef _Float16 h8 __attribute__((ext_vector_type(8)));
typedef __amdgpu_buffer_rsrc_t rsrc_t;

constexpr int NR = 8;
constexpr unsigned PIECE = 8192;
constexpr unsigned SRC_STRIDE = 384u * PIECE;
constexpr unsigned W_OFF = 128u * PIECE;
constexpr unsigned PAR_STRIDE = 24u << 20;
constexpr unsigned FLAG_OFF = 48u << 20;
constexpr unsigned FL_OFF = FLAG_OFF + (704u << 10);
constexpr unsigned FX_REL = 6144;
constexpr unsigned FZ_REL = 10240;
constexpr unsigned FE_REL = 12288;
constexpr unsigned REP_STRIDE = 32768u + 4352u;
constexpr unsigned FLAG_PAR = 16384;
constexpr unsigned CTR_OFF = FLAG_OFF + 16384;
constexpr unsigned TMO_OFF = FLAG_OFF + 40960;
constexpr unsigned QUEUE_OFF = FLAG_OFF + 45056;
constexpr unsigned YRDY_OFF = FLAG_OFF;
constexpr unsigned HF_OFF = FLAG_OFF + 57344;
constexpr unsigned PRDY_OFF = FLAG_OFF + 61440;
constexpr unsigned AD_OFF = FLAG_OFF + (1u << 20);
constexpr unsigned MIR_OFF = FLAG_OFF + (648u << 10);
constexpr unsigned YH_REL = 4096;
constexpr unsigned AV_OFF = FLAG_OFF + (640u << 10);
constexpr unsigned DBG_OFF = FLAG_OFF + 65536;
constexpr unsigned OWN_OFF = FLAG_OFF + (6u << 20);
constexpr unsigned OWN_PAR = 8u << 20;
constexpr unsigned YST_OFF = OWN_OFF + 2u * OWN_PAR;
constexpr unsigned WS_USED = YST_OFF + (8u << 20);
constexpr unsigned OUT_BYTES = 512u * 4096u * 4u;
constexpr unsigned IN_BYTES = 512u * 4096u * 4u;
constexpr unsigned long long DEADLINE = 50000000ull;
constexpr int AUX_SYS = 17;
#define AUX_LD 16
constexpr int XR = 17;
constexpr int NTHREADS = 640;
constexpr int NT = (64 + NCLS - 1) / NCLS;
constexpr unsigned RC = RG / NCLS * NCLS;
constexpr unsigned NL = 2u * L3;
constexpr unsigned LH = L3 / 2u;
static_assert(L3 % 2u == 0u, "");
constexpr unsigned CPX = 4u * NCLS;

struct Args {
  const float* x;
  const float* w;
  char* myws;
  unsigned rank;
  unsigned pad_;
  char* wsd[NR];
  float* out;
  unsigned tab[128];
};

struct __attribute__((aligned(128))) Slot {
  float d[32 * 32];
  unsigned st, pad_[31];
};
struct __attribute__((aligned(128))) Smem {
  u4 xl[8][32 - XR][64];
  Slot slot[8];
  unsigned st[8];
  unsigned abort_;
  unsigned tick, pad_[2];
  u4 fifo[8];
  u4 fifo2[8];
};

__device__ __forceinline__ rsrc_t mk(const void* p, unsigned bytes) {
  return __builtin_amdgcn_make_buffer_rsrc((void*)p, (short)0, (int)bytes, 0x00020000);
}
__device__ __forceinline__ unsigned lds_ld(const unsigned* p) {
  return __builtin_amdgcn_readfirstlane(__hip_atomic_load(p, __ATOMIC_RELAXED, __HIP_MEMORY_SCOPE_WORKGROUP));
}
__device__ __forceinline__ void lds_st(unsigned* p, unsigned v) {
  __hip_atomic_store(p, v, __ATOMIC_RELAXED, __HIP_MEMORY_SCOPE_WORKGROUP);
}
__device__ __forceinline__ unsigned lds_addr(const void* p) { return (unsigned)(size_t)(const __attribute__((address_space(3))) void*)p; }
__device__ __forceinline__ unsigned poll1(const char* p, unsigned off) {
  unsigned v;
  asm volatile("s_load_dword %0, %1, %2 glc\n\ts_waitcnt lgkmcnt(0)" : "=s"(v) : "s"(p), "s"(off) : "memory");
  return v;
}
__device__ __forceinline__ u4 poll4(const char* p, unsigned off, unsigned off2, unsigned& v2) {
  u4 v;
  asm volatile("s_load_dwordx4 %0, %2, %3 glc\n\ts_load_dword %1, %2, %4 glc\n\ts_waitcnt lgkmcnt(0)" : "=&s"(v), "=&s"(v2) : "s"(p), "s"(off), "s"(off2) : "memory");
  return v;
}
__device__ __forceinline__ unsigned long long now() { return __builtin_amdgcn_s_memrealtime(); }

#define STAMP(slot) do {} while (0)

__device__ __forceinline__ unsigned wave_code(unsigned m) {
  const unsigned c = ((m >> 23) + 1u) >> 4;
  unsigned r = 0;
#pragma unroll
  for (unsigned b = 8; b; b >>= 1)
    if (__builtin_amdgcn_ballot_w64(c >= (r | b)) != 0ull) r |= b;
  return r;
}
__device__ __forceinline__ unsigned tag_of(unsigned call) { return (call % 15u + 1u) << 4; }
__device__ __forceinline__ unsigned wflag(unsigned g, unsigned sx) { const unsigned t = (g * 43u) >> 8; return ((g - t * NCLS) * 8u + sx) * 128u + t * 4u; }

#define WG_BARRIER() asm volatile("s_waitcnt lgkmcnt(0)\n\ts_barrier" ::: "memory")

__device__ __forceinline__ unsigned next_call(unsigned raw) {
  unsigned call = __builtin_amdgcn_readfirstlane(raw) + 1u;
  if (call > 0xfffffeu) call = 1u;
  asm volatile("" : "+s"(call) :: "memory");
  return call;
}
__device__ __forceinline__ char* lane_ws(unsigned r, unsigned rank) {
  typedef char* const __attribute__((address_space(4))) * kp_t;
  return ((kp_t)((const __attribute__((address_space(4))) char*)__builtin_amdgcn_kernarg_segment_ptr() + offsetof(Args, wsd)))[(r - rank - 1u) & 7u];
}
__device__ __forceinline__ void flag_st(char* p, unsigned v) {
  asm volatile("global_store_byte %0, %1, off sc0 sc1" :: "v"(p), "v"(v) : "memory");
}

__host__ __device__ __forceinline__ unsigned piece_src(bool isx, unsigned G, unsigned q) { return isx ? G * 65536u + q * 512u : q * 128u * 16384u + G * 128u; }
__device__ __forceinline__ int lane_src(bool isx, int lane) { return isx ? (lane >> 4) * 2048 + (lane & 15) * 16 : (lane >> 3) * 4 * 16384 + (lane & 7) * 16; }
__device__ __forceinline__ void load_piece(f4 (&v)[16], const float* px, const float* pw, bool isx, unsigned so, int vo) {
  const rsrc_t rs = mk(isx ? px : pw, IN_BYTES);
  const unsigned s0 = isx ? 256u : 16384u, s1 = isx ? 8192u : 32768u, s2 = isx ? 16384u : 64u * 16384u, s3 = isx ? 32768u : 32u * 16384u;
#pragma unroll
  for (int f = 0; f < 16; ++f) {
    v[f] = __builtin_bit_cast(f4, __builtin_amdgcn_raw_buffer_load_b128(rs, vo, so + (f & 1) * s0 + ((f >> 1) & 1) * s1 + ((f >> 2) & 1) * s2 + (f >> 3) * s3, 0));
    __builtin_amdgcn_sched_barrier(0);
  }
}
__device__ __forceinline__ unsigned conv_piece(f4 (&v)[16], bool isx, u4* tr, int lane) {
  float m = 0.f, m2 = 0.f;
#pragma unroll
  for (int f = 0; f < 16; ++f) {
    asm("v_max3_f32 %0, %1, |%2|, |%3|" : "=v"(m) : "v"(m), "v"(v[f][0]), "v"(v[f][1]));
    asm("v_max3_f32 %0, %1, |%2|, |%3|" : "=v"(m2) : "v"(m2), "v"(v[f][2]), "v"(v[f][3]));
  }
  const unsigned code = wave_code(max(__builtin_bit_cast(unsigned, m), __builtin_bit_cast(unsigned, m2)));
  const float scl = __builtin_bit_cast(float, (127u + 15u + 112u - code * 16u) << 23);
#pragma unroll
  for (int f = 0; f < 16; ++f) v[f] *= scl;
  if (isx) {
#pragma unroll
    for (int f = 0; f < 8; ++f) {
      h8 r;
#pragma unroll
      for (int j = 0; j < 8; ++j) r[j] = (_Float16)v[2 * f + (j >> 2)][j & 3];
      tr[(lane & 15) * 33 + 4 * f + (lane >> 4)] = __builtin_bit_cast(u4, r);
    }
  } else {
#pragma unroll
    for (int f = 0; f < 8; ++f) {
      h8 r;
#pragma unroll
      for (int j = 0; j < 8; ++j) r[j] = (_Float16)v[(f >> 2) * 8 + j][f & 3];
      tr[((f >> 2) * 8 + (lane >> 3)) * 33 + 4 * (lane & 7) + (f & 3)] = __builtin_bit_cast(u4, r);
    }
  }
  asm volatile("" ::: "memory");
  return code;
}
__host__ __device__ __forceinline__ unsigned xgroup(unsigned d, unsigned rg) {
  return rg < 16u ? 16u * d + rg : 16u * (((d & 4u) ^ 4u) + ((rg >> 2) & 3u)) + 4u * (d & 3u) + (rg & 3u);
}

__host__ __device__ __forceinline__ unsigned xitem(unsigned d, unsigned j) {
  return ((j & 4u) ? 16u + 4u * ((d - (j >> 5)) & 3u) : 4u * ((d + (j >> 5)) & 3u)) + ((j >> 3) & 3u);
}

__global__ void __launch_bounds__(NTHREADS) fused_gemm_kernel(Args a) {
  __shared__ Smem sm;
#define STAMP_ENTRY() do {} while (0)
  const int tid = threadIdx.x;
  const int lane = tid & 63;
  const int wv = __builtin_amdgcn_readfirstlane(tid >> 6);
  const unsigned b = blockIdx.x;
  const unsigned xc = b & 7u, jb = (b >> 3) & 31u;
  typedef unsigned u8v __attribute__((ext_vector_type(8)));
  u8v a0;
  char* pd;
  unsigned long long tb2;
  asm volatile("s_load_dwordx8 %0, %2, 0\n\ts_load_dwordx2 %1, %2, %3 offset:%4" : "=&s"(a0), "=&s"(tb2)
               : "s"(__builtin_amdgcn_kernarg_segment_ptr()), "s"((b << 3) & 0x1f8u), "n"(offsetof(Args, tab)));
  __builtin_amdgcn_sched_barrier(0);
  asm volatile("s_load_dwordx2 %0, %1, %2 offset:%3" : "=&s"(pd) : "s"(__builtin_amdgcn_kernarg_segment_ptr()), "s"(b & 56u), "n"(offsetof(Args, wsd)));
#define XG(c) ((((c) >> 2) & 7u) * 16u + ((((c) >> 2) + ((c) >> 7)) & 3u) * 4u + (((c) >> 5) & 3u))
  const bool lk = jb < 7u;
  unsigned so0 = piece_src(true, XG(b - 64u), b & 3u), mk0 = b - 56u;
  asm volatile("" : "+s"(mk0));
  mk0 = (unsigned)((int)mk0 >> 8);
  so0 &= ~mk0;
  unsigned co = CTR_OFF + b * 64u + ((unsigned)wv & mk0) * 4u;
  unsigned first = (((24u + jb) >> 5) & 1u) - 2u * mk0;
  int vox = lane_src(true, lane), vow = lane_src(false, lane);
  asm volatile("" : "+s"(so0), "+s"(co), "+s"(first));
  asm volatile("" : "+v"(vox), "+v"(vow));
  unsigned sh0 = wv == 1 ? 32u : 0u;
  if (lk && wv == 1) asm volatile("s_barrier" ::: "memory");
  if (!lk) __builtin_amdgcn_s_sleep(D0);
  asm volatile("" : "+s"(first), "+s"(sh0));
  asm volatile("s_waitcnt lgkmcnt(0)" : "+s"(a0), "+s"(pd), "+s"(tb2));
  const float* const px = (const float*)(((size_t)a0[1] << 32) | a0[0]);
  const float* const pw = (const float*)(((size_t)a0[3] << 32) | a0[2]);
  char* const myws = (char*)(__attribute__((address_space(1))) char*)(((size_t)a0[5] << 32) | a0[4]);
  const unsigned rank = a0[6] & 7u;
  const unsigned h = rank >> 2;
  const rsrc_t rsMy = mk(myws, WS_USED);
  unsigned* const tmo = (unsigned*)(myws + TMO_OFF);
#define TIMED_OUT() (now() - t0 > DEADLINE || __hip_atomic_load(tmo, __ATOMIC_RELAXED, __HIP_MEMORY_SCOPE_AGENT) == call)
#define SET_TIMEOUT() __hip_atomic_store(tmo, call, __ATOMIC_RELAXED, __HIP_MEMORY_SCOPE_AGENT)
  auto publish = [&](const u4* tr, unsigned so, char* fp, bool on, unsigned fv) {
    u4 o[8];
#pragma unroll
    for (int f = 0; f < 8; ++f) o[f] = tr[66 * f + (lane >> 5) * 33 + (lane & 31)];
    __builtin_amdgcn_sched_barrier(0);
#pragma unroll
    for (int f = 0; f < 8; ++f) __builtin_amdgcn_raw_buffer_store_b128(o[f], rsMy, lane * 16, so + f * 1024, AUX_SYS);
    asm volatile("s_waitcnt vmcnt(0)" : "+v"(fv) :: "memory");
    if (on) flag_st(fp, fv);
  };
  auto xflag = [&](unsigned G, unsigned q, bool direct) {
    const unsigned o = G >> 4, hp = ((o & 4u) ^ 4u) + ((G >> 2) & 3u);
    const unsigned lo = G & 15u, lh = 16u + (o & 3u) * 4u + (G & 3u);
    const unsigned cr = (lane & 1) ? hp : o, cl = (lane & 1) ? lh : lo;
    unsigned fo = lane < 2 ? MIR_OFF + cr * 128u + cl * 4u + q : AD_OFF + rank * 2u * FLAG_PAR + (cl * 4u + q) * 128u;
    if (lane >= 2 && (!direct || o == rank || cr == rank)) fo = 0u;
    if (lane >= 4) fo = lane < 4 + NREP && (o == rank || hp == rank) ? FL_OFF + (unsigned)(lane - 4) * REP_STRIDE + FX_REL + (o == rank ? lo : lh) * 128u + q : 0u;
    return fo;
  };
  unsigned call;

  if (__builtin_expect((unsigned)wv < first, 1)) {
    static_assert(NW == 2 && NW + 2 + NF <= 10, "hop pieces are pushed");
    const unsigned tb0 = (unsigned)(tb2 >> sh0) & mk0;
    const bool isx = (int)tb0 >= 0;
    f4 v[16];
    const unsigned ctr_raw = __builtin_amdgcn_raw_buffer_load_b32(rsMy, 0, co, AUX_SYS);
    __builtin_amdgcn_sched_barrier(0);
    load_piece(v, px, pw, isx, (tb0 & 0xffffffu) | so0, isx ? vox : vow);
    asm volatile("" ::: "memory");
    if (lk && wv != 1) asm volatile("s_barrier" ::: "memory");
    unsigned e0, e1 = 0u, e2 = 0u, so = OWN_OFF + ((so0 >> 16) * 4u + (b & 3u)) * PIECE;
    char* fp = myws;
    if (lk) {
      const unsigned p = (rank + 1u + jb) & 7u, idx = (tb0 >> 24) & 63u, q = isx ? b & 3u : 0u;
      e0 = isx ? 1u : HF_OFF + (p & 3u) * 256u + idx;
      e1 = (rank ^ p) * SRC_STRIDE + (isx ? 0u : W_OFF) + (idx * 4u + q) * PIECE;
      e2 = FL_OFF + (isx ? FX_REL + idx * 128u + (rank ^ p) * 4u : wflag(idx, rank ^ p)) + q;
    } else {
      const unsigned G = so0 >> 16;
      char* const po = a.wsd[((G >> 4) - rank - 1u) & 7u];
      char* const ph = a.wsd[(((G >> 4) & 4u) + ((G >> 2) & 3u) - rank + 3u) & 7u];
      e0 = xflag(G, b & 3u, true);
      fp = (lane == 2 ? po : lane == 3 ? ph : myws) + e0;
    }
    asm volatile("" : "+s"(so) :: "memory");
    asm volatile("" : "+v"(e0), "+v"(e1), "+v"(e2), "+v"(fp) :: "memory");
    call = next_call(ctr_raw);
    const unsigned par = call & 1u;
    unsigned e3 = tag_of(call);
    so += par * OWN_PAR; e1 += par * PAR_STRIDE; e2 += par * FLAG_PAR; fp += par * FLAG_PAR;
    asm volatile("" : "+s"(so), "+s"(e3) :: "memory");
    asm volatile("" : "+v"(e1), "+v"(e2), "+v"(fp) :: "memory");
    e3 |= conv_piece(v, isx, (u4*)&sm.xl[wv][0][0], lane);
    if (lk) {
      if (lane == 0) sm.fifo[wv] = (u4){e0, e1, e2, e3};
      asm volatile("" ::: "memory");
      STAMP(16 + 4 * wv);
      STAMP_ENTRY();
      if (lane == 0) __builtin_amdgcn_raw_buffer_store_b32(call, rsMy, 0, co, AUX_SYS);
      return;
    }
    publish((const u4*)&sm.xl[0][0][0], so, fp, e0 != 0u, e3);
    STAMP(24);
  }
  if (jb < 7u && wv >= NW + 2 && wv < NW + 2 + NF) {
    const unsigned p = (rank + 1u + jb) & 7u;
    const bool cross = ((p ^ rank) & 4u) != 0u;
    const unsigned nhop = cross ? 16u : 0u;
    const unsigned fi = (unsigned)wv - (NW + 2);
    asm volatile("s_barrier" ::: "memory");
    const unsigned ctr_raw = __builtin_amdgcn_raw_buffer_load_b32(rsMy, 0, co, AUX_SYS);
    STAMP_ENTRY();
    const unsigned long long t0 = now();
    call = next_call(ctr_raw);
    if (lane == 0) __builtin_amdgcn_raw_buffer_store_b32(call, rsMy, 0, co, AUX_SYS);
    const unsigned nin = cross ? 384u - RC : 384u + RG;
    const unsigned nx = cross ? 128u : 128u - 4u * NL, nl = cross ? nin : nin - 4u * NL;
    const unsigned nitems = nin + ((cross && !(EXP & 1)) ? 128u : 0u);
    const unsigned dbase = (call & 1u) * PAR_STRIDE;
    const unsigned frel = (call & 1u) * FLAG_PAR;
    const unsigned obase = OWN_OFF + (call & 1u) * OWN_PAR;
    const unsigned hfb = HF_OFF + (call & 1u) * 1024u + (p & 3u) * 256u;
    const unsigned tag = tag_of(call);
    unsigned posted = 0, spins = 0;
    unsigned i = (NW + fi) * 8u + xc;
    __builtin_amdgcn_s_sleep(FDELAY);
    while (i < nitems) {
      unsigned so, dof, fo = 0, fv = 0, isy = 0;
      bool bad = false;
      if (i >= nin) {
        const unsigned j = i - nin;
        so = YST_OFF + ((call & 1u) * 512u + (rank & 3u) * 128u + j) * PIECE;
        dof = (128u * (p & 3u) + 32u * (j & 3u)) * 16384u + (512u * (p & 4u) + 64u * (j >> 2)) * 4u;
        isy = 32u;
        while (true) {
          unsigned hv;
          const u4 yv = poll4(myws, YRDY_OFF + ((call & 1u) * 512u + (p & 3u) * 128u + j) * 16u, AV_OFF + YH_REL + frel + (p & 3u) * 256u + (j >> 1) * 4u, hv);
          if ((j < JD ? ((hv >> (16u * (j & 1u))) & 0xffffu) == tag * 0x101u : yv[0] == call && yv[1] == call) && yv[2] == call && yv[3] == call) break;
          if ((++spins & 63u) == 0u && TIMED_OUT()) { SET_TIMEOUT(); bad = true; break; }
          __builtin_amdgcn_s_sleep(YSLEEP);
          asm volatile("" ::: "memory");
        }
        if (bad && lane == 0) __builtin_amdgcn_raw_buffer_store_b32(0x7fc00000u, mk(a.out, OUT_BYTES), 0, dof, AUX_SYS);
      } else {
        unsigned idx, q;
        const bool isx = i < nx || i >= nl;
        bool fwd = false, direct = false;
        const unsigned lt = (i - nl + LROT * (p ^ rank)) & 15u;
        if (isx) {
          unsigned j = i - nhop;
          if (i >= nl) j = 8u * (16u - NL + ((i - nl) >> 4) * 4u + (lt & 3u)) + (lt >> 2);
          else if (!cross && j >= 8u * (16u - NL)) j = 2u * (j & ~3u) - 8u * (16u - NL) + 4u + (j & 3u);
          else if (cross) {
            const unsigned t = j - 32u * ((rank - p) & 3u);
            if ((int)t >= 0) j += t < 16u ? t & 12u : 16u;
          }
          idx = xitem(rank, j); q = j & 3u;
          direct = j < 48u;
        } else {
          unsigned j = i - nx;
          const unsigned n1 = cross ? RC * 3u : RG;
          const bool rel = j < n1;
          if (!rel) j -= n1;
          const unsigned pe = rel ? 3u : 4u;
          const unsigned r = (j * (rel ? 3641u : 2731u)) >> 16, rem = j - r * (NCLS * pe);
          const unsigned g0 = r * NCLS + (!rel && cross ? RC : 0u);
          const unsigned ng = min((unsigned)NCLS, 64u - g0);
          const unsigned k = (rem * (ng == 6u ? 10923u : 16384u)) >> 16;
          fwd = rel && !cross;
          idx = fwd ? j : g0 + rem - k * ng;
          q = fwd ? 0u : k + 4u - pe;
          bad = cross && q == 0u && (idx < RG || ((idx - rank + p) & 3u) == 0u);
        }
        const unsigned src = fwd ? 4u * (h ^ 1u) + ((p - idx) & 3u) : p;
        const unsigned po = dbase + (isx ? 0u : W_OFF) + (idx * 4u + q) * PIECE;
        dof = po + (src ^ rank) * SRC_STRIDE;
        fo = frel + (isx ? FX_REL + idx * 128u + (src ^ rank) * 4u : wflag(idx, src ^ rank)) + q;
        if (i >= nl) {
          const unsigned hq = (lt & 3u) * 4u + q, ij = ((i - nl) >> 4) * 4u + (p ^ rank);
          dof = dbase + (hq * 4u * LH + ij) * PIECE;
          fo = frel + FZ_REL + hq * 128u + ij;
        }
        so = fwd ? po + (src ^ p) * SRC_STRIDE : obase + ((isx ? xgroup(rank, idx) : 128u + 64u * h + idx) * 4u + q) * PIECE;
        const unsigned ao = fwd ? hfb + idx : frel + (direct ? AD_OFF + p * 2u * FLAG_PAR + (idx * 4u + q) * 128u : AV_OFF + p * 512u + (isx ? idx : 32u + idx) * 4u + q);
        while (!bad) {
          fv = (poll1(myws, ao & ~3u) >> (8u * (ao & 3u))) & 255u;
          if ((fv & 0xf0u) == tag) break;
          if ((++spins & 63u) == 0u && TIMED_OUT()) { SET_TIMEOUT(); bad = true; break; }
          __builtin_amdgcn_s_sleep(FSLEEP);
          asm volatile("" ::: "memory");
        }
      }
      if (!bad) {
        unsigned tk = 0;
        if (lane == 0) tk = atomicAdd(&sm.tick, 1u);
        tk = __builtin_amdgcn_readfirstlane(tk);
        const u4 e = {((tk + 1u) << 24) | (fo << 6) | isy | fi, so, dof, fv};
        asm volatile("" ::: "memory");
        if (lane == 0) sm.fifo2[tk & 7u] = e;
        asm volatile("" ::: "memory");
        if (tk + NW < 60) STAMP(16 + 4 * (tk + NW));
        ++posted;
        while (lds_ld(&sm.st[fi]) != posted) __builtin_amdgcn_s_sleep(1);
        asm volatile("" ::: "memory");
      }
      i = 0;
      if (lane == 0) i = __hip_atomic_fetch_add((unsigned*)(myws + QUEUE_OFF + (call & 1u) * 1024u + jb * 128u), 1u, __ATOMIC_RELAXED, __HIP_MEMORY_SCOPE_SYSTEM);
      i = __builtin_amdgcn_readfirstlane(i) + (NW + NF) * 8u;
    }
    if (lane == 0) {
      const unsigned tk = atomicAdd(&sm.tick, 1u);
      sm.fifo2[tk & 7u] = (u4){((tk + 1u) << 24) | 16u, 0u, 0u, 0u};
    }
    return;
  }
  if (jb < 7u && wv == NW) {
    __builtin_amdgcn_s_waitcnt(0x0f70);
    if (lane < 8) { sm.fifo[lane] = (u4){0u, 0u, 0u, 0u}; sm.fifo2[lane] = (u4){0u, 0u, 0u, 0u}; sm.st[lane] = 0u; }
    if (lane == 0) sm.tick = 0u;
    WG_BARRIER();
    __builtin_amdgcn_s_setprio(3);
    STAMP_ENTRY();
    const rsrc_t rsD = mk(pd, WS_USED);
    const int lp = (lane >> 5) * 33 + (lane & 31);
    const unsigned p = (rank + 1u + jb) & 7u;
    char* const fws = lane_ws((p & 4u) | ((p + (unsigned)lane) & 3u), rank);
    unsigned pfo = TMO_OFF + 64u, pfv = 0;
    bool pfl = false;
    for (unsigned n = 0; n < NW; ++n) {
      u4 e;
      while (true) {
        asm volatile("" ::: "memory");
        e = sm.fifo[n];
        if (__builtin_amdgcn_readfirstlane(e[0]) != 0u) break;
      }
      asm volatile("" ::: "memory");
      const u4* const tr = (const u4*)&sm.xl[n][0][0];
      u4 o[8];
#pragma unroll
      for (int f = 0; f < 8; ++f) o[f] = tr[66 * f + lp];
      const unsigned so = __builtin_amdgcn_readfirstlane(e[1]);
#pragma unroll
      for (int f = 0; f < 8; ++f) {
        asm volatile("s_waitcnt vmcnt(%0)" :: "n"(((f + 8 - (WIN - 1)) % 8 != 0 && (f + 8 - (WIN - 1)) % 8 < WIN - 1) ? WIN : WIN - 1) : "memory");
        if (f == WIN - 1 && n != 0) { if (pfl) flag_st(fws + pfo, pfv); }
        __builtin_amdgcn_raw_buffer_store_b128(o[f], rsD, lane * 16, so + f * 1024, AUX_SYS);
        asm volatile("" ::: "memory");
      }
      pfo = (lane & 3) == 0 ? e[2] + (unsigned)(lane >> 2) * REP_STRIDE : e[0] + (so >= PAR_STRIDE ? 1024u : 0u);
      pfv = e[3];
      pfl = (lane & 3) == 0 ? lane < 4 * NREP : (lane < 4 && e[0] != 1u);
    }
    asm volatile("s_waitcnt vmcnt(0)" ::: "memory");
    if (pfl) flag_st(fws + pfo, pfv);
    STAMP(12);
    return;
  }
  if (jb < 7u && wv == NW + 1) {
    __builtin_amdgcn_s_waitcnt(0x0f70);
    asm volatile("s_barrier" ::: "memory");
    __builtin_amdgcn_s_setprio(3);
    STAMP_ENTRY();
    const rsrc_t rsP = mk(pd, WS_USED);
    const rsrc_t rsO = mk(a.out, OUT_BYTES);
    const int voy = (lane >> 4) * 16384 + (lane & 15) * 16;
    const int vol = lane * 16;
    const unsigned fvo = lane < NREP ? (unsigned)lane * REP_STRIDE : WS_USED;
    unsigned d1 = WS_USED, y1 = 0, fo1 = WS_USED, fv1 = 0, fo2 = WS_USED, fv2 = 0, ends = 0, n = 0;
    bool ym = false;
    u4 e = {0u, 0u, 0u, 0u};
    while (true) {
      unsigned e0 = __builtin_amdgcn_readfirstlane(e[0]);
      if ((e0 >> 24) != ((n + 1u) & 255u)) {
        asm volatile("" ::: "memory");
        e = sm.fifo2[n & 7u];
        e0 = __builtin_amdgcn_readfirstlane(e[0]);
      }
      unsigned so = WS_USED, d0 = WS_USED, y0 = 0, fo0 = WS_USED, fv0 = 0;
      if ((e0 >> 24) == ((n + 1u) & 255u)) {
        ++n;
        if ((e0 & 16u) != 0u) { ++ends; continue; }
        so = __builtin_amdgcn_readfirstlane(e[1]);
        d0 = __builtin_amdgcn_readfirstlane(e[2]);
        fv0 = __builtin_amdgcn_readfirstlane(e[3]);
        y0 = e0 & 32u;
        fo0 = y0 ? WS_USED : FL_OFF + ((e0 >> 6) & 0x7fffu);
        asm volatile("" ::: "memory");
        if (lane == 0) __hip_atomic_fetch_add(&sm.st[e0 & 7u], 1u, __ATOMIC_RELAXED, __HIP_MEMORY_SCOPE_WORKGROUP);
      } else if (d1 == WS_USED && fo2 == WS_USED) {
        if (ends == NF) break;
        __builtin_amdgcn_s_sleep(1);
        continue;
      }
      const rsrc_t rs = y1 ? rsO : rsMy;
      const int vo = y1 ? voy : vol;
      const unsigned st = y1 ? 4u * 16384u : 1024u;
      if (y0 != 0u && !ym) { ym = true; asm volatile("s_waitcnt vmcnt(0)" ::: "memory"); }
      if (ym) {
        unsigned da = d1, sb = so + 4096u, pa = y0 ? d0 : WS_USED;
#define YSTEP(F, W, FL, SO, OFS) "s_waitcnt vmcnt(" W ")\n\t" FL "s_cbranch_vccz Lpy" #F "_%=\n\t" \
        "v_pk_add_f32 v[136+4*" #F ":137+4*" #F "], v[136+4*" #F ":137+4*" #F "], v[104+4*" #F ":105+4*" #F "]\n\t" \
        "v_pk_add_f32 v[138+4*" #F ":139+4*" #F "], v[138+4*" #F ":139+4*" #F "], v[106+4*" #F ":107+4*" #F "]\n" \
        "Lpy" #F "_%=:\n\tbuffer_store_dwordx4 v[136+4*" #F ":139+4*" #F "], %[vo], %[rs], %[da] offen sc0 sc1\n\t" \
        "buffer_load_dwordx4 v[104+4*" #F ":107+4*" #F "], %[voy], %[ro], %[pa] offen sc0 sc1\n\t" \
        "buffer_load_dwordx4 v[136+4*" #F ":139+4*" #F "], %[vol], %[rp], %[" SO "] offen offset:" #OFS " sc0 sc1\n\t" \
        "s_add_u32 %[da], %[da], %[st]\n\ts_add_u32 %[pa], %[pa], 0x10000\n\t"
#define YOPS : [da] "+s"(da), [pa] "+s"(pa) \
          : [vo] "v"(vo), [vol] "v"(vol), [voy] "v"(voy), [fv] "v"(fv2), [fvo] "v"(fvo), [rs] "s"(rs), [rp] "s"(rsP), [rm] "s"(rsMy), [ro] "s"(rsO), \
            [sa] "s"(so), [sb] "s"(sb), [st] "s"(st), [fo] "s"(fo2), [y1] "s"(y1) : "memory", "scc", "vcc", "v104", "v167"
        asm volatile("s_cmp_lg_u32 %[y1], 0\n\ts_cselect_b64 vcc, -1, 0\n\ts_nop 4\n\t"
          YSTEP(0, "22", "", "sa", 0) YSTEP(1, "22", "", "sa", 1024) YSTEP(2, "22", "", "sa", 2048) YSTEP(3, "22", "", "sa", 3072)
          YSTEP(4, "22", "", "sb", 0) YSTEP(5, "22", "", "sb", 1024) YSTEP(6, "22", "", "sb", 2048) YOPS);
        e = sm.fifo2[n & 7u];
        asm volatile("s_cmp_lg_u32 %[y1], 0\n\ts_cselect_b64 vcc, -1, 0\n\ts_nop 4\n\t"
          YSTEP(7, "21", "buffer_store_byte %[fv], %[fvo], %[rm], %[fo] offen sc0 sc1\n\t", "sb", 3072) YOPS);
      } else {
        unsigned da = d1, sb = so + 4096u;
#define PSTEP(F, R, SO, OFS) "s_waitcnt vmcnt(%[w" #F "])\n\t" PFLAG##F \
        "buffer_store_dwordx4 " R ", %[vo], %[rs], %[da] offen sc0 sc1\n\t" \
        "buffer_load_dwordx4 " R ", %[vol], %[rp], %[" SO "] offen offset:" #OFS " sc0 sc1\n\t" \
        "s_add_u32 %[da], %[da], %[st]\n\t"
#define PFLAG0
#define PFLAG1
#define PFLAG2
#define PFLAG3
#define PFLAG4
#define PFLAG5
#define PFLAG6
#define PFLAG7 "buffer_store_byte %[fv], %[fvo], %[rm], %[fo] offen sc0 sc1\n\t"
#define PWAIT(F) [w##F] "n"(14 - 2 * (8 - PW) + (F + 8 - PW < 7 ? 1 : 0))
#define POPS : [da] "+s"(da) \
          : [vo] "v"(vo), [vol] "v"(vol), [fv] "v"(fv2), [fvo] "v"(fvo), [rs] "s"(rs), [rp] "s"(rsP), [rm] "s"(rsMy), \
            [sa] "s"(so), [sb] "s"(sb), [st] "s"(st), [fo] "s"(fo2), \
            PWAIT(0), PWAIT(1), PWAIT(2), PWAIT(3), PWAIT(4), PWAIT(5), PWAIT(6), PWAIT(7) \
          : "memory", "scc", "v136", "v137", "v138", "v139", "v140", "v141", "v142", "v143", "v144", "v145", "v146", "v147", "v148", "v149", "v150", "v151", \
            "v152", "v153", "v154", "v155", "v156", "v157", "v158", "v159", "v160", "v161", "v162", "v163", "v164", "v165", "v166", "v167"
        asm volatile("s_nop 4\n\t"
          PSTEP(0, "v[136:139]", "sa", 0) PSTEP(1, "v[140:143]", "sa", 1024) PSTEP(2, "v[144:147]", "sa", 2048) PSTEP(3, "v[148:151]", "sa", 3072)
          PSTEP(4, "v[152:155]", "sb", 0) PSTEP(5, "v[156:159]", "sb", 1024) PSTEP(6, "v[160:163]", "sb", 2048) POPS);
        e = sm.fifo2[n & 7u];
        asm volatile(PSTEP(7, "v[164:167]", "sb", 3072) POPS);
      }
      fo2 = fo1; fv2 = fv1;
      d1 = d0; y1 = y0; fo1 = fo0; fv1 = fv0;
    }
    asm volatile("s_waitcnt vmcnt(0)" ::: "memory");
    return;
  }
  STAMP_ENTRY();
  const unsigned long long t0 = now();
  if (jb < 8u) {
    if (jb == 7u && wv == 0) {
      call = next_call(__builtin_amdgcn_raw_buffer_load_b32(rsMy, 0, co, AUX_SYS));
      if (lane == 0) __builtin_amdgcn_raw_buffer_store_b32(call, rsMy, 0, co, AUX_SYS);
      if (xc == 7u) {
        if (lane < 8) __builtin_amdgcn_raw_buffer_store_b32(0u, rsMy, lane * 128, QUEUE_OFF + ((call & 1u) ^ 1u) * 1024u, AUX_SYS);
        return;
      }
      const unsigned p = (rank + 1u + xc) & 7u, l = (unsigned)lane, frel = (call & 1u) * FLAG_PAR;
      const rsrc_t rsD = mk(a.wsd[xc], WS_USED);
      const bool on = l < 24u + (((p ^ rank) & 4u) != 0u && !(EXP & 1) ? JD / 8u : 0u);
      const unsigned so = !on ? WS_USED : MIR_OFF + l * 16u + (l < 8u ? p * 128u : l < 24u ? 896u + (p >> 2) * 256u : 1664u + (p & 3u) * 256u);
      const unsigned dof = AV_OFF + l * 16u + (l < 24u ? rank * 512u : YH_REL - 384u + (rank & 3u) * 256u);
      const unsigned tag4 = tag_of(call) * 0x01010101u;
      u4 last = {0u, 0u, 0u, 0u};
      __builtin_amdgcn_s_sleep(127);
      while (true) {
        const u4 cur = __builtin_amdgcn_raw_buffer_load_b128(rsMy, so, frel, AUX_SYS);
        const u4 df = cur ^ last, dt = cur ^ tag4;
        if ((df[0] | df[1] | df[2] | df[3]) != 0u) __builtin_amdgcn_raw_buffer_store_b128(cur, rsD, dof, frel, AUX_SYS);
        last = cur;
        if (__builtin_amdgcn_ballot_w64(on && ((dt[0] | dt[1] | dt[2] | dt[3]) & 0xf0f0f0f0u) != 0u) == 0ull) break;
        if (TIMED_OUT()) { SET_TIMEOUT(); break; }
        __builtin_amdgcn_s_sleep(FWS);
      }
      STAMP(15);
    }
    return;
  }

  const unsigned cidx = xc * CPX + (jb - 8u);
  const unsigned cls = cidx >> 5;
  const unsigned rg = ((cidx & 1u) << 4) | ((cidx & 31u) >> 1);
  static_assert(NCLS == 6, "");
  const int nt = cidx < 128u ? NT : NT - 1;
  if (wv == 0) {
    asm volatile("s_barrier" ::: "memory");
  } else {
    const unsigned ctr_raw = __builtin_amdgcn_raw_buffer_load_b32(rsMy, 0, co, AUX_SYS);
    if (wv == 1 && lane < 8) sm.slot[lane].st = 0u;
    if (tid == 64) sm.abort_ = 0u;
    const unsigned c = (unsigned)wv * 192u + (b - 64u);
    const bool isx = c < 512u;
    const unsigned cg = c < 960u ? ((c - 512u) * 9363u) >> 16 : (c - 960u + RG) & 63u, cr = c < 960u ? c - 512u - cg * 7u : 7u;
    const unsigned G = isx ? XG(c) : (cr < 4u ? h : h ^ 1u) * 64u + cg;
    const unsigned q = isx ? c & 3u : cr < 4u ? cr : (cr - 3u) & 3u;
    const unsigned wf = wflag(G & 63u, 0u);
    const unsigned fo = isx ? xflag(G, q, false) : lane == 0 ? MIR_OFF + 1024u + G * 4u + q :
                        lane > NREP ? 0u : FL_OFF + (unsigned)(lane - 1) * REP_STRIDE + (cr < 4u ? wf : FE_REL + (wf >> 10) * 128u + (wf & 127u)) + q;
    call = next_call(ctr_raw);
    WG_BARRIER();
    if (tid == NTHREADS - 64) __builtin_amdgcn_raw_buffer_store_b32(call, rsMy, 0, co, AUX_SYS);
    if (c < 1024u) {
      __builtin_amdgcn_s_sleep(CD0);
      for (int i = 1; i < wv; ++i) __builtin_amdgcn_s_sleep(CDELAY);
      f4 v[16];
      load_piece(v, px, pw, isx, piece_src(isx, G, q), isx ? vox : vow);
      u4* const tr = (u4*)&sm.xl[wv & 7][0][0];
      const unsigned code = conv_piece(v, isx, tr, lane);
      publish(tr, OWN_OFF + (call & 1u) * OWN_PAR + (((isx ? 0u : 128u) + G) * 4u + q) * PIECE, myws + (fo + (call & 1u) * FLAG_PAR), fo != 0u, tag_of(call) | code);
      STAMP(24 + wv);
    }
  }
  const unsigned dbase = (call & 1u) * PAR_STRIDE;
  const unsigned fbase = FL_OFF + (call & 1u) * FLAG_PAR + (cidx % NREP) * REP_STRIDE;
  const unsigned obase = OWN_OFF + (call & 1u) * OWN_PAR;
  if (EXP & 1) return;
#define LNN(x) ((x) > nt ? (x) - nt + NT : (x))
  const unsigned g2 = cidx < 128u ? cidx >> 1 : cidx - 128u, half = cidx >> 7;
  const unsigned lp = ((rg - 4u * rank) & 15u) - (16u - NL);
  const bool latecu = rg < 16u && lp < NL;
  const unsigned ve = rg < 16u ? 8u : 4u + ((rank ^ (rg >> 2)) & 3u);
  const int vo2 = (lane >> 3) * 16384 + ((lane ^ (lane >> 3)) & 7) * 16;

  if (wv < 8) {
    const unsigned v = (unsigned)wv;
    const unsigned tag4 = tag_of(call) * 0x01010101u;
    const unsigned lb = lds_addr(&sm.xl[v][0][0]);
    const rsrc_t rsX = mk(myws - lb, WS_USED + lb);
    const bool ew = v == ve;
    const unsigned rb = v == 0u || ew ? obase + xgroup(rank, ew ? rg - 16u : rg) * 4u * PIECE : dbase + v * SRC_STRIDE + rg * 4u * PIECE;
    const unsigned ab = (v == 0u || ew ? obase + (128u + 64u * (ew ? h ^ 1u : h)) * 4u * PIECE : dbase + v * SRC_STRIDE + W_OFF) + cls * 4u * PIECE;
    const unsigned og = (v >> 2) ^ 1u, hq = (half ? 2u + og : cidx & 1u) * 4u + (v & 3u);
    if (latecu && v - 1u < 3u && !half) return;
#define VR(B, I) "v[" B "+4*(" I "):" B "+4*(" I ")+3]"
#define LD1(R, S, F) "buffer_load_dwordx4 " R ", v1, %[rs], " S " offen offset:1024*" F " sc1\n\t"
#define LDI(X, F) LD1(VR(X, F), "s[74+" F "/4]", "(" F "&3)")
#define LDN(X, F)
#define LOAD8(X) LDI(X, "0") LDI(X, "1") LDI(X, "2") LDI(X, "3") LDI(X, "4") LDI(X, "5") LDI(X, "6") LDI(X, "7")
#define RES(Q, S, R7) LD1(VR("100", "4*" Q), S, "0") LD1(VR("100", "4*" Q "+1"), S, "1") LD1(VR("100", "4*" Q "+2"), S, "2") LD1(VR("100", "4*" Q "+3"), S, "3") \
      "s_addk_i32 " S ", 0x1000\n\t" LD1(VR("36", "4*" Q), S, "0") LD1(VR("36", "4*" Q "+1"), S, "1") LD1(VR("36", "4*" Q "+2"), S, "2") LD1(R7, S, "3")
#define XW(I) "ds_write_b128 v1, " VR("36", I) " offset:1024*" I "\n\t"
#define XRD(Q, I) "ds_read_b128 " VR("20", I) ", v1 offset:4096*" Q "+1024*" I "\n\t"
#define POLL "s_load_dwordx4 s[64:67], %[ws], s70 glc\n\t"
#define ZACC "v_mov_b64 v[4:5], 0\n\tv_mov_b64 v[6:7], 0\n\tv_mov_b64 v[8:9], 0\n\tv_mov_b64 v[10:11], 0\n\t" \
      "v_mov_b64 v[12:13], 0\n\tv_mov_b64 v[14:15], 0\n\tv_mov_b64 v[16:17], 0\n\tv_mov_b64 v[18:19], 0\n\t"
#define PEEK "s_xor_b64 s[80:81], s[64:65], s[98:99]\n\ts_quadmask_b64 s[80:81], s[80:81]\n\ts_andn2_b32 s80, s68, s80\n\t"
#define PICK(N, NONE) PEEK "s_cbranch_scc0 " NONE "\n\t" \
      "s_ff1_i32_b32 s80, s80\n\ts_lshl_b32 s74, s80, 12\n\ts_add_u32 s74, s74, s69\n\ts_add_u32 s75, s74, 0x1000\n\t" \
      "s_and_b32 s83, s80, 8\n\ts_cselect_b32 s81, s65, s64\n\ts_cselect_b32 s73, s96, 0\n\ts_add_u32 s73, s73, s72\n\ts_lshl_b32 s83, 0xff, s83\n\t" \
      "s_and_b32 s81, s81, 0x0f0f0f0f\n\ts_add_u32 s81, s81, s94\n\ts_lshl2_add_u32 s82, s80, 0x7fffc\n\ts_bfe_u32 s81, s81, s82\n\t" \
      "s_lshl_b32 s81, s81, 8\n\ts_or_b32 s73, s73, s80\n\ts_or_b32 s73, s73, s81\n\t" \
      "s_bitset0_b32 s68, s80\n\ts_and_b32 s83, s83, s68\n\ts_cbranch_scc0 La" N "%=\nLb" N "%=:\n\t"
#define DBGST(S, SH, O)
#define ADV(N) "La" N "%=:\n\t" DBGST("s73", "16", "0x4f0000") "s_bitset1_b32 s73, 7\n" \
      "Ld" N "%=:\n\ts_and_b32 s81, s68, 0xff\n\ts_cbranch_scc1 Lb" N "%=\n\t" \
      "s_mov_b32 s64, s65\n\ts_mov_b32 s65, s66\n\ts_mov_b32 s66, s67\n\ts_mov_b32 s67, 0\n\t" \
      "s_add_u32 s69, s69, s90\n\ts_add_u32 s70, s70, 4\n\ts_add_u32 s72, s72, s96\n\ts_lshr_b32 s68, s68, 8\n\ts_sub_u32 s71, s71, 1\n\t" \
      "s_cmp_gt_i32 s71, 0\n\ts_cselect_b32 s81, s91, 0\n\ts_or_b32 s68, s68, s81\n\ts_cbranch_scc1 Ld" N "%=\n\ts_branch Lb" N "%=\n"
#define MF(W, XA, X, F, LD) "s_waitcnt vmcnt(" W ")\n\tv_mfma_f32_32x32x16_f16 v[4:19], " XA ", " VR(X, F) ", v[4:19]\n\t" LD(X, F)
#define WL(F) "15"
#define WM(F) "15-" F
#define WS(F) "7-" F
#define BODY(X, Q, W, LD, MID, R3, X7, END) XRD(Q, "0") XRD(Q, "1") XRD(Q, "2") R3 "ds_read_b32 v3, v2 offset:4096\n\t" \
      MF(W("0"), VR("100", "4*" Q), X, "0", LD) MF(W("1"), VR("100", "4*" Q "+1"), X, "1", LD) MF(W("2"), VR("100", "4*" Q "+2"), X, "2", LD) MF(W("3"), VR("100", "4*" Q "+3"), X, "3", LD) \
      "s_waitcnt lgkmcnt(0)\n\t" MID \
      MF(W("4"), "v[20:23]", X, "4", LD) MF(W("5"), "v[24:27]", X, "5", LD) MF(W("6"), "v[28:31]", X, "6", LD) MF(W("7"), X7, X, "7", LD) \
      "s_branch " END "\n"
#define TREE(N, X, D, W, LD, MID, END) "s_bitcmp1_b32 " D ", 2\n\ts_cbranch_scc1 Lh" N "%=\n\ts_bitcmp1_b32 " D ", 1\n\ts_cbranch_scc1 Li" N "%=\n\t" \
      BODY(X, "0", W, LD, MID, XRD("0", "3"), "v[32:35]", END) "Li" N "%=:\n\t" BODY(X, "1", W, LD, MID, XRD("1", "3"), "v[32:35]", END) \
      "Lh" N "%=:\n\ts_bitcmp1_b32 " D ", 1\n\ts_cbranch_scc1 Lj" N "%=\n\t" \
      BODY(X, "2", W, LD, MID, XRD("2", "3"), "v[32:35]", END) "Lj" N "%=:\n\t" BODY(X, "3", W, LD, MID, "", "v[164:167]", END)
#define TAIL(D, NEW, NEXT) "s_add_u32 s79, s79, 1\n\ts_bitcmp1_b32 " D ", 7\n\ts_mov_b32 " D ", " NEW "\n\ts_cbranch_scc0 " NEXT "\n\ts_call_b64 s[84:85], Lflush%=\n\ts_branch " NEXT "\n"
#define STEP(X, Y, DX, DY) "Ls" X "%=:\n\t" \
      "s_lshr_b32 s80, " DX ", 8\n\ts_cmp_eq_u32 s79, 0\n\ts_cselect_b32 s78, s80, s78\n\ts_cmp_lg_u32 s78, s80\n\ts_cbranch_scc1 Lk" X "%=\n" \
      "Lp" X "%=:\n\t" PICK(X, "Ln" X "%=") \
      "s_cmp_eq_u32 " DY ", 0\n\ts_cbranch_scc1 Ly" X "%=\n\t" \
      TREE("l" X, X, DX, WL, LDI, "", "Lt" X "%=") \
      "Ly" X "%=:\n\t" LOAD8(Y) "s_mov_b32 " DY ", s73\n\ts_branch Lp" X "%=\n" \
      "Lk" X "%=:\n\ts_call_b64 s[84:85], Lflush%=\n\ts_branch Ls" X "%=\n" \
      ADV(X) \
      "Ln" X "%=:\n\ts_mov_b32 s73, 0\n\ts_cmp_lg_u32 " DY ", 0\n\ts_cbranch_scc1 Lm" X "%=\n\t" \
      TREE("s" X, X, DX, WS, LDN, POLL, "Lu" X "%=") \
      "Lm" X "%=:\n\t" TREE("m" X, X, DX, WM, LDN, "", "Lt" X "%=") \
      "Lu" X "%=:\n\t" TAIL(DX, "0", "Lidle%=") \
      "Lt" X "%=:\n\t" TAIL(DX, "s73", "Ls" Y "%=")
#define LDX(I) "v_ldexp_f32 v[4+" I "], v[4+" I "], s82\n\t"
#define LDX4(I) LDX("4*" I) LDX("4*" I "+1") LDX("4*" I "+2") LDX("4*" I "+3")
#define FW1(I) LDX4(I) "ds_write2st64_b32 v2, v[4+4*" I "], v[5+4*" I "] offset0:4*" I " offset1:4*" I "+1\n\t" \
      "ds_write2st64_b32 v2, v[6+4*" I "], v[7+4*" I "] offset0:4*" I "+2 offset1:4*" I "+3\n\t"
#define FW2(I) LDX4(I) "v_xor_b32 v20, 32*" I ", v2\n\tds_write_b128 v20, " VR("4", I) "\n\t"
#define ABORT "v_mov_b32 v20, %[ex]\n\tv_mov_b32 v21, 1\n\tds_write_b32 v20, v21\n\t"
#define STR_(X) #X
#define STR(X) STR_(X)
#define RWAIT(L) "s_xor_b32 s81, s94, %[tag]\n\ts_and_b32 s81, s81, 0xf0f0f0f0\n\ts_cbranch_scc0 Lgo%=\n\t" \
      "s_sleep 4\n\ts_add_u32 s86, s86, 1\n\ts_cmp_le_u32 s86, 0x200000\n\ts_cbranch_scc1 " L "\n\t" ABORT
#define DBGI(X)
      asm volatile(
        "v_mbcnt_lo_u32_b32 v0, -1, 0\n\tv_mbcnt_hi_u32_b32 v0, -1, v0\n\tv_lshl_add_u32 v1, v0, 4, %[lb]\n\tv_lshl_add_u32 v2, v0, 2, %[sl]\n\t"
        "s_mov_b32 s70, %[fo]\n\ts_mov_b32 s80, %[rb]\n\ts_mov_b32 s69, %[ab]\n\ts_mov_b32 s71, %[tl]\n\ts_mov_b32 s72, 0x80000\n\t"
        "s_mov_b32 s90, %[ts1]\n\ts_movk_i32 s91, 0xaa\n\ts_movk_i32 s92, 0x2000\n\ts_mov_b32 s93, 0\n\ts_mov_b64 s[86:87], 0\n\t" DBGI("s_mov_b32 s95, %[dbg]\n\t")
        "s_mov_b32 s96, s72\n\ts_mov_b32 s98, %[tag]\n\ts_mov_b32 s99, %[tag]\n\ts_cmp_eq_u32 s71, 0\n\ts_cbranch_scc1 Lend%=\n"
        "Lq%=:\n\ts_load_dword s94, %[ws], %[fx] glc\n\ts_waitcnt lgkmcnt(0)\n\t" RWAIT("Lq%=")
        "Lgo%=:\n\ts_and_b32 s94, s94, 0x0f0f0f0f\n\ts_add_u32 s81, s80, s92\n\ts_add_u32 s82, s81, s92\n\ts_add_u32 s83, s82, s92\n\t"
        "s_cmp_lg_u32 s93, 0\n\ts_cbranch_scc1 Lr%=\n\t"
        RES("0", "s80", VR("36", "3")) "Lr%=:\n\t" RES("1", "s81", VR("36", "7")) RES("2", "s82", VR("36", "11")) RES("3", "s83", "v[164:167]")
        POLL
        "s_lshr_b32 s68, s91, 8\n\ts_or_b32 s68, s68, s91\n\ts_mov_b64 s[76:77], 0\n\ts_mov_b64 s[78:79], 0\n\t" ZACC
        "s_waitcnt vmcnt(0)\n\t"
        XW("0") XW("1") XW("2") XW("3") XW("4") XW("5") XW("6") XW("7") XW("8") XW("9") XW("10") XW("11") XW("12") XW("13") XW("14")
        DBGST("s92", "17", "-832")
        "Lidle%=:\n\ts_waitcnt lgkmcnt(0)\n"
        "Lc%=:\n\t" PICK("0", "Lnone%=") LOAD8("36") "s_mov_b32 s76, s73\n\t"
        PEEK "s_cbranch_scc1 Ls36%=\n\ts_cmp_eq_u32 s68, 0\n\ts_cbranch_scc1 Ls36%=\n\t" POLL "s_waitcnt lgkmcnt(0)\n"
        STEP("36", "68", "s76", "s77")
        STEP("68", "36", "s77", "s76")
        ADV("0")
        "Lflush%=:\n\tv_readfirstlane_b32 s81, v3\n\ts_and_b32 s82, s78, 0xff\n\ts_lshl4_add_u32 s82, s82, -254\n\ts_cmp_lg_u32 s81, 0\n\ts_cbranch_scc1 Lw%=\n"
        "Lg%=:\n\ts_lshr_b32 s81, s78, 8\n\ts_add_u32 s81, s81, s79\n\ts_cmp_lg_u32 s93, 0\n\ts_cbranch_scc1 Lx%=\n\t"
        FW1("0") FW1("1") FW1("2") FW1("3") "s_branch Lz%=\n"
        "Lx%=:\n\t" FW2("0") FW2("1") FW2("2") FW2("3")
        "Lz%=:\n\tv_mov_b32 v3, s81\n\ts_mov_b64 exec, 1\n\tds_write_b32 v2, v3 offset:4096\n\ts_mov_b64 exec, -1\n\t"
        DBGST("s78", "8", "0") ZACC "s_mov_b32 s79, 0\n\ts_setpc_b64 s[84:85]\n"
        "Lw%=:\n\ts_sleep " STR(CSLEEP) "\n\tds_read_b32 v3, v2 offset:4096\n\ts_add_u32 s87, s87, 1\n\ts_waitcnt lgkmcnt(0)\n\tv_readfirstlane_b32 s81, v3\n\t"
        "s_cmp_eq_u32 s81, 0\n\ts_cbranch_scc1 Lg%=\n\ts_cmp_lt_u32 s87, 0x2000000\n\ts_cbranch_scc1 Lw%=\n\t" ABORT "s_branch Lg%=\n"
        "Lnone%=:\n\ts_cmp_eq_u32 s68, 0\n\ts_cbranch_scc1 Lend%=\n\ts_add_u32 s86, s86, 1\n\ts_cmp_gt_u32 s86, 0x200000\n\ts_cbranch_scc1 Lo%=\n\t"
        "s_sleep " STR(PSLEEP) "\n\t" POLL "s_branch Lidle%=\n"
        "Lo%=:\n\t" ABORT "s_mov_b64 s[64:65], s[98:99]\n\ts_branch Lc%=\n"
        "Lend%=:\n\ts_cmp_eq_u32 s93, %[rb2]\n\ts_cbranch_scc1 Lfin%=\n\t"
        "s_mov_b32 s93, %[rb2]\n\ts_mov_b32 s80, %[rb2]\n\ts_mov_b32 s69, %[ab2]\n\ts_mov_b32 s70, %[fo2]\n\ts_and_b32 s71, %[t2], 15\n\t"
        "s_andn2_b32 s72, %[t2], 15\n\ts_mov_b32 s96, %[ti]\n\t"
        "s_mov_b32 s90, 0x8000\n\ts_mov_b32 s91, %[m2]\n\ts_mov_b32 s92, %[ss]\n\t" DBGI("s_mov_b32 s95, %[dbg2]\n\t")
        "v_lshrrev_b32 v2, 5, v0\n\tv_xor_b32 v2, v2, v0\n\tv_and_b32 v2, 7, v2\n\tv_lshl_add_u32 v2, v2, 4, %[sl]\n\t"
        "v_and_b32 v3, 31, v0\n\tv_lshl_add_u32 v2, v3, 7, v2\n"
        "Lv%=:\n\ts_load_dword s81, %[ws], %[fw] offset:128 glc\n\ts_load_dword s82, %[ws], %[fw] offset:256 glc\n\ts_load_dword s83, %[ws], %[fw] offset:384 glc\n\t"
        "s_and_b32 s94, %[tag], 0xff\n\ts_waitcnt lgkmcnt(0)\n\ts_bfe_u32 s81, s81, %[sh]\n\ts_bfe_u32 s82, s82, %[sh]\n\ts_bfe_u32 s83, s83, %[sh]\n\t"
        "s_lshl_b32 s81, s81, 8\n\ts_lshl_b32 s83, s83, 8\n\ts_or_b32 s94, s94, s81\n\ts_or_b32 s82, s82, s83\n\ts_pack_ll_b32_b16 s94, s94, s82\n\t" RWAIT("Lv%=")
        "s_branch Lgo%=\n"
        "Lfin%=:\n\ts_endpgm"
        :
        : [rs] "s"(rsX), [ws] "s"(myws), [tag] "s"(tag4), [ex] "s"(lds_addr(&sm.abort_)), [lb] "s"(lb), [sl] "s"(lds_addr(&sm.slot[v])),
          [fx] "s"(fbase + FX_REL + (ew ? (rg - 16u) * 128u : rg * 128u + v * 4u)), [rb] "s"(rb), [fo] "s"(fbase + (ew ? FE_REL + cls * 128u : (cls * 8u + v) * 128u)), [ab] "s"(ab - 4096u), [tl] "s"(latecu && v - 1u < 3u ? 0 : nt),
          [fw] "s"(fbase + wflag(g2, 0u)), [sh] "s"(0x80000u + 8u * (v & 3u)), [rb2] "s"(half >= og ? dbase + W_OFF + (g2 * 4u + (v & 3u)) * PIECE : 0u),
          [fo2] "s"(fbase + FZ_REL + hq * 128u), [ab2] "s"(dbase + hq * 4u * LH * PIECE - 4096u),
          [t2] "s"((((unsigned)nt + 1u + og) << 19) + LH - OOO), [m2] "n"(0xa8 << (8 * OOO)), [ti] "s"(0x80000u << half),
          [ts1] "n"(NCLS * 4u * PIECE), [ss] "n"(SRC_STRIDE)
        : "memory", "scc", "v167",
          "s64", "s65", "s66", "s67", "s68", "s69", "s70", "s71", "s72", "s73", "s74", "s75", "s76", "s77", "s78", "s79",
          "s80", "s81", "s82", "s83", "s84", "s85", "s86", "s87", "s88", "s89", "s90", "s91", "s92", "s93", "s94", "s95", "s96", "s98", "s99");
  } else {
    const bool own = rg < 16u;
    const unsigned o = own ? rank : ((4u * (h ^ 1u) + ((rg >> 2) & 3u)) & 7u);
    const rsrc_t rsO = mk(a.out, OUT_BYTES);
    const rsrc_t rsD = mk(a.wsd[(o - rank - 1u) & 7u], WS_USED);
    const int lr = (lane >> 4) + ((lane >> 3) & 1) * 4;
    unsigned* const sp = lane < 8 ? &sm.slot[lane].st : &sm.abort_;
    unsigned spins = 0, sv;
    auto await = [&](unsigned lo, unsigned hi) {
      __builtin_amdgcn_s_setprio(0);
      while (true) {
        sv = __hip_atomic_load(sp, __ATOMIC_RELAXED, __HIP_MEMORY_SCOPE_WORKGROUP);
        if (__builtin_amdgcn_ballot_w64((sv >> 3) - lo <= hi - lo && (((sv >> 3) ^ lo) & 1u) == 0u) != 0ull) break;
        if ((++spins & 1023u) == 0u && now() - t0 > 3 * DEADLINE) {
          lds_st(&sm.abort_, 1u);
          if (lane == 0) __builtin_amdgcn_raw_buffer_store_b32(0x7fc00000u, mk(a.out, OUT_BYTES), 0, 0, AUX_SYS);
          __builtin_amdgcn_endpgm();
        }
        __builtin_amdgcn_s_sleep(CSLEEP);
      }
      __builtin_amdgcn_s_setprio(EPRIO);
      asm volatile("" ::: "memory");
    };
    auto absorb = [&](bool mine, f4 (&s)[4], unsigned& need) {
      unsigned mm = (unsigned)__builtin_amdgcn_ballot_w64(mine);
      while (mm != 0u) {
        f4 p[2][4];
        const int v1 = __builtin_ctz(mm);
        mm &= mm - 1u;
#pragma unroll
        for (int i = 0; i < 4; ++i) p[0][i] = ((const f4*)sm.slot[v1].d)[i * 64 + lane];
        need -= __builtin_amdgcn_readlane(sv, v1) & 7u;
        if (mm != 0u) {
          const int v2 = __builtin_ctz(mm);
          mm &= mm - 1u;
#pragma unroll
          for (int i = 0; i < 4; ++i) p[1][i] = ((const f4*)sm.slot[v2].d)[i * 64 + lane];
          need -= __builtin_amdgcn_readlane(sv, v2) & 7u;
#pragma unroll
          for (int i = 0; i < 4; ++i) p[0][i] += p[1][i];
        }
#pragma unroll
        for (int i = 0; i < 4; ++i) s[i] += p[0][i];
      }
      asm volatile("" ::: "memory");
      if (mine) lds_st(sp, 0u);
      if (__builtin_amdgcn_readlane(sv, 8) != 0u) s[0] = (f4){__builtin_nanf(""), 0.f, 0.f, 0.f};
    };
    unsigned ts = (unsigned)wv - 7u;
    for (; ts <= (unsigned)nt; ts += 2u) {
      f4 s[4], se[4];
#pragma unroll
      for (int i = 0; i < 4; ++i) s[i] = se[i] = (f4){0.f, 0.f, 0.f, 0.f};
      const unsigned g = (ts - 1u) * NCLS + cls;
      const unsigned yi = (call & 1u) * 512u + (o & 3u) * 128u + (g >> 1) * 4u + (rg & 3u);
      unsigned need = latecu ? 20u : own ? 32u : 28u, neede = own ? 0u : 4u, efo = WS_USED;
      do {
        await(ts, ts);
        absorb((sv >> 3) == ts && (unsigned)lane != ve, s, need);
        if (efo != WS_USED && need != 0u) {
          asm volatile("s_waitcnt vmcnt(0)" ::: "memory");
          if (lane == 0) __builtin_amdgcn_raw_buffer_store_b32(call, rsMy, 0, efo, AUX_SYS);
          efo = WS_USED;
        }
        if ((__builtin_amdgcn_readlane(sv, ve) >> 3) == ts) {
          absorb((unsigned)lane == ve, se, neede);
          if (neede == 0u) {
#pragma unroll
            for (int i = 0; i < 4; ++i) __builtin_amdgcn_raw_buffer_store_b128(__builtin_bit_cast(u4, se[i]), rsO, lr * 16384 + (lane & 7) * 16, (32u * (rg - 16u) + (unsigned)i * 8u) * 16384u + (2048u * (h ^ 1u) + 32u * g) * 4u, AUX_SYS);
            efo = YRDY_OFF + yi * 16u + 8u + (g & 1u) * 4u;
          }
        }
      } while ((need | neede) != 0u);
      unsigned yfo = WS_USED;
      if (!own) {
#pragma unroll
        for (int i = 0; i < 4; ++i) __builtin_amdgcn_raw_buffer_store_b128(__builtin_bit_cast(u4, s[i]), rsMy, lr * 256 + (lane & 7) * 16, YST_OFF + yi * PIECE + (g & 1u) * 128u + (unsigned)i * 2048u, AUX_SYS);
        yfo = YRDY_OFF + ((rank & 3u) * 128u + (yi & 0x27fu)) * 16u + (g & 1u) * 4u;
      } else {
        const unsigned so = 32u * rg * 16384u + (2048u * h + 32u * g) * 4u;
#pragma unroll
        for (int i = 0; i < 4; ++i) __builtin_amdgcn_raw_buffer_store_b128(__builtin_bit_cast(u4, s[i]), rsO, lr * 16384 + (lane & 7) * 16, so + (unsigned)i * 8u * 16384u, AUX_SYS);
        if (latecu) yfo = PRDY_OFF + (lp * 64u + g) * 4u;
      }
      STAMP(95 + ts);
      if (yfo != WS_USED) {
        asm volatile("s_waitcnt vmcnt(0)" ::: "memory");
        if (!own && g < JD / 2u) {
          if (lane == 0) __builtin_amdgcn_raw_buffer_store_b8((unsigned char)tag_of(call), rsMy, 0, MIR_OFF + (call & 1u) * FLAG_PAR + 2048u + (yi & 0x1ffu) * 2u + (g & 1u), AUX_SYS);
        } else if (lane == 0) __builtin_amdgcn_raw_buffer_store_b32(call, rsD, 0, yfo, AUX_SYS);
        if (lane == 0 && efo != WS_USED) __builtin_amdgcn_raw_buffer_store_b32(call, rsMy, 0, efo, AUX_SYS);
      }
    }
    const unsigned u = ts - 1u - (unsigned)nt, l0 = half ? 2u + u : 4u * u + (cidx & 1u), ls = 8u >> half;
    f4 s[LH][4];
    unsigned need[LH], left = 0;
#define SO2(K) (((4u * rank + 16u - NL + l0 + ls * (K)) & 15u) * 32u * 16384u + (2048u * h + 32u * g2) * 4u)
#pragma unroll
    for (unsigned k = 0; k < LH; ++k) {
      need[k] = 12u;
      if (l0 + ls * k < NL) {
        ++left;
        while (poll1(myws, PRDY_OFF + ((l0 + ls * k) * 64u + g2) * 4u) != call) {
          if ((++spins & 63u) == 0u && now() - t0 > 3 * DEADLINE) { lds_st(&sm.abort_, 1u); break; }
          __builtin_amdgcn_s_sleep(4);
        }
      }
#pragma unroll
      for (int i = 0; i < 4; ++i) s[k][i] = __builtin_bit_cast(f4, __builtin_amdgcn_raw_buffer_load_b128(rsO, vo2, l0 + ls * k < NL ? SO2(k) + (unsigned)i * 8u * 16384u : OUT_BYTES, AUX_LD));
    }
    while (left != 0u) {
      await(ts, ts + 2u * LH);
#pragma unroll
      for (unsigned k = 0; k < LH; ++k) {
        if (__builtin_amdgcn_ballot_w64((sv >> 3) == ts + 2u * k) == 0ull) continue;
        absorb((sv >> 3) == ts + 2u * k, s[k], need[k]);
        if (need[k] != 0u) continue;
#pragma unroll
        for (int i = 0; i < 4; ++i) __builtin_amdgcn_raw_buffer_store_b128(__builtin_bit_cast(u4, s[k][i]), rsO, vo2, SO2(k) + (unsigned)i * 8u * 16384u, AUX_SYS);
        --left;
        STAMP(95 + LNN((int)(ts + 2u * k)));
      }
    }
  }
}

extern "C" void kernel_launch(int rank, int nranks, ncclComm_t comm,
                              hipStream_t stream,
                              void* const* d_in, const int* in_sizes, int n_in,
                              void* d_out, int out_size,
                              const int* in_dtypes, int out_dtype,
                              void* const* d_out_all,
                              void* const* d_ws_all, size_t ws_bytes) {
  (void)comm; (void)in_dtypes; (void)out_dtype; (void)d_out_all;
  if (nranks != NR || rank < 0 || rank >= NR || n_in < 2 || ws_bytes < WS_USED ||
      in_sizes[0] != 4096 * 512 || in_sizes[1] != 512 * 4096 || out_size != 512 * 4096) {
    fprintf(stderr, "kernel_launch: unsupported configuration\n");
    return;
  }
  Args a;
  a.x = (const float*)d_in[0];
  a.w = (const float*)d_in[1];
  for (int p = 0; p < NR; ++p) a.wsd[p] = (char*)d_ws_all[(rank + 1 + p) & 7];
  a.out = (float*)d_out;
  for (unsigned n = 0; n < 128u; ++n) {
    const unsigned jb = n >> 4, i = (n & 1u) * 8u + ((n >> 1) & 7u), p = ((unsigned)rank + 1u + jb) & 7u;
    const bool isx = ((p ^ (unsigned)rank) & 4u) == 0u;
    const unsigned idx = isx ? xitem(p, i) : i * 4u + ((jb + 1u) & 3u);
    a.tab[n] = piece_src(isx, isx ? xgroup(p, idx) : (p & 4u) * 16u + idx, isx ? i & 3u : 0u) | (idx << 24) | (isx ? 0u : 0x80000000u);
  }
  a.myws = (char*)d_ws_all[rank];
  a.rank = (unsigned)rank;
  a.pad_ = 0u;
  hipLaunchKernelGGL(fused_gemm_kernel, dim3(256), dim3(NTHREADS), 0, stream, a);
  HIP_CHECK(hipGetLastError());
}
```
